```python
import numpy as np

kernel_name = "gpu_gemm_rs_m4096_k4096_n32768_bf16_silu"


ABSMAX_RTOL = 0.008

COMPUTE_DTYPE = "bfloat16"
EPILOGUE = "silu"
IN_QUANT = None
OUT_QUANT = None
IN_DTYPES = {"x": "bfloat16", "w_mat": "bfloat16"}
WORLD_SIZE = 8

def _to_bf16(x):
    u = np.ascontiguousarray(x, dtype=np.float32).view(np.uint32)
    return ((u + 0x7FFF + ((u >> 16) & 1)) & 0xFFFF0000).view(np.float32)

def setup_inputs():

    x = _to_bf16(np.random.randn(4096, 4096).astype(np.float32))
    w_mat = _to_bf16((np.random.randn(4096, 32768) / np.sqrt(4096)).astype(np.float32))
    return {"x": x, "w_mat": w_mat}

def _epilogue(y):
    return (y / (1.0 + np.exp(-np.clip(y, -60.0, 60.0)))).astype(np.float32)

def reference(x, w_mat):
    y = (x @ w_mat).astype(np.float32)
    return _epilogue(y)
```

```cpp
#include <hip/hip_runtime.h>
#include <rccl/rccl.h>
#include <cstdio>

#define HIP_CHECK(expr) do { hipError_t _e = (expr); if (_e != hipSuccess) { \
    fprintf(stderr, "HIP error %s at %s:%d\n", hipGetErrorString(_e), __FILE__, __LINE__); \
    return; } } while (0)

#define LAS __attribute__((address_space(3)))
typedef short s16x4 __attribute__((ext_vector_type(4)));
typedef short bf16x8 __attribute__((ext_vector_type(8)));
typedef float f32x4 __attribute__((ext_vector_type(4)));
typedef unsigned u32x4 __attribute__((ext_vector_type(4)));

namespace {
constexpr int NR = 8;
constexpr int GM = 4096, GK = 4096, GN = 32768;
constexpr int KS = GK / NR;
constexpr int NB = GN / NR;
constexpr int MS = GM / NR;
constexpr unsigned XP = KS * 2;
constexpr unsigned WP = GN * 2;
constexpr int HTB = 128 * 64 * 2;
constexpr int STAGE_BYTES = 8 * HTB;
constexpr int LDS_BYTES = STAGE_BYTES + 64;
constexpr int GRID = 256;
#define CPX 4
constexpr int NCOMM = 8 * CPX;
constexpr unsigned long long DEADLINE = 50000000ull;

constexpr size_t OFF_CNT = 0;
constexpr size_t OFF_TMO = 32768;
constexpr size_t OFF_FX = 65536;
constexpr size_t OFF_FW = OFF_FX + 8192;
constexpr size_t OFF_FO = OFF_FW + 8192;
constexpr size_t OFF_DBG = 131072;
constexpr size_t OFF_XS = 1u << 20;
constexpr size_t XS_PAR = (size_t)NR * GM * XP;
constexpr size_t OFF_WS = OFF_XS + 2 * XS_PAR;
constexpr size_t WS_PAR = (size_t)KS * WP;
constexpr size_t WS_NEED = OFF_WS + 2 * WS_PAR;

struct Params {
    const char* x; const char* w;
    char* ws[NR]; char* out[NR];
    int rank;
    unsigned sched[128];
};

__device__ __forceinline__ int lds_byte(int r, int c) { const int st = (r >> 4) * 2 + (c >> 5), rr = r & 15, cc = c & 31, ob = rr * 64 + cc * 2; return st * 1024 + (ob ^ (((ob >> 9) & 1) << 5)); }
__device__ __forceinline__ void stage_rc(int b, int& R, int& C) { const int st = b / 1024, sb = b % 1024, swz = sb ^ (((sb >> 9) & 1) << 5); R = (st >> 1) * 16 + swz / 64; C = (st & 1) * 32 + (swz % 64) / 2; }
__device__ __forceinline__ __amdgpu_buffer_rsrc_t mk(const void* p, unsigned bytes) { return __builtin_amdgcn_make_buffer_rsrc((void*)p, 0, bytes, 0x00020000); }
__device__ __forceinline__ unsigned cvt_pk_bf16(float lo, float hi) { unsigned r; asm volatile("v_cvt_pk_bf16_f32 %0, %1, %2" : "=v"(r) : "v"(lo), "v"(hi)); return r; }
__device__ __forceinline__ float silu(float y) {
    const float c = __builtin_amdgcn_fmed3f(y, -60.0f, 60.0f);
    const float e = __builtin_amdgcn_exp2f(c * -1.44269504088896341f);
    return y * __builtin_amdgcn_rcpf(1.0f + e);
}
__device__ __forceinline__ unsigned ld_agent(const unsigned* p) { return __hip_atomic_load(p, __ATOMIC_RELAXED, __HIP_MEMORY_SCOPE_AGENT); }
__device__ __forceinline__ unsigned long long ld_sys64(const void* p) { return __hip_atomic_load((const unsigned long long*)p, __ATOMIC_RELAXED, __HIP_MEMORY_SCOPE_SYSTEM); }
__device__ __forceinline__ void st_sys(void* p, unsigned v) { __hip_atomic_store((unsigned*)p, v, __ATOMIC_RELAXED, __HIP_MEMORY_SCOPE_SYSTEM); }
__device__ __forceinline__ int panel_of(int u, int d) { return u < 2 ? 2 * d + u : (u < 9 ? 2 * ((d + u - 1) & 7) : 2 * ((d + u - 8) & 7) + 1); }

#define DBG_STAMP(k) do { } while (0)

constexpr int SAUX = 17;

struct Task { __amdgpu_buffer_rsrc_t rs, rd; unsigned stride; unsigned voff; char* flag; };

__device__ __forceinline__ void comm_role(const Params& P, int cw, unsigned call) {
    const int lane = threadIdx.x & 63, wid = __builtin_amdgcn_readfirstlane(threadIdx.x >> 6);
    const int g = P.rank;
    char* myws = P.ws[g];
    if (wid < 7) {
        const int d = (g + wid + 1) & 7, q = 7 - wid;
        char* dws = P.ws[d];
        char* dxs = dws + OFF_XS + (call & 1) * XS_PAR + (size_t)q * GM * XP;
        char* dwsb = dws + OFF_WS + (call & 1) * WS_PAR + (size_t)q * NB * 2;
        const unsigned vx = (lane >> 3) * XP + (lane & 7) * 16, vw = (lane >> 4) * WP + (lane & 15) * 16;
        auto desc = [&](int s, Task& t) {
            if (s < 512) {
                const unsigned e = (P.sched[s >> 2] >> (((s >> 1) & 1) * 16)) & 0xffffu;
                const int c = e & 7, idx = (e >> 3) & 15, h = s & 1;
                if (e & 128) {
                    const int pn = panel_of(idx, d);
                    const size_t o = (size_t)(pn * 256 + h * 128) * XP + c * 128;
                    t.rs = mk(P.x + o, 127 * XP + 128); t.rd = mk(dxs + o, 127 * XP + 128); t.stride = 8 * XP; t.voff = vx;
                    t.flag = dws + OFF_FX + ((pn * 64 + c * 8 + q) * 2 + h) * 4;
                } else {
                    const size_t o = (size_t)c * 64 * WP + idx * 512 + h * 256;
                    t.rs = mk(P.w + o + (size_t)d * NB * 2, 63 * WP + 256); t.rd = mk(dwsb + o, 63 * WP + 256); t.stride = 4 * WP; t.voff = vw;
                    t.flag = dws + OFF_FW + ((idx * 64 + c * 8 + q) * 2 + h) * 4;
                }
            } else { t.rs = mk(P.x, 0); t.rd = mk(myws, 0); t.stride = 0; t.voff = 0; t.flag = nullptr; }
        };
        Task cur, nxt; char* pflag = nullptr;
        desc(cw, cur);
        u32x4 a0, a1, b0, b1;
        a0 = __builtin_amdgcn_raw_buffer_load_b128(cur.rs, cur.voff, 0, 0);
        a1 = __builtin_amdgcn_raw_buffer_load_b128(cur.rs, cur.voff, cur.stride, 0);
        {
            const u32x4 z = {0u, 0u, 0u, 0u}; const __amdgpu_buffer_rsrc_t rn = mk(myws, 0);
            __builtin_amdgcn_sched_barrier(0);
            __builtin_amdgcn_raw_buffer_store_b128(z, rn, cur.voff, 0, SAUX); __builtin_amdgcn_raw_buffer_store_b128(z, rn, cur.voff, 16, SAUX);
            __builtin_amdgcn_sched_barrier(0);
        }
        for (int s = cw; s < 512; s += NCOMM) {
            desc(s + NCOMM, nxt);
#pragma unroll
            for (int it = 0; it < 8; it += 2) {
                b0 = __builtin_amdgcn_raw_buffer_load_b128(cur.rs, cur.voff, (2 * it + 2) * cur.stride, 0);
                b1 = __builtin_amdgcn_raw_buffer_load_b128(cur.rs, cur.voff, (2 * it + 3) * cur.stride, 0);
                __builtin_amdgcn_sched_barrier(0);
                __builtin_amdgcn_raw_buffer_store_b128(a0, cur.rd, cur.voff, (2 * it) * cur.stride, SAUX);
                __builtin_amdgcn_raw_buffer_store_b128(a1, cur.rd, cur.voff, (2 * it + 1) * cur.stride, SAUX);
                __builtin_amdgcn_sched_barrier(0);
                if (it + 2 < 8) {
                    a0 = __builtin_amdgcn_raw_buffer_load_b128(cur.rs, cur.voff, (2 * it + 4) * cur.stride, 0);
                    a1 = __builtin_amdgcn_raw_buffer_load_b128(cur.rs, cur.voff, (2 * it + 5) * cur.stride, 0);
                } else {
                    a0 = __builtin_amdgcn_raw_buffer_load_b128(nxt.rs, nxt.voff, 0, 0);
                    a1 = __builtin_amdgcn_raw_buffer_load_b128(nxt.rs, nxt.voff, nxt.stride, 0);
                }
                __builtin_amdgcn_sched_barrier(0);
                __builtin_amdgcn_raw_buffer_store_b128(b0, cur.rd, cur.voff, (2 * it + 2) * cur.stride, SAUX);
                __builtin_amdgcn_raw_buffer_store_b128(b1, cur.rd, cur.voff, (2 * it + 3) * cur.stride, SAUX);
                __builtin_amdgcn_sched_barrier(0);
                if (it == 0 && pflag) {
                    asm volatile("s_waitcnt vmcnt(8)" ::: "memory");
                    if (lane == 0) st_sys(pflag, call);
                    asm volatile("" ::: "memory");
                }
            }
            pflag = cur.flag; cur = nxt;
        }
        asm volatile("s_waitcnt vmcnt(0)" ::: "memory");
        if (lane == 0) st_sys(pflag, call);
        DBG_STAMP(1);
    } else if (cw == 0) {
        const char* fo = myws + OFF_FO + lane * 16;
        unsigned* tmo = (unsigned*)(myws + OFF_TMO);
        const unsigned long long t0 = __builtin_amdgcn_s_memrealtime();
        bool dead = false;
        for (;;) {
            const unsigned long long v0 = ld_sys64(fo), v1 = ld_sys64(fo + 8);
            const bool ok = lane < 8 || ((unsigned)v0 >= call && (unsigned)(v0 >> 32) >= call && (unsigned)v1 >= call && (unsigned)(v1 >> 32) >= call);
            if (__all(ok)) break;
            if (__builtin_amdgcn_readfirstlane(ld_agent(tmo)) == call || __builtin_amdgcn_s_memrealtime() - t0 > DEADLINE) { dead = true; break; }
            __builtin_amdgcn_s_sleep(4);
        }
        __builtin_amdgcn_fence(__ATOMIC_ACQUIRE, "");
        if (dead && lane == 0) {
            __hip_atomic_store(tmo, call, __ATOMIC_RELAXED, __HIP_MEMORY_SCOPE_AGENT);
            *(unsigned short*)(P.out[g] + (size_t)g * NB * 2) = 0x7fc0;
        }
        DBG_STAMP(1);
    }
}

__device__ __forceinline__ void gemm_tile(const Params& P, LAS unsigned char* lds, int pm, int pn, unsigned call) {
    const int tid = threadIdx.x, wid = __builtin_amdgcn_readfirstlane(tid >> 6), lane = tid & 63, wr = wid >> 2, wc = wid & 3, fr = lane & 15, fq = lane >> 4;
    const int g = P.rank;
    char* myws = P.ws[g];
    unsigned voffA[2], voffB[2];
#pragma unroll
    for (int i = 0; i < 2; ++i) {
        int R, C; stage_rc(tid * 16 + i * 8192, R, C); voffA[i] = (unsigned)R * XP + (unsigned)C * 2u;
        const int row = lane >> 3, ch = (lane & 7) ^ (((row >> 1) & 1) << 2);
        voffB[i] = (unsigned)(((wid >> 1) + 4 * i) * 8 + row) * WP + (unsigned)(wid & 1) * 128u + (unsigned)ch * 16u;
    }
    const unsigned ldsw = (unsigned)wid * 1024u;
    const int aoff = lds_byte(wr * 64 + fr, fq * 8);
    const int boff = fq * 2048 + (wc >> 1) * 1024 + (fr >> 2) * 128 + ((4 * (wc & 1) + (fr & 3)) ^ (((fr >> 3) & 1) << 2)) * 16;
    constexpr size_t hA = (size_t)128 * XP, hB = 256;
    const char* xl = P.x + (size_t)pm * 256 * XP;
    const char* wl = P.w + (size_t)g * NB * 2 + pn * 512;
    const char* xs = myws + OFF_XS + (call & 1) * XS_PAR + (size_t)pm * 256 * XP;
    const char* wsb = myws + OFF_WS + (call & 1) * WS_PAR + pn * 512;
    auto ktile = [&](int t, const char*& a, const char*& b, int& bit) {
        if (t < 8) { a = xl + t * 128; b = wl + (size_t)t * 64 * WP; bit = 0; }
        else { const unsigned u = t - 8, c = u / 7u, q = u - c * 7u + 1u; a = xs + (size_t)q * GM * XP + c * 128; b = wsb + (size_t)c * 64 * WP + (size_t)q * NB * 2; bit = c * 8 + q; }
    };
#define SA(b, h) (((b) * 2 + (h)) * HTB)
#define SB(b, h) ((4 + (b) * 2 + (h)) * HTB)
#define STAGE(bufoff, gbase, voff) do { _Pragma("unroll") for (int _i = 0; _i < 2; ++_i) \
        __builtin_amdgcn_global_load_lds((const unsigned*)((const char*)(gbase) + (voff)[_i]), (LAS unsigned*)(lds + (bufoff) + ldsw + _i * 8192), 16, 0, 0); } while (0)
#define LDA(dst, b, h) do { _Pragma("unroll") for (int m = 0; m < 4; ++m) _Pragma("unroll") for (int k = 0; k < 2; ++k) dst[m][k] = *(const LAS bf16x8*)(lds + SA(b, h) + aoff + m * 2048 + k * 1024); } while (0)
#define TRD(dst, off) asm volatile("ds_read_b64_tr_b16 %0, %1 offset:%2" : "=v"(dst) : "v"(baddr), "i"(off) : "memory")
#define LDB(dst, b, h) do { _Pragma("unroll") for (int n = 0; n < 2; ++n) _Pragma("unroll") for (int k = 0; k < 2; ++k) { \
        TRD(dst[n][k][0], ((b) * 2 + (h)) * HTB + k * 8192 + n * 8); TRD(dst[n][k][1], ((b) * 2 + (h)) * HTB + k * 8192 + 512 + n * 8); } } while (0)
#define WAIT_B(Bt) asm volatile("s_waitcnt lgkmcnt(0)" : "+v"(Bt[0][0][0]), "+v"(Bt[0][0][1]), "+v"(Bt[0][1][0]), "+v"(Bt[0][1][1]), "+v"(Bt[1][0][0]), "+v"(Bt[1][0][1]), "+v"(Bt[1][1][0]), "+v"(Bt[1][1][1]) :: "memory")
#define BF(Bt, n, k) __builtin_shufflevector(Bt[n][k][0], Bt[n][k][1], 0, 1, 2, 3, 4, 5, 6, 7)
#define MMA(ai, bj, At, Bt) do { __builtin_amdgcn_s_setprio(1); _Pragma("unroll") for (int m = 0; m < 4; ++m) _Pragma("unroll") for (int n = 0; n < 2; ++n) _Pragma("unroll") for (int k = 0; k < 2; ++k) \
        acc[ai][bj][m][n] = __builtin_amdgcn_mfma_f32_16x16x32_bf16(BF(Bt, n, k), At[m][k], acc[ai][bj][m][n], 0, 0, 0); __builtin_amdgcn_s_setprio(0); } while (0)
#define WAIT_V(n) asm volatile("s_waitcnt vmcnt(" #n ")" ::: "memory")
#define WAIT_L(n) asm volatile("s_waitcnt lgkmcnt(" #n ")" ::: "memory")
#define BAR __builtin_amdgcn_s_barrier()
#define SCHED __builtin_amdgcn_sched_barrier(0)
    f32x4 acc[2][2][4][2];
#pragma unroll
    for (int a = 0; a < 2; ++a)
#pragma unroll
        for (int b = 0; b < 2; ++b)
#pragma unroll
            for (int m = 0; m < 4; ++m)
#pragma unroll
                for (int n = 0; n < 2; ++n) acc[a][b][m][n] = (f32x4){0.f, 0.f, 0.f, 0.f};
    bf16x8 At[4][2]; s16x4 B0[2][2][2], B1[2][2][2];
    const unsigned baddr = (unsigned)(size_t)lds + 4 * HTB + boff;
    unsigned long long rdy = 1ull;
    LAS unsigned long long* rdy_lds = (LAS unsigned long long*)(lds + STAGE_BYTES);
    STAGE(SB(0, 0), wl, voffB); STAGE(SA(0, 0), xl, voffA); STAGE(SB(0, 1), wl + hB, voffB); STAGE(SA(0, 1), xl + hA, voffA);
    if (wr == 1) BAR;
    WAIT_V(4); BAR;
    STAGE(SB(1, 0), wl + (size_t)64 * WP, voffB); STAGE(SA(1, 0), xl + 128, voffA); STAGE(SB(1, 1), wl + (size_t)64 * WP + hB, voffB);
    WAIT_V(6); BAR;
    const char* a1 = xl + 128;
    for (int t = 0; t < 64; t += 2) {
        const char *a2, *b2, *a3, *b3; int bit2, bit3;
        ktile((t + 2) & 63, a2, b2, bit2); ktile((t + 3) & 63, a3, b3, bit3);
        const unsigned long long need = (1ull << bit2) | (1ull << bit3);
        if ((rdy & need) != need) {
            if (wid == 0) {
                const char* fx = myws + OFF_FX + (pm * 64 + lane) * 8; const char* fw = myws + OFF_FW + (pn * 64 + lane) * 8;
                unsigned* tmo = (unsigned*)(myws + OFF_TMO);
                const unsigned long long t0 = __builtin_amdgcn_s_memrealtime();
                unsigned long long mask;
                for (;;) {
                    const unsigned long long vx = ld_sys64(fx), vw = ld_sys64(fw);
                    const bool ok = (lane & 7) == 0 || ((unsigned)vx >= call && (unsigned)(vx >> 32) >= call && (unsigned)vw >= call && (unsigned)(vw >> 32) >= call);
                    mask = __ballot(ok);
                    if ((mask & need) == need) break;
                    if (__builtin_amdgcn_readfirstlane(ld_agent(tmo)) == call || __builtin_amdgcn_s_memrealtime() - t0 > DEADLINE) {
                        if (lane == 0) __hip_atomic_store(tmo, call, __ATOMIC_RELAXED, __HIP_MEMORY_SCOPE_AGENT);
                        mask = ~0ull; break; }
                    __builtin_amdgcn_s_sleep(8);
                }
                __builtin_amdgcn_fence(__ATOMIC_ACQUIRE, "");
                if (lane == 0) *rdy_lds = mask;
                asm volatile("s_waitcnt vmcnt(0) lgkmcnt(0)" ::: "memory");
            }
            asm volatile("" ::: "memory"); BAR; asm volatile("" ::: "memory");
            const unsigned long long mv = *rdy_lds;
            rdy = ((unsigned long long)(unsigned)__builtin_amdgcn_readfirstlane((unsigned)(mv >> 32)) << 32) | (unsigned)__builtin_amdgcn_readfirstlane((unsigned)mv);
        }
        LDB(B0, 0, 0); SCHED; LDA(At, 0, 0); STAGE(SA(1, 1), a1 + hA, voffA);
        WAIT_L(8); BAR; WAIT_B(B0); MMA(0, 0, At, B0); BAR; SCHED;
        LDB(B1, 0, 1); STAGE(SB(0, 0), b2, voffB);
        BAR; WAIT_B(B1); MMA(0, 1, At, B1); BAR;
        LDA(At, 0, 1); STAGE(SA(0, 0), a2, voffA);
        BAR; WAIT_L(0); MMA(1, 0, At, B0); BAR; SCHED;
        STAGE(SB(0, 1), b2 + hB, voffB);
        WAIT_V(6); BAR; MMA(1, 1, At, B1); BAR;
        LDB(B0, 1, 0); SCHED; LDA(At, 1, 0); STAGE(SA(0, 1), a2 + hA, voffA);
        WAIT_L(8); BAR; WAIT_B(B0); MMA(0, 0, At, B0); BAR; SCHED;
        LDB(B1, 1, 1); STAGE(SB(1, 0), b3, voffB);
        BAR; WAIT_B(B1); MMA(0, 1, At, B1); BAR;
        LDA(At, 1, 1); STAGE(SA(1, 0), a3, voffA);
        BAR; WAIT_L(0); MMA(1, 0, At, B0); BAR; SCHED;
        STAGE(SB(1, 1), b3 + hB, voffB);
        WAIT_V(6); BAR; MMA(1, 1, At, B1); BAR;
        a1 = a3;
    }
    DBG_STAMP(2);
    const int dest = pm >> 1;
    const bool bad = __builtin_amdgcn_readfirstlane(ld_agent((const unsigned*)(myws + OFF_TMO))) == call;
    {
        int tz = threadIdx.x; asm volatile("" : "+v"(tz));
        const int l2 = tz & 63, fr2 = l2 & 15, fq2 = l2 >> 4;
        const __amdgpu_buffer_rsrc_t ro = mk(P.out[dest] + (size_t)(pm & 1) * 256 * WP + (size_t)g * NB * 2 + pn * 512, 255 * WP + 512);
        const unsigned vo = (unsigned)(wr * 64 + fr2) * WP + (unsigned)(wc * 32 + 8 * fq2) * 2u;
#pragma unroll
        for (int ai = 0; ai < 2; ++ai)
#pragma unroll
            for (int m = 0; m < 4; ++m)
#pragma unroll
                for (int bj = 0; bj < 2; ++bj) {
                    const f32x4 v0 = acc[ai][bj][m][0], v1 = acc[ai][bj][m][1];
                    u32x4 o; o.x = cvt_pk_bf16(silu(v0[0]), silu(v0[1])); o.y = cvt_pk_bf16(silu(v0[2]), silu(v0[3]));
                    o.z = cvt_pk_bf16(silu(v1[0]), silu(v1[1])); o.w = cvt_pk_bf16(silu(v1[2]), silu(v1[3]));
                    if (bad) o = (u32x4){0x7fc07fc0u, 0x7fc07fc0u, 0x7fc07fc0u, 0x7fc07fc0u};
                    __builtin_amdgcn_raw_buffer_store_b128(o, ro, vo, (unsigned)(ai * 128 + m * 16) * WP + bj * 256, SAUX);
                }
    }
    WAIT_V(0);
    if (wr == 0) BAR;
    BAR;
    if (tid == 0 && dest != g) st_sys(P.ws[dest] + OFF_FO + ((((g - dest) & 7) * 32) + (pm & 1) * 16 + pn) * 4, call);
    DBG_STAMP(3);
#undef SA
#undef SB
#undef STAGE
#undef LDA
#undef LDB
#undef TRD
#undef WAIT_B
#undef BF
#undef MMA
}

__global__ __launch_bounds__(512, 2) void fused_gemm_rs(const Params P) {
    extern __shared__ __attribute__((aligned(16))) unsigned char shm[];
    const int b = blockIdx.x, xcd = b & 7, slot = b >> 3;
    unsigned* cnt = (unsigned*)(P.ws[P.rank] + OFF_CNT + (size_t)b * 128);
    const unsigned call = __builtin_amdgcn_readfirstlane(ld_agent(cnt)) + 1u;
    DBG_STAMP(0);
    if (slot < CPX) {
        comm_role(P, slot * 8 + xcd, call);
        __syncthreads();
    } else {
        const int g = P.rank, s = slot - 4;
        const int pn = 2 * xcd + (s & 1), qq = s >> 1;
        int nl = -1;
        if (s < 0) nl = 3;
        else if (qq >= 7) { const int k = (qq - 7 + 14 - 2 * xcd) % 7; if (k < 2 && (2 * k + (s & 1)) < CPX) nl = 2 * k + (s & 1); }
        for (int u = (nl >= 0 ? 0 : 1); u < (s < 0 ? 1 : 2); ++u) {
            int pm, pc;
            if (u == 0) { pm = 2 * g + (nl >> 1); pc = 2 * xcd + (nl & 1); }
            else { pm = qq < 7 ? 2 * ((g + 1 + qq) & 7) : 2 * ((g + qq - 6) & 7) + 1; pc = pn; }
            gemm_tile(P, (LAS unsigned char*)shm, pm, pc, call);
        }
    }
    if (threadIdx.x == 0) __hip_atomic_store(cnt, call, __ATOMIC_RELAXED, __HIP_MEMORY_SCOPE_AGENT);
}

void build_schedule(unsigned* sched) {
    unsigned short e[256]; int n = 0;
    auto W = [&](int c) { for (int j = 0; j < 16; ++j) e[n++] = (unsigned short)((j << 3) | c); };
    auto X = [&](int u0, int u1, int c) { for (int u = u0; u < u1; ++u) e[n++] = (unsigned short)(128 | (u << 3) | c); };
    int ca = 0, cb = 0;
    for (int c = 0; c < 8; ++c) { X(0, 2, c); W(c); if (c >= 3) X(2, 9, ca++); }
    while (ca < 8) { X(9, 16, cb++); X(2, 9, ca++); }
    while (cb < 8) X(9, 16, cb++);
    for (int i = 0; i < 128; ++i) sched[i] = e[2 * i] | ((unsigned)e[2 * i + 1] << 16);
}
}

extern "C" void kernel_launch(int rank, int nranks, ncclComm_t comm, hipStream_t stream,
                              void* const* d_in, const int* in_sizes, int n_in,
                              void* d_out, int out_size, const int* in_dtypes, int out_dtype,
                              void* const* d_out_all, void* const* d_ws_all, size_t ws_bytes) {
    if (nranks != NR || (unsigned)rank >= (unsigned)NR || n_in < 2 || ws_bytes < WS_NEED ||
        in_sizes[0] != GM * KS || in_sizes[1] != KS * GN || out_size != MS * GN) {
        fprintf(stderr, "kernel_launch: unexpected problem shape\n");
        return;
    }
    HIP_CHECK(hipFuncSetAttribute((const void*)fused_gemm_rs, hipFuncAttributeMaxDynamicSharedMemorySize, LDS_BYTES));
    Params P;
    P.x = (const char*)d_in[0]; P.w = (const char*)d_in[1];
    for (int p = 0; p < NR; ++p) { P.ws[p] = (char*)d_ws_all[p]; P.out[p] = (char*)d_out_all[p]; }
    P.rank = rank;
    build_schedule(P.sched);
    hipLaunchKernelGGL(fused_gemm_rs, dim3(GRID), dim3(512), LDS_BYTES, stream, P);
    HIP_CHECK(hipGetLastError());
}
```

```cpp
#include <hip/hip_runtime.h>
#include <rccl/rccl.h>
#include <cstdio>

#define HIP_CHECK(expr) do { hipError_t _e = (expr); if (_e != hipSuccess) { \
    fprintf(stderr, "HIP error %s at %s:%d\n", hipGetErrorString(_e), __FILE__, __LINE__); \
    return; } } while (0)

#define LAS __attribute__((address_space(3)))
typedef short s16x4 __attribute__((ext_vector_type(4)));
typedef short bf16x8 __attribute__((ext_vector_type(8)));
typedef float f32x4 __attribute__((ext_vector_type(4)));
typedef unsigned u32x4 __attribute__((ext_vector_type(4)));

namespace {
constexpr int NR = 8;
constexpr int GM = 4096, GK = 4096, GN = 32768;
constexpr int KS = GK / NR;
constexpr int NB = GN / NR;
constexpr int MS = GM / NR;
constexpr unsigned XP = KS * 2;
constexpr unsigned WP = GN * 2;
constexpr int HTB = 128 * 64 * 2;
constexpr int STAGE_BYTES = 8 * HTB;
constexpr int LDS_BYTES = STAGE_BYTES + 64;
constexpr int GRID = 256;
#define CPX 4
constexpr int NCOMM = 8 * CPX;
constexpr unsigned long long DEADLINE = 50000000ull;

constexpr size_t OFF_CNT = 0;
constexpr size_t OFF_TMO = 32768;
constexpr size_t OFF_FX = 65536;
constexpr size_t OFF_FW = OFF_FX + 8192;
constexpr size_t OFF_FO = OFF_FW + 8192;
constexpr size_t OFF_Q = OFF_FO + 1024;
constexpr size_t OFF_PD = OFF_Q + 2048;
constexpr size_t OFF_DBG = 131072;
constexpr size_t OFF_XS = 1u << 20;
constexpr size_t XS_PAR = (size_t)NR * GM * XP;
constexpr size_t OFF_WS = OFF_XS + 2 * XS_PAR;
constexpr size_t WS_PAR = (size_t)KS * WP;
constexpr size_t OFF_PS = OFF_WS + 2 * WS_PAR;
constexpr size_t PS_PAR = (size_t)256 * 131072;
constexpr size_t WS_NEED = OFF_PS + 2 * PS_PAR;
constexpr unsigned NPUSH = 512, NPULL = 256, QSTEP = NPUSH + NPULL + NCOMM;

struct Params {
    const char* x; const char* w;
    char* ws[NR]; char* out[NR];
    int rank;
    unsigned sched[128];
};

__device__ __forceinline__ int lds_byte(int r, int c) { const int st = (r >> 4) * 2 + (c >> 5), rr = r & 15, cc = c & 31, ob = rr * 64 + cc * 2; return st * 1024 + (ob ^ (((ob >> 9) & 1) << 5)); }
__device__ __forceinline__ void stage_rc(int b, int& R, int& C) { const int st = b / 1024, sb = b % 1024, swz = sb ^ (((sb >> 9) & 1) << 5); R = (st >> 1) * 16 + swz / 64; C = (st & 1) * 32 + (swz % 64) / 2; }
__device__ __forceinline__ __amdgpu_buffer_rsrc_t mk(const void* p, unsigned bytes) { return __builtin_amdgcn_make_buffer_rsrc((void*)p, 0, bytes, 0x00020000); }
__device__ __forceinline__ unsigned cvt_pk_bf16(float lo, float hi) { unsigned r; asm volatile("v_cvt_pk_bf16_f32 %0, %1, %2" : "=v"(r) : "v"(lo), "v"(hi)); return r; }
__device__ __forceinline__ float silu(float y) {
    const float c = __builtin_amdgcn_fmed3f(y, -60.0f, 60.0f);
    const float e = __builtin_amdgcn_exp2f(c * -1.44269504088896341f);
    return y * __builtin_amdgcn_rcpf(1.0f + e);
}
__device__ __forceinline__ unsigned ld_agent(const unsigned* p) { return __hip_atomic_load(p, __ATOMIC_RELAXED, __HIP_MEMORY_SCOPE_AGENT); }
__device__ __forceinline__ unsigned long long ld_sys64(const void* p) { return __hip_atomic_load((const unsigned long long*)p, __ATOMIC_RELAXED, __HIP_MEMORY_SCOPE_SYSTEM); }
__device__ __forceinline__ void st_sys(void* p, unsigned v) { __hip_atomic_store((unsigned*)p, v, __ATOMIC_RELAXED, __HIP_MEMORY_SCOPE_SYSTEM); }
__device__ __forceinline__ int panel_of(int u, int d) { return u < 2 ? 2 * d + u : (u < 9 ? 2 * ((d + u - 1) & 7) : 2 * ((d + u - 8) & 7) + 1); }

#define DBG_STAMP(k) do { } while (0)

#define GATE_EXTRA 0
#define PD 2
constexpr int SAUX = 17;

struct Task { __amdgpu_buffer_rsrc_t rs, rd; unsigned stride; unsigned voff; char* flag; };

__device__ __forceinline__ void comm_role(const Params& P, int cw, unsigned call) {
    const int lane = threadIdx.x & 63, hw = __builtin_amdgcn_readfirstlane(threadIdx.x >> 6);
    if (hw >= 7) return;
    const int wid = (hw + cw) % 7;
    const int g = P.rank;
    char* myws = P.ws[g];
    const int d = (g + wid + 1) & 7, q = 7 - wid;
    char* dws = P.ws[d];
    const __amdgpu_buffer_rsrc_t qctr = mk(myws + OFF_Q + (wid + 1) * 256, 4);
    const unsigned qbase = (call - 1u) * QSTEP;
    auto draw = [&]() { return (unsigned)__builtin_amdgcn_raw_ptr_buffer_atomic_add_i32(1, qctr, lane * 4, 0, 0); };
    unsigned tk = draw();
    unsigned s = __builtin_amdgcn_readfirstlane(tk) - qbase;
    if (s < NPUSH) {
        char* dxs = dws + OFF_XS + (call & 1) * XS_PAR + (size_t)q * GM * XP;
        char* dwsb = dws + OFF_WS + (call & 1) * WS_PAR + (size_t)q * NB * 2;
        const unsigned vx = (lane >> 3) * XP + (lane & 7) * 16, vw = (lane >> 4) * WP + (lane & 15) * 16;
        auto desc = [&](unsigned s, Task& t) {
            if (s < NPUSH) {
                const unsigned e = (P.sched[s >> 2] >> (((s >> 1) & 1) * 16)) & 0xffffu;
                const int c = e & 7, idx = (e >> 3) & 15, h = s & 1;
                if (e & 128) {
                    const int pn = panel_of(idx, d);
                    const size_t o = (size_t)(pn * 256 + h * 128) * XP + c * 128;
                    t.rs = mk(P.x + o, 127 * XP + 128); t.rd = mk(dxs + o, 127 * XP + 128); t.stride = 8 * XP; t.voff = vx;
                    t.flag = dws + OFF_FX + ((pn * 64 + c * 8 + q) * 2 + h) * 4;
                } else {
                    const size_t o = (size_t)c * 64 * WP + idx * 512 + h * 256;
                    t.rs = mk(P.w + o + (size_t)d * NB * 2, 63 * WP + 256); t.rd = mk(dwsb + o, 63 * WP + 256); t.stride = 4 * WP; t.voff = vw;
                    t.flag = dws + OFF_FW + ((idx * 64 + c * 8 + q) * 2 + h) * 4;
                }
            } else { t.rs = mk(P.x, 0); t.rd = mk(myws, 0); t.stride = 0; t.voff = 0; t.flag = nullptr; }
        };
        Task cur, nxt; char* pflag = nullptr;
        desc(s, cur);
        u32x4 a0, a1, b0, b1;
        a0 = __builtin_amdgcn_raw_buffer_load_b128(cur.rs, cur.voff, 0, 0);
        a1 = __builtin_amdgcn_raw_buffer_load_b128(cur.rs, cur.voff, cur.stride, 0);
        {
            const u32x4 z = {0u, 0u, 0u, 0u}; const __amdgpu_buffer_rsrc_t rn = mk(myws, 0);
            __builtin_amdgcn_sched_barrier(0);
            __builtin_amdgcn_raw_buffer_store_b128(z, rn, cur.voff, 0, SAUX); __builtin_amdgcn_raw_buffer_store_b128(z, rn, cur.voff, 16, SAUX);
            __builtin_amdgcn_sched_barrier(0);
        }
        while (s < NPUSH) {
            tk = draw();
#pragma unroll
            for (int it = 0; it < 8; it += 2) {
                b0 = __builtin_amdgcn_raw_buffer_load_b128(cur.rs, cur.voff, (2 * it + 2) * cur.stride, 0);
                b1 = __builtin_amdgcn_raw_buffer_load_b128(cur.rs, cur.voff, (2 * it + 3) * cur.stride, 0);
                __builtin_amdgcn_sched_barrier(0);
                __builtin_amdgcn_raw_buffer_store_b128(a0, cur.rd, cur.voff, (2 * it) * cur.stride, SAUX);
                __builtin_amdgcn_raw_buffer_store_b128(a1, cur.rd, cur.voff, (2 * it + 1) * cur.stride, SAUX);
                __builtin_amdgcn_sched_barrier(0);
                if (it + 2 < 8) {
                    a0 = __builtin_amdgcn_raw_buffer_load_b128(cur.rs, cur.voff, (2 * it + 4) * cur.stride, 0);
                    a1 = __builtin_amdgcn_raw_buffer_load_b128(cur.rs, cur.voff, (2 * it + 5) * cur.stride, 0);
                } else {
                    s = __builtin_amdgcn_readfirstlane(tk) - qbase;
                    desc(s, nxt);
                    a0 = __builtin_amdgcn_raw_buffer_load_b128(nxt.rs, nxt.voff, 0, 0);
                    a1 = __builtin_amdgcn_raw_buffer_load_b128(nxt.rs, nxt.voff, nxt.stride, 0);
                }
                __builtin_amdgcn_sched_barrier(0);
                __builtin_amdgcn_raw_buffer_store_b128(b0, cur.rd, cur.voff, (2 * it + 2) * cur.stride, SAUX);
                __builtin_amdgcn_raw_buffer_store_b128(b1, cur.rd, cur.voff, (2 * it + 3) * cur.stride, SAUX);
                __builtin_amdgcn_sched_barrier(0);
                if (it == 0 && pflag) {
                    asm volatile("s_waitcnt vmcnt(8)" ::: "memory");
                    if (lane == 0) st_sys(pflag, call);
                    asm volatile("" ::: "memory");
                }
            }
            pflag = cur.flag; cur = nxt;
        }
        asm volatile("s_waitcnt vmcnt(0)" ::: "memory");
        if (lane == 0) st_sys(pflag, call);
    }
    if (lane == 0) __hip_atomic_fetch_add((unsigned*)(dws + OFF_PD + q * 128), 1u, __ATOMIC_RELAXED, __HIP_MEMORY_SCOPE_SYSTEM);
    DBG_STAMP(1);
    {
        const unsigned k = s - NPUSH, want = (call - 1u) * NCOMM + (k + GATE_EXTRA < NCOMM ? k + GATE_EXTRA : NCOMM - 1) + 1u;
        const unsigned* pd = (const unsigned*)(myws + OFF_PD + (wid + 1) * 128);
        unsigned* tmo = (unsigned*)(myws + OFF_TMO);
        const unsigned long long t0 = __builtin_amdgcn_s_memrealtime();
        while (__builtin_amdgcn_readfirstlane(__hip_atomic_load(pd, __ATOMIC_RELAXED, __HIP_MEMORY_SCOPE_SYSTEM)) < want) {
            if (__builtin_amdgcn_readfirstlane(ld_agent(tmo)) == call || __builtin_amdgcn_s_memrealtime() - t0 > DEADLINE) {
                if (lane == 0) __hip_atomic_store(tmo, call, __ATOMIC_RELAXED, __HIP_MEMORY_SCOPE_AGENT);
                break; }
            __builtin_amdgcn_s_sleep(4);
        }
    }
    DBG_STAMP(3);
    {
        const char* dps = dws + OFF_PS + (call & 1) * PS_PAR + (size_t)(2 * g) * 16 * 131072;
        char* ob = P.out[g] + (size_t)d * NB * 2;
        const unsigned* fo = (const unsigned*)(myws + OFF_FO) + (wid + 1) * 32 + (lane & 31);
        unsigned* tmo = (unsigned*)(myws + OFF_TMO);
        const unsigned vl = lane * 16, vs = (lane & 15) * WP + (lane >> 4) * 16;
        unsigned rdy = 0; bool bad = false, pre = false;
        u32x4 v[PD];
        __amdgpu_buffer_rsrc_t rs = mk(myws, 0);
        while (s < NPUSH + NPULL) {
            const unsigned u = s - NPUSH, tile = u >> 3, part = u & 7;
            if (!pre) {
                const unsigned long long t0 = __builtin_amdgcn_s_memrealtime();
                for (;;) {
                    const unsigned f = __hip_atomic_load(fo, __ATOMIC_RELAXED, __HIP_MEMORY_SCOPE_SYSTEM);
                    rdy = (unsigned)__ballot(f >= call);
                    if ((rdy >> tile) & 1u) break;
                    if (__builtin_amdgcn_readfirstlane(ld_agent(tmo)) == call || __builtin_amdgcn_s_memrealtime() - t0 > DEADLINE) {
                        if (lane == 0) __hip_atomic_store(tmo, call, __ATOMIC_RELAXED, __HIP_MEMORY_SCOPE_AGENT);
                        rdy = ~0u; bad = true; break; }
                    __builtin_amdgcn_s_sleep(4);
                }
                __builtin_amdgcn_fence(__ATOMIC_ACQUIRE, "");
                rs = mk(dps + (size_t)u * 16384, bad ? 0 : 16384);
#pragma unroll
                for (int k = 0; k < PD; ++k) v[k] = __builtin_amdgcn_raw_buffer_load_b128(rs, vl, k * 1024, SAUX);
            }
            tk = draw();
            const __amdgpu_buffer_rsrc_t rd = mk(ob + (size_t)((tile >> 4) * 256 + part * 32) * WP + (tile & 15) * 512, 31 * WP + 512);
#pragma unroll
            for (int k = 0; k < 16; ++k) {
                if (k + PD == 16) {
                    s = __builtin_amdgcn_readfirstlane(tk) - qbase;
                    const unsigned un = s - NPUSH;
                    pre = un < NPULL && ((rdy >> ((un >> 3) & 31)) & 1u);
                    rs = mk(dps + (size_t)(un & 255) * 16384, pre && !bad ? 16384 : 0);
                }
                __builtin_amdgcn_sched_barrier(0);
                u32x4 o = v[k % PD];
                if (bad) o = (u32x4){0x7fc07fc0u, 0x7fc07fc0u, 0x7fc07fc0u, 0x7fc07fc0u};
                __builtin_amdgcn_raw_buffer_store_b128(o, rd, vs + (k >> 3) * 16 * WP + (k & 7) * 64, 0, 0);
                __builtin_amdgcn_sched_barrier(0);
                v[k % PD] = __builtin_amdgcn_raw_buffer_load_b128(rs, vl, ((k + PD) & 15) * 1024, SAUX);
            }
        }
    }
    DBG_STAMP(2);
}

__device__ __forceinline__ void gemm_tile(const Params& P, LAS unsigned char* lds, int pm, int pn, unsigned call) {
    const int tid = threadIdx.x, wid = __builtin_amdgcn_readfirstlane(tid >> 6), lane = tid & 63, wr = wid >> 2, wc = wid & 3, fr = lane & 15, fq = lane >> 4;
    const int g = P.rank;
    char* myws = P.ws[g];
    unsigned voffA[2], voffB[2];
#pragma unroll
    for (int i = 0; i < 2; ++i) {
        int R, C; stage_rc(tid * 16 + i * 8192, R, C); voffA[i] = (unsigned)R * XP + (unsigned)C * 2u;
        const int row = lane >> 3, ch = (lane & 7) ^ (((row >> 1) & 1) << 2);
        voffB[i] = (unsigned)(((wid >> 1) + 4 * i) * 8 + row) * WP + (unsigned)(wid & 1) * 128u + (unsigned)ch * 16u;
    }
    const unsigned ldsw = (unsigned)wid * 1024u;
    const int aoff = lds_byte(wr * 64 + fr, fq * 8);
    const int boff = fq * 2048 + (wc >> 1) * 1024 + (fr >> 2) * 128 + ((4 * (wc & 1) + (fr & 3)) ^ (((fr >> 3) & 1) << 2)) * 16;
    constexpr size_t hA = (size_t)128 * XP, hB = 256;
    const char* xl = P.x + (size_t)pm * 256 * XP;
    const char* wl = P.w + (size_t)g * NB * 2 + pn * 512;
    const char* xs = myws + OFF_XS + (call & 1) * XS_PAR + (size_t)pm * 256 * XP;
    const char* wsb = myws + OFF_WS + (call & 1) * WS_PAR + pn * 512;
    auto ktile = [&](int t, const char*& a, const char*& b, int& bit) {
        if (t < 8) { a = xl + t * 128; b = wl + (size_t)t * 64 * WP; bit = 0; }
        else { const unsigned u = t - 8, c = u / 7u, q = u - c * 7u + 1u; a = xs + (size_t)q * GM * XP + c * 128; b = wsb + (size_t)c * 64 * WP + (size_t)q * NB * 2; bit = c * 8 + q; }
    };
#define SA(b, h) (((b) * 2 + (h)) * HTB)
#define SB(b, h) ((4 + (b) * 2 + (h)) * HTB)
#define STAGE(bufoff, gbase, voff) do { _Pragma("unroll") for (int _i = 0; _i < 2; ++_i) \
        __builtin_amdgcn_global_load_lds((const unsigned*)((const char*)(gbase) + (voff)[_i]), (LAS unsigned*)(lds + (bufoff) + ldsw + _i * 8192), 16, 0, 0); } while (0)
#define LDA(dst, b, h) do { _Pragma("unroll") for (int m = 0; m < 4; ++m) _Pragma("unroll") for (int k = 0; k < 2; ++k) dst[m][k] = *(const LAS bf16x8*)(lds + SA(b, h) + aoff + m * 2048 + k * 1024); } while (0)
#define TRD(dst, off) asm volatile("ds_read_b64_tr_b16 %0, %1 offset:%2" : "=v"(dst) : "v"(baddr), "i"(off) : "memory")
#define LDB(dst, b, h) do { _Pragma("unroll") for (int n = 0; n < 2; ++n) _Pragma("unroll") for (int k = 0; k < 2; ++k) { \
        TRD(dst[n][k][0], ((b) * 2 + (h)) * HTB + k * 8192 + n * 8); TRD(dst[n][k][1], ((b) * 2 + (h)) * HTB + k * 8192 + 512 + n * 8); } } while (0)
#define WAIT_B(Bt) asm volatile("s_waitcnt lgkmcnt(0)" : "+v"(Bt[0][0][0]), "+v"(Bt[0][0][1]), "+v"(Bt[0][1][0]), "+v"(Bt[0][1][1]), "+v"(Bt[1][0][0]), "+v"(Bt[1][0][1]), "+v"(Bt[1][1][0]), "+v"(Bt[1][1][1]) :: "memory")
#define BF(Bt, n, k) __builtin_shufflevector(Bt[n][k][0], Bt[n][k][1], 0, 1, 2, 3, 4, 5, 6, 7)
#define MMA(ai, bj, At, Bt) do { __builtin_amdgcn_s_setprio(1); _Pragma("unroll") for (int m = 0; m < 4; ++m) _Pragma("unroll") for (int n = 0; n < 2; ++n) _Pragma("unroll") for (int k = 0; k < 2; ++k) \
        acc[ai][bj][m][n] = __builtin_amdgcn_mfma_f32_16x16x32_bf16(BF(Bt, n, k), At[m][k], acc[ai][bj][m][n], 0, 0, 0); __builtin_amdgcn_s_setprio(0); } while (0)
#define WAIT_V(n) asm volatile("s_waitcnt vmcnt(" #n ")" ::: "memory")
#define WAIT_L(n) asm volatile("s_waitcnt lgkmcnt(" #n ")" ::: "memory")
#define BAR __builtin_amdgcn_s_barrier()
#define SCHED __builtin_amdgcn_sched_barrier(0)
    f32x4 acc[2][2][4][2];
#pragma unroll
    for (int a = 0; a < 2; ++a)
#pragma unroll
        for (int b = 0; b < 2; ++b)
#pragma unroll
            for (int m = 0; m < 4; ++m)
#pragma unroll
                for (int n = 0; n < 2; ++n) acc[a][b][m][n] = (f32x4){0.f, 0.f, 0.f, 0.f};
    bf16x8 At[4][2]; s16x4 B0[2][2][2], B1[2][2][2];
    const unsigned baddr = (unsigned)(size_t)lds + 4 * HTB + boff;
    unsigned long long rdy = 1ull;
    LAS unsigned long long* rdy_lds = (LAS unsigned long long*)(lds + STAGE_BYTES);
    STAGE(SB(0, 0), wl, voffB); STAGE(SA(0, 0), xl, voffA); STAGE(SB(0, 1), wl + hB, voffB); STAGE(SA(0, 1), xl + hA, voffA);
    if (wr == 1) BAR;
    WAIT_V(4); BAR;
    STAGE(SB(1, 0), wl + (size_t)64 * WP, voffB); STAGE(SA(1, 0), xl + 128, voffA); STAGE(SB(1, 1), wl + (size_t)64 * WP + hB, voffB);
    WAIT_V(6); BAR;
    const char* a1 = xl + 128;
    for (int t = 0; t < 64; t += 2) {
        const char *a2, *b2, *a3, *b3; int bit2, bit3;
        ktile((t + 2) & 63, a2, b2, bit2); ktile((t + 3) & 63, a3, b3, bit3);
        const unsigned long long need = (1ull << bit2) | (1ull << bit3);
        if ((rdy & need) != need) {
            if (wid == 0) {
                const char* fx = myws + OFF_FX + (pm * 64 + lane) * 8; const char* fw = myws + OFF_FW + (pn * 64 + lane) * 8;
                unsigned* tmo = (unsigned*)(myws + OFF_TMO);
                const unsigned long long t0 = __builtin_amdgcn_s_memrealtime();
                unsigned long long mask;
                for (;;) {
                    const unsigned long long vx = ld_sys64(fx), vw = ld_sys64(fw);
                    const bool ok = (lane & 7) == 0 || ((unsigned)vx >= call && (unsigned)(vx >> 32) >= call && (unsigned)vw >= call && (unsigned)(vw >> 32) >= call);
                    mask = __ballot(ok);
                    if ((mask & need) == need) break;
                    if (__builtin_amdgcn_readfirstlane(ld_agent(tmo)) == call || __builtin_amdgcn_s_memrealtime() - t0 > DEADLINE) {
                        if (lane == 0) __hip_atomic_store(tmo, call, __ATOMIC_RELAXED, __HIP_MEMORY_SCOPE_AGENT);
                        mask = ~0ull; break; }
                    __builtin_amdgcn_s_sleep(8);
                }
                __builtin_amdgcn_fence(__ATOMIC_ACQUIRE, "");
                if (lane == 0) *rdy_lds = mask;
                asm volatile("s_waitcnt vmcnt(0) lgkmcnt(0)" ::: "memory");
            }
            asm volatile("" ::: "memory"); BAR; asm volatile("" ::: "memory");
            const unsigned long long mv = *rdy_lds;
            rdy = ((unsigned long long)(unsigned)__builtin_amdgcn_readfirstlane((unsigned)(mv >> 32)) << 32) | (unsigned)__builtin_amdgcn_readfirstlane((unsigned)mv);
        }
        LDB(B0, 0, 0); SCHED; LDA(At, 0, 0); STAGE(SA(1, 1), a1 + hA, voffA);
        WAIT_L(8); BAR; WAIT_B(B0); MMA(0, 0, At, B0); BAR; SCHED;
        LDB(B1, 0, 1); STAGE(SB(0, 0), b2, voffB);
        BAR; WAIT_B(B1); MMA(0, 1, At, B1); BAR;
        LDA(At, 0, 1); STAGE(SA(0, 0), a2, voffA);
        BAR; WAIT_L(0); MMA(1, 0, At, B0); BAR; SCHED;
        STAGE(SB(0, 1), b2 + hB, voffB);
        WAIT_V(6); BAR; MMA(1, 1, At, B1); BAR;
        LDB(B0, 1, 0); SCHED; LDA(At, 1, 0); STAGE(SA(0, 1), a2 + hA, voffA);
        WAIT_L(8); BAR; WAIT_B(B0); MMA(0, 0, At, B0); BAR; SCHED;
        LDB(B1, 1, 1); STAGE(SB(1, 0), b3, voffB);
        BAR; WAIT_B(B1); MMA(0, 1, At, B1); BAR;
        LDA(At, 1, 1); STAGE(SA(1, 0), a3, voffA);
        BAR; WAIT_L(0); MMA(1, 0, At, B0); BAR; SCHED;
        STAGE(SB(1, 1), b3 + hB, voffB);
        WAIT_V(6); BAR; MMA(1, 1, At, B1); BAR;
        a1 = a3;
    }
    if (wr == 0) BAR;
    DBG_STAMP(2);
    const int dest = pm >> 1;
    const bool bad = __builtin_amdgcn_readfirstlane(ld_agent((const unsigned*)(myws + OFF_TMO))) == call;
    {
        int tz = threadIdx.x; asm volatile("" : "+v"(tz));
        const int l2 = tz & 63, fr2 = l2 & 15, fq2 = l2 >> 4;
        const bool mine = dest == g;
        const __amdgpu_buffer_rsrc_t ro = mine ? mk(P.out[g] + (size_t)(pm & 1) * 256 * WP + (size_t)g * NB * 2 + pn * 512, 255 * WP + 512)
                                               : mk(myws + OFF_PS + (call & 1) * PS_PAR + (size_t)(pm * 16 + pn) * 131072, 131072);
        const unsigned vo = mine ? (unsigned)(wr * 64 + fr2) * WP + (unsigned)(wc * 32 + 8 * fq2) * 2u : (unsigned)(wr * 4 * 8192 + wc * 1024 + l2 * 16);
        const unsigned sg = mine ? 16 * WP : 8192u, sb = mine ? 256u : 4096u;
#pragma unroll
        for (int ai = 0; ai < 2; ++ai)
#pragma unroll
            for (int m = 0; m < 4; ++m)
#pragma unroll
                for (int bj = 0; bj < 2; ++bj) {
                    const f32x4 v0 = acc[ai][bj][m][0], v1 = acc[ai][bj][m][1];
                    u32x4 o; o.x = cvt_pk_bf16(silu(v0[0]), silu(v0[1])); o.y = cvt_pk_bf16(silu(v0[2]), silu(v0[3]));
                    o.z = cvt_pk_bf16(silu(v1[0]), silu(v1[1])); o.w = cvt_pk_bf16(silu(v1[2]), silu(v1[3]));
                    if (bad) o = (u32x4){0x7fc07fc0u, 0x7fc07fc0u, 0x7fc07fc0u, 0x7fc07fc0u};
                    __builtin_amdgcn_raw_buffer_store_b128(o, ro, vo + (unsigned)(ai * 8 + m) * sg + bj * sb, 0, SAUX);
                }
    }
    WAIT_V(0);
    BAR;
    if (tid == 0 && dest != g) st_sys(P.ws[dest] + OFF_FO + ((((g - dest) & 7) * 32) + (pm & 1) * 16 + pn) * 4, call);
    DBG_STAMP(3);
#undef SA
#undef SB
#undef STAGE
#undef LDA
#undef LDB
#undef TRD
#undef WAIT_B
#undef BF
#undef MMA
}

__global__ __launch_bounds__(512, 2) void fused_gemm_rs(const Params P) {
    extern __shared__ __attribute__((aligned(16))) unsigned char shm[];
    const int b = blockIdx.x, xcd = b & 7, slot = b >> 3;
    unsigned* cnt = (unsigned*)(P.ws[P.rank] + OFF_CNT + (size_t)b * 128);
    const unsigned call = __builtin_amdgcn_readfirstlane(ld_agent(cnt)) + 1u;
    DBG_STAMP(0);
    if (slot < CPX) {
        comm_role(P, b, call);
        __syncthreads();
    } else {
        const int g = P.rank, s = slot - 4;
        int pm, pc, nl = -1;
        pm = 2 * ((g + 1 + (s % 7)) & 7) + (s >= 14); pc = xcd + 8 * ((s / 7) & 1);
        if (s < 0) nl = 24 + xcd;
        else if (s < CPX) nl = s * 8 + xcd;
        for (int u = (s < 0 ? 1 : 0); u < (nl >= 0 ? 2 : 1); ++u) {
            if (u == 1) { pm = 2 * g + (nl >> 4); pc = nl & 15; }
            gemm_tile(P, (LAS unsigned char*)shm, pm, pc, call);
        }
    }
    if (threadIdx.x == 0) __hip_atomic_store(cnt, call, __ATOMIC_RELAXED, __HIP_MEMORY_SCOPE_AGENT);
}

void build_schedule(unsigned* sched) {
    unsigned short e[256]; int n = 0;
    auto W = [&](int j0, int j1, int c) { for (int j = j0; j < j1; ++j) e[n++] = (unsigned short)((j << 3) | c); };
    auto X = [&](int u0, int u1, int c) { for (int u = u0; u < u1; ++u) e[n++] = (unsigned short)(128 | (u << 3) | c); };
    for (int c = 0; c < 8; ++c) { X(2, 9, c); W(0, 8, c); if (c & 1) W(8, 16, c >> 1); }
    for (int c = 0; c < 4; ++c) { W(8, 16, 4 + c); X(9, 16, c); X(0, 2, c); }
    for (int c = 4; c < 8; ++c) { X(9, 16, c); X(0, 2, c); }
    for (int i = 0; i < 128; ++i) sched[i] = e[2 * i] | ((unsigned)e[2 * i + 1] << 16);
}
}

extern "C" void kernel_launch(int rank, int nranks, ncclComm_t comm, hipStream_t stream,
                              void* const* d_in, const int* in_sizes, int n_in,
                              void* d_out, int out_size, const int* in_dtypes, int out_dtype,
                              void* const* d_out_all, void* const* d_ws_all, size_t ws_bytes) {
    if (nranks != NR || (unsigned)rank >= (unsigned)NR || n_in < 2 || ws_bytes < WS_NEED ||
        in_sizes[0] != GM * KS || in_sizes[1] != KS * GN || out_size != MS * GN) {
        fprintf(stderr, "kernel_launch: unexpected problem shape\n");
        return;
    }
    HIP_CHECK(hipFuncSetAttribute((const void*)fused_gemm_rs, hipFuncAttributeMaxDynamicSharedMemorySize, LDS_BYTES));
    Params P;
    P.x = (const char*)d_in[0]; P.w = (const char*)d_in[1];
    for (int p = 0; p < NR; ++p) { P.ws[p] = (char*)d_ws_all[p]; P.out[p] = (char*)d_out_all[p]; }
    P.rank = rank;
    build_schedule(P.sched);
    hipLaunchKernelGGL(fused_gemm_rs, dim3(GRID), dim3(512), LDS_BYTES, stream, P);
    HIP_CHECK(hipGetLastError());
}
```

```cpp
#include <hip/hip_runtime.h>
#include <rccl/rccl.h>
#include <cstdio>

#define HIP_CHECK(expr) do { hipError_t _e = (expr); if (_e != hipSuccess) { \
    fprintf(stderr, "HIP error %s at %s:%d\n", hipGetErrorString(_e), __FILE__, __LINE__); \
    return; } } while (0)

#define LAS __attribute__((address_space(3)))
typedef short s16x4 __attribute__((ext_vector_type(4)));
typedef short bf16x8 __attribute__((ext_vector_type(8)));
typedef float f32x4 __attribute__((ext_vector_type(4)));
typedef float f32x2 __attribute__((ext_vector_type(2)));
typedef __bf16 bf16x2 __attribute__((ext_vector_type(2)));
typedef unsigned u32x4 __attribute__((ext_vector_type(4)));

namespace {
constexpr int NR = 8;
constexpr int GM = 4096, GK = 4096, GN = 32768;
constexpr int KS = GK / NR;
constexpr int NB = GN / NR;
constexpr int MS = GM / NR;
constexpr unsigned XP = KS * 2;
constexpr unsigned WP = GN * 2;
constexpr int HTB = 128 * 64 * 2;
constexpr int STAGE_BYTES = 8 * HTB;
constexpr int LDS_BYTES = STAGE_BYTES + 64;
constexpr int GRID = 256;
constexpr int CPX = 4;
constexpr int NCOMM = 8 * CPX;
#define J1 1
#define J2 0
constexpr int NPW0 = 24;
constexpr int NPW = NPW0 + 32 * J1 + 56 * J2;
constexpr unsigned long long DEADLINE = 50000000ull;

constexpr size_t OFF_CNT = 0;
constexpr size_t OFF_TMO = 32768;
constexpr size_t OFF_FX = 65536;
constexpr size_t OFF_FW = OFF_FX + 8192;
constexpr size_t OFF_FO = OFF_FW + 8192;
constexpr size_t OFF_Q = OFF_FO + 2048;
constexpr size_t OFF_PD = OFF_Q + 4096;
constexpr size_t OFF_DBG = 131072;
constexpr size_t OFF_XS = 1u << 20;
constexpr size_t XS_PAR = (size_t)NR * GM * XP;
constexpr size_t OFF_WS = OFF_XS + 2 * XS_PAR;
constexpr size_t WS_PAR = (size_t)KS * WP;
constexpr size_t OFF_PS = OFF_WS + 2 * WS_PAR;
constexpr size_t PS_PAR = (size_t)256 * 131072;
constexpr size_t OFF_DS = OFF_PS + 2 * PS_PAR;
constexpr size_t WS_NEED = OFF_DS + (size_t)256 * 262144;
constexpr unsigned NPUSH = 480;
constexpr unsigned PUSH_STEP = NPUSH;
constexpr unsigned NPULL = 1024, PULL_STEP = NPULL + 2 * NPW;

struct Params {
    const char* x; const char* w;
    char* ws[NR]; char* out;
    int rank;
    unsigned sched[60];
};

__device__ __forceinline__ int lds_byte(int r, int c) { const int st = (r >> 4) * 2 + (c >> 5), rr = r & 15, cc = c & 31, ob = rr * 64 + cc * 2; return st * 1024 + (ob ^ (((ob >> 9) & 1) << 5)); }
__device__ __forceinline__ void stage_rc(int b, int& R, int& C) { const int st = b / 1024, sb = b % 1024, swz = sb ^ (((sb >> 9) & 1) << 5); R = (st >> 1) * 16 + swz / 64; C = (st & 1) * 32 + (swz % 64) / 2; }
__device__ __forceinline__ __amdgpu_buffer_rsrc_t mk(const void* p, unsigned bytes) { return __builtin_amdgcn_make_buffer_rsrc((void*)p, 0, bytes, 0x00020000); }
__device__ __forceinline__ unsigned pk_bf16(float lo, float hi) { const bf16x2 r = __builtin_convertvector((f32x2){lo, hi}, bf16x2); return __builtin_bit_cast(unsigned, r); }
__device__ __forceinline__ float silu(float y) {
    const float c = __builtin_amdgcn_fmed3f(y, -60.0f, 60.0f);
    const float e = __builtin_amdgcn_exp2f(c * -1.44269504088896341f);
    return y * __builtin_amdgcn_rcpf(1.0f + e);
}
__device__ __forceinline__ unsigned ld_agent(const unsigned* p) { return __hip_atomic_load(p, __ATOMIC_RELAXED, __HIP_MEMORY_SCOPE_AGENT); }
__device__ __forceinline__ unsigned ld_sys(const void* p) { return __hip_atomic_load((const unsigned*)p, __ATOMIC_RELAXED, __HIP_MEMORY_SCOPE_SYSTEM); }
__device__ __forceinline__ unsigned long long ld_sys64(const void* p) { return __hip_atomic_load((const unsigned long long*)p, __ATOMIC_RELAXED, __HIP_MEMORY_SCOPE_SYSTEM); }
__device__ __forceinline__ void st_sys(void* p, unsigned v) { __hip_atomic_store((unsigned*)p, v, __ATOMIC_RELAXED, __HIP_MEMORY_SCOPE_SYSTEM); }
__device__ __forceinline__ bool give_up(unsigned* tmo, unsigned call, unsigned long long t0) {
    if (__builtin_amdgcn_readfirstlane(ld_agent(tmo)) != call && __builtin_amdgcn_s_memrealtime() - t0 <= DEADLINE) return false;
    if ((threadIdx.x & 63) == 0) __hip_atomic_store(tmo, call, __ATOMIC_RELAXED, __HIP_MEMORY_SCOPE_AGENT);
    return true;
}
__device__ __forceinline__ int panel_of(int u, int d, int q) {
    if (u < 2) return 2 * d + u;
    const int hi = u >= 8, j = u - (hi ? 7 : 1);
    return 2 * ((d + j + (j >= q)) & 7) + hi;
}

#define DBG_STAMP(k) do { } while (0)

#define GATE0 24
#define GATE1 28
#define RET_IT 4
#define DP 1
#define DD 3
#define OAUX 17
constexpr int SAUX = 17;
constexpr unsigned QNAN2 = 0x7fc07fc0u;

struct Task { __amdgpu_buffer_rsrc_t rs; const char* dst; unsigned bytes, stride, voff, flag; };

__device__ __forceinline__ void push_role(const Params& P, int cw, unsigned cntv) {
    const int lane = threadIdx.x & 63, hw = __builtin_amdgcn_readfirstlane(threadIdx.x >> 6);
    if (hw >= 7) return;
    const int wid = (hw + cw) % 7;
    const int g = P.rank;
    char* myws = P.ws[g];
    const int d = (g + wid + 1) & 7, q = 7 - wid;
    char* dws = P.ws[d];
    const __amdgpu_buffer_rsrc_t qctr = mk(myws + OFF_Q + (wid + 1) * 256, 4);
    const unsigned vx = (lane >> 3) * XP + (lane & 7) * 16, vw = (lane >> 4) * WP + (lane & 15) * 16;
    auto desc = [&](unsigned s, Task& t) {
        if (s < NPUSH) {
            const unsigned e = (P.sched[s >> 3] >> (((s >> 1) & 3) * 8)) & 0xffu;
            const int c = e & 7, idx = (e >> 3) & 15, h = s & 1;
            if (e & 128) {
                const int pn = panel_of(idx, d, q);
                const size_t o = (size_t)(pn * 256 + h * 128) * XP + c * 128;
                t.bytes = 127 * XP + 128; t.rs = mk(P.x + o, t.bytes); t.dst = dws + OFF_XS + (size_t)q * GM * XP + o; t.stride = 8 * XP; t.voff = vx;
                t.flag = OFF_FX + ((pn * 64 + c * 8 + q) * 2 + h) * 4;
            } else {
                const size_t o = (size_t)c * 64 * WP + idx * 512 + h * 256;
                t.bytes = 63 * WP + 256; t.rs = mk(P.w + o + (size_t)d * NB * 2, t.bytes); t.dst = dws + OFF_WS + (size_t)q * NB * 2 + o; t.stride = 4 * WP; t.voff = vw;
                t.flag = OFF_FW + ((idx * 64 + c * 8 + q) * 2 + h) * 4;
            }
        } else { t.bytes = 0; t.rs = mk(P.x, 0); t.dst = myws; t.stride = 0; t.voff = 0; t.flag = 0; }
    };
    static_assert(XS_PAR == WS_PAR, "one parity offset");
    Task cur, nxt; unsigned pflag = 0, s = cw;
    desc(s, cur);
    u32x4 a0, a1, b0, b1;
    a0 = __builtin_amdgcn_raw_buffer_load_b128(cur.rs, cur.voff, 0, 0);
    a1 = __builtin_amdgcn_raw_buffer_load_b128(cur.rs, cur.voff, cur.stride, 0);
    {
        const u32x4 z = {0u, 0u, 0u, 0u}; const __amdgpu_buffer_rsrc_t rn = mk(myws, 0);
        __builtin_amdgcn_sched_barrier(0);
        __builtin_amdgcn_raw_buffer_store_b128(z, rn, cur.voff, 0, SAUX); __builtin_amdgcn_raw_buffer_store_b128(z, rn, cur.voff, 16, SAUX);
        __builtin_amdgcn_sched_barrier(0);
    }
    const unsigned call = __builtin_amdgcn_readfirstlane(cntv) + 1u;
    const unsigned qbase = (call - 1u) * PUSH_STEP - NCOMM;
    const size_t par = (call & 1) * XS_PAR;
    while (s < NPUSH) {
        const unsigned tk = (unsigned)__builtin_amdgcn_raw_ptr_buffer_atomic_add_i32(1, qctr, lane * 4, 0, 0);
        const __amdgpu_buffer_rsrc_t rd = mk(cur.dst + par, cur.bytes);
#pragma unroll
        for (int it = 0; it < 8; it += 2) {
            b0 = __builtin_amdgcn_raw_buffer_load_b128(cur.rs, cur.voff, (2 * it + 2) * cur.stride, 0);
            b1 = __builtin_amdgcn_raw_buffer_load_b128(cur.rs, cur.voff, (2 * it + 3) * cur.stride, 0);
            __builtin_amdgcn_sched_barrier(0);
            __builtin_amdgcn_raw_buffer_store_b128(a0, rd, cur.voff, (2 * it) * cur.stride, SAUX);
            __builtin_amdgcn_raw_buffer_store_b128(a1, rd, cur.voff, (2 * it + 1) * cur.stride, SAUX);
            __builtin_amdgcn_sched_barrier(0);
            asm volatile("s_nop 1");
            __builtin_amdgcn_sched_barrier(0);
            if (it == RET_IT) {
                s = __builtin_amdgcn_readfirstlane(tk) - qbase;
                if (s >= NPUSH && lane == 0) __hip_atomic_fetch_add((unsigned*)(dws + OFF_PD + q * 128), 1u, __ATOMIC_RELAXED, __HIP_MEMORY_SCOPE_SYSTEM);
            }
            if (it + 2 < 8) {
                a0 = __builtin_amdgcn_raw_buffer_load_b128(cur.rs, cur.voff, (2 * it + 4) * cur.stride, 0);
                a1 = __builtin_amdgcn_raw_buffer_load_b128(cur.rs, cur.voff, (2 * it + 5) * cur.stride, 0);
            } else {
                desc(s, nxt);
                a0 = __builtin_amdgcn_raw_buffer_load_b128(nxt.rs, nxt.voff, 0, 0);
                a1 = __builtin_amdgcn_raw_buffer_load_b128(nxt.rs, nxt.voff, nxt.stride, 0);
            }
            __builtin_amdgcn_sched_barrier(0);
            __builtin_amdgcn_raw_buffer_store_b128(b0, rd, cur.voff, (2 * it + 2) * cur.stride, SAUX);
            __builtin_amdgcn_raw_buffer_store_b128(b1, rd, cur.voff, (2 * it + 3) * cur.stride, SAUX);
            __builtin_amdgcn_sched_barrier(0);
            asm volatile("s_nop 1");
            __builtin_amdgcn_sched_barrier(0);
            if (it == 0 && pflag) {
                asm volatile("s_waitcnt vmcnt(8)" ::: "memory");
                if (lane == 0) st_sys(dws + pflag, call);
                asm volatile("" ::: "memory");
            }
        }
        pflag = cur.flag; cur = nxt;
    }
    asm volatile("s_waitcnt vmcnt(0)" ::: "memory");
    if (lane == 0) st_sys(dws + pflag, call);
    DBG_STAMP(1);
}

__device__ __forceinline__ void pull_role(const Params& P, int pw, unsigned call) {
    const int lane = threadIdx.x & 63, hw = __builtin_amdgcn_readfirstlane(threadIdx.x >> 6);
    if (hw >= 7) return;
    const int wid = (hw + pw) % 7;
    const int g = P.rank;
    char* myws = P.ws[g];
    const int d = (g + wid + 1) & 7;
    const __amdgpu_buffer_rsrc_t qctr = mk(myws + OFF_Q + (8 + wid + 1) * 256, 4);
    const unsigned qbase = (call - 1u) * PULL_STEP;
    unsigned* tmo = (unsigned*)(myws + OFF_TMO);
    unsigned tk0 = (unsigned)__builtin_amdgcn_raw_ptr_buffer_atomic_add_i32(1, qctr, lane * 4, 0, 0);
    unsigned tk = (unsigned)__builtin_amdgcn_raw_ptr_buffer_atomic_add_i32(1, qctr, lane * 4, 0, 0);
    bool bad = false;
    {
        const unsigned want = (call - 1u) * NCOMM + (pw < NPW0 ? ((pw + 1) * GATE0 + NPW0 - 1) / NPW0 : GATE1);
        const char* pd = myws + OFF_PD + (wid + 1) * 128;
        const unsigned long long t0 = __builtin_amdgcn_s_memrealtime();
        while (__builtin_amdgcn_readfirstlane(ld_sys(pd)) < want) {
            if (give_up(tmo, call, t0)) { bad = true; break; }
            __builtin_amdgcn_s_sleep(4);
        }
    }
    DBG_STAMP(5);
    const char* dps = P.ws[d] + OFF_PS + (call & 1) * PS_PAR + (size_t)(2 * g) * 16 * 131072;
    const char* dds = myws + OFF_DS + (size_t)(wid + 1) * 32 * 262144;
    char* ob = P.out + (size_t)d * NB * 2;
    const char* fo = myws + OFF_FO + ((wid + 1) * 64 + lane) * 4;
    const unsigned vl = lane * 16, vs = (lane & 15) * WP + (lane >> 4) * 16;
    unsigned rdy = bad ? ~0u : 0u; bool pre = false;
    u32x4 vp[4]; f32x4 vd[4][2];
    __amdgpu_buffer_rsrc_t rs = mk(myws, 0), rsd = mk(myws, 0);
    unsigned u = __builtin_amdgcn_readfirstlane(tk0) - qbase;
#define LOAD_P(item) vp[item] = __builtin_amdgcn_raw_buffer_load_b128(rs, vl + (item) * 1024, 0, SAUX)
#define LOAD_D(item) do { vd[item][0] = __builtin_bit_cast(f32x4, __builtin_amdgcn_raw_buffer_load_b128(rsd, vl + (item) * 2048, 0, SAUX)); \
        vd[item][1] = __builtin_bit_cast(f32x4, __builtin_amdgcn_raw_buffer_load_b128(rsd, vl + (item) * 2048 + 1024, 0, SAUX)); } while (0)
    while (u < NPULL) {
        const unsigned tile = u >> 5, part = u & 31;
        if (!pre) {
            const unsigned long long t0 = __builtin_amdgcn_s_memrealtime();
            while (!((rdy >> tile) & 1u)) {
                const unsigned long long m = __ballot(ld_sys(fo) >= call);
                rdy = (unsigned)m & (unsigned)(m >> 32);
                if ((rdy >> tile) & 1u) break;
                if (give_up(tmo, call, t0)) { rdy = ~0u; bad = true; break; }
                __builtin_amdgcn_s_sleep(4);
            }
            __builtin_amdgcn_fence(__ATOMIC_ACQUIRE, "");
            rs = mk(dps + (size_t)u * 4096, bad ? 0 : 4096); rsd = mk(dds + (size_t)u * 8192, 8192);
#pragma unroll
            for (int k = 0; k < DD; ++k) {
                if (k >= DD - DP) LOAD_P(k - (DD - DP));
                LOAD_D(k);
                __builtin_amdgcn_sched_barrier(0);
                __builtin_amdgcn_raw_buffer_store_b128((u32x4){0u, 0u, 0u, 0u}, mk(myws, 0), vl, 0, OAUX);
                __builtin_amdgcn_sched_barrier(0);
            }
        }
        const unsigned un = __builtin_amdgcn_readfirstlane(tk) - qbase;
        tk = (unsigned)__builtin_amdgcn_raw_ptr_buffer_atomic_add_i32(1, qctr, lane * 4, 0, 0);
        const __amdgpu_buffer_rsrc_t rd = mk(ob + (size_t)((tile >> 4) * 256 + (part >> 1) * 16) * WP + (tile & 15) * 512 + (part & 1) * 256, 15 * WP + 256);
        pre = un < NPULL && ((rdy >> ((un >> 5) & 31)) & 1u);
#pragma unroll
        for (int k = 0; k < 4; ++k) {
            if (k + DP == 4) rs = mk(dps + (size_t)(un & 1023) * 4096, pre && !bad ? 4096 : 0);
            if (k + DD == 4) rsd = mk(dds + (size_t)(un & 1023) * 8192, pre ? 8192 : 0);
            asm volatile("" : "+v"(vp[k]), "+v"(vd[k][0]), "+v"(vd[k][1]));
            __builtin_amdgcn_sched_barrier(0);
            LOAD_P((k + DP) & 3);
            LOAD_D((k + DD) & 3);
            __builtin_amdgcn_sched_barrier(0);
            const u32x4 p = vp[k]; const f32x4 d0 = vd[k][0], d1 = vd[k][1];
            u32x4 o;
            o.x = pk_bf16(silu(__uint_as_float(p.x << 16) + d0[0]), silu(__uint_as_float(p.x & 0xffff0000u) + d0[1]));
            o.y = pk_bf16(silu(__uint_as_float(p.y << 16) + d0[2]), silu(__uint_as_float(p.y & 0xffff0000u) + d0[3]));
            o.z = pk_bf16(silu(__uint_as_float(p.z << 16) + d1[0]), silu(__uint_as_float(p.z & 0xffff0000u) + d1[1]));
            o.w = pk_bf16(silu(__uint_as_float(p.w << 16) + d1[2]), silu(__uint_as_float(p.w & 0xffff0000u) + d1[3]));
            if (bad) o = (u32x4){QNAN2, QNAN2, QNAN2, QNAN2};
            __builtin_amdgcn_raw_buffer_store_b128(o, rd, vs + k * 64, 0, OAUX);
            __builtin_amdgcn_sched_barrier(0);
        }
        u = un;
    }
#undef LOAD_P
#undef LOAD_D
    DBG_STAMP(6);
}

__device__ __forceinline__ void gemm_tile(const Params& P, LAS unsigned char* lds, int mode, int pm, int pn, int qd, unsigned call) {
    const int tid = threadIdx.x, wid = __builtin_amdgcn_readfirstlane(tid >> 6), lane = tid & 63, wr = wid >> 2, wc = wid & 3, fr = lane & 15, fq = lane >> 4;
    const int g = P.rank;
    char* myws = P.ws[g];
    unsigned voffA[2], voffB[2];
#pragma unroll
    for (int i = 0; i < 2; ++i) {
        int R, C; stage_rc(tid * 16 + i * 8192, R, C); voffA[i] = (unsigned)R * XP + (unsigned)C * 2u;
        const int row = lane >> 3, ch = (lane & 7) ^ (((row >> 1) & 1) << 2);
        voffB[i] = (unsigned)(((wid >> 1) + 4 * i) * 8 + row) * WP + (unsigned)(wid & 1) * 128u + (unsigned)ch * 16u;
    }
    const unsigned ldsw = (unsigned)wid * 1024u;
    const int aoff = lds_byte(wr * 64 + fr, fq * 8);
    const int boff = fq * 2048 + (wc >> 1) * 1024 + (fr >> 2) * 128 + ((4 * (wc & 1) + (fr & 3)) ^ (((fr >> 3) & 1) << 2)) * 16;
    constexpr size_t hA = (size_t)128 * XP, hB = 256;
    const int nt = mode == 0 ? 64 : (mode == 1 ? 56 : 8);
    const char* xl = P.x + (size_t)pm * 256 * XP;
    const char* wl = P.w + (size_t)(mode == 2 ? (g + qd) & 7 : g) * NB * 2 + pn * 512;
    const char* xs = myws + OFF_XS + (call & 1) * XS_PAR + (size_t)pm * 256 * XP;
    const char* wsb = myws + OFF_WS + (call & 1) * WS_PAR + pn * 512;
    auto ktile = [&](int t, const char*& a, const char*& b, int& bit) {
        if (t < 8) { a = xl + t * 128; b = wl + (size_t)t * 64 * WP; bit = 0; }
        else { const unsigned u = t - 8, c = mode == 0 ? u / 7u : u / 6u, j = u - c * (mode == 0 ? 7u : 6u) + 1u, q = j + (mode != 0 && j >= (unsigned)qd);
            a = xs + (size_t)q * GM * XP + c * 128; b = wsb + (size_t)c * 64 * WP + (size_t)q * NB * 2; bit = c * 8 + q; }
    };
#define SA(b, h) (((b) * 2 + (h)) * HTB)
#define SB(b, h) ((4 + (b) * 2 + (h)) * HTB)
#define STAGE(bufoff, gbase, voff) do { _Pragma("unroll") for (int _i = 0; _i < 2; ++_i) \
        __builtin_amdgcn_global_load_lds((const unsigned*)((const char*)(gbase) + (voff)[_i]), (LAS unsigned*)(lds + (bufoff) + ldsw + _i * 8192), 16, 0, 0); } while (0)
#define LDA(dst, b, h) do { _Pragma("unroll") for (int m = 0; m < 4; ++m) _Pragma("unroll") for (int k = 0; k < 2; ++k) dst[m][k] = *(const LAS bf16x8*)(lds + SA(b, h) + aoff + m * 2048 + k * 1024); } while (0)
#define TRD(dst, off) asm volatile("ds_read_b64_tr_b16 %0, %1 offset:%2" : "=v"(dst) : "v"(baddr), "i"(off) : "memory")
#define LDB(dst, b, h) do { _Pragma("unroll") for (int n = 0; n < 2; ++n) _Pragma("unroll") for (int k = 0; k < 2; ++k) { \
        TRD(dst[n][k][0], ((b) * 2 + (h)) * HTB + k * 8192 + n * 8); TRD(dst[n][k][1], ((b) * 2 + (h)) * HTB + k * 8192 + 512 + n * 8); } } while (0)
#define WAIT_B(Bt) asm volatile("s_waitcnt lgkmcnt(0)" : "+v"(Bt[0][0][0]), "+v"(Bt[0][0][1]), "+v"(Bt[0][1][0]), "+v"(Bt[0][1][1]), "+v"(Bt[1][0][0]), "+v"(Bt[1][0][1]), "+v"(Bt[1][1][0]), "+v"(Bt[1][1][1]) :: "memory")
#define BF(Bt, n, k) __builtin_shufflevector(Bt[n][k][0], Bt[n][k][1], 0, 1, 2, 3, 4, 5, 6, 7)
#define MMA(ai, bj, At, Bt) do { __builtin_amdgcn_s_setprio(1); _Pragma("unroll") for (int m = 0; m < 4; ++m) _Pragma("unroll") for (int n = 0; n < 2; ++n) _Pragma("unroll") for (int k = 0; k < 2; ++k) \
        acc[ai][bj][m][n] = __builtin_amdgcn_mfma_f32_16x16x32_bf16(BF(Bt, n, k), At[m][k], acc[ai][bj][m][n], 0, 0, 0); __builtin_amdgcn_s_setprio(0); } while (0)
#define WAIT_V(n) asm volatile("s_waitcnt vmcnt(" #n ")" ::: "memory")
#define WAIT_L(n) asm volatile("s_waitcnt lgkmcnt(" #n ")" ::: "memory")
#define BAR __builtin_amdgcn_s_barrier()
#define SCHED __builtin_amdgcn_sched_barrier(0)
    f32x4 acc[2][2][4][2];
#pragma unroll
    for (int a = 0; a < 2; ++a)
#pragma unroll
        for (int b = 0; b < 2; ++b)
#pragma unroll
            for (int m = 0; m < 4; ++m)
#pragma unroll
                for (int n = 0; n < 2; ++n) acc[a][b][m][n] = (f32x4){0.f, 0.f, 0.f, 0.f};
    bf16x8 At[4][2]; s16x4 B0[2][2][2], B1[2][2][2];
    const unsigned baddr = (unsigned)(size_t)lds + 4 * HTB + boff;
    unsigned long long rdy = 1ull;
    LAS unsigned long long* rdy_lds = (LAS unsigned long long*)(lds + STAGE_BYTES);
    STAGE(SB(0, 0), wl, voffB); STAGE(SA(0, 0), xl, voffA); STAGE(SB(0, 1), wl + hB, voffB); STAGE(SA(0, 1), xl + hA, voffA);
    if (wr == 1) BAR;
    WAIT_V(4); BAR;
    STAGE(SB(1, 0), wl + (size_t)64 * WP, voffB); STAGE(SA(1, 0), xl + 128, voffA); STAGE(SB(1, 1), wl + (size_t)64 * WP + hB, voffB);
    WAIT_V(6); BAR;
    const char* a1 = xl + 128;
    for (int t = 0; t < nt; t += 2) {
        const char *a2, *b2, *a3, *b3; int bit2, bit3;
        const int t2 = t + 2 < nt ? t + 2 : 0;
        ktile(t2, a2, b2, bit2); ktile(t2 + 1, a3, b3, bit3);
        const unsigned long long need = (1ull << bit2) | (1ull << bit3);
        if ((rdy & need) != need) {
            if (wid == 0) {
                const char* fx = myws + OFF_FX + (pm * 64 + lane) * 8; const char* fw = myws + OFF_FW + (pn * 64 + lane) * 8;
                const unsigned long long t0 = __builtin_amdgcn_s_memrealtime();
                unsigned long long mask;
                for (;;) {
                    const unsigned long long vx = ld_sys64(fx), vw = ld_sys64(fw);
                    const bool ok = lane == 0 || ((unsigned)vx >= call && (unsigned)(vx >> 32) >= call && (unsigned)vw >= call && (unsigned)(vw >> 32) >= call);
                    mask = __ballot(ok);
                    if ((mask & need) == need) break;
                    if (give_up((unsigned*)(myws + OFF_TMO), call, t0)) { mask = ~0ull; break; }
                    __builtin_amdgcn_s_sleep(8);
                }
                __builtin_amdgcn_fence(__ATOMIC_ACQUIRE, "");
                if (lane == 0) *rdy_lds = mask;
                asm volatile("s_waitcnt vmcnt(0) lgkmcnt(0)" ::: "memory");
            }
            asm volatile("" ::: "memory"); BAR; asm volatile("" ::: "memory");
            const unsigned long long mv = *rdy_lds;
            rdy = ((unsigned long long)(unsigned)__builtin_amdgcn_readfirstlane((unsigned)(mv >> 32)) << 32) | (unsigned)__builtin_amdgcn_readfirstlane((unsigned)mv);
        }
        LDB(B0, 0, 0); SCHED; LDA(At, 0, 0); STAGE(SA(1, 1), a1 + hA, voffA);
        WAIT_L(8); BAR; WAIT_B(B0); MMA(0, 0, At, B0); BAR; SCHED;
        LDB(B1, 0, 1); STAGE(SB(0, 0), b2, voffB);
        BAR; WAIT_B(B1); MMA(0, 1, At, B1); BAR;
        LDA(At, 0, 1); STAGE(SA(0, 0), a2, voffA);
        BAR; WAIT_L(0); MMA(1, 0, At, B0); BAR; SCHED;
        STAGE(SB(0, 1), b2 + hB, voffB);
        WAIT_V(6); BAR; MMA(1, 1, At, B1); BAR;
        LDB(B0, 1, 0); SCHED; LDA(At, 1, 0); STAGE(SA(0, 1), a2 + hA, voffA);
        WAIT_L(8); BAR; WAIT_B(B0); MMA(0, 0, At, B0); BAR; SCHED;
        LDB(B1, 1, 1); STAGE(SB(1, 0), b3, voffB);
        BAR; WAIT_B(B1); MMA(0, 1, At, B1); BAR;
        LDA(At, 1, 1); STAGE(SA(1, 0), a3, voffA);
        BAR; WAIT_L(0); MMA(1, 0, At, B0); BAR; SCHED;
        STAGE(SB(1, 1), b3 + hB, voffB);
        WAIT_V(6); BAR; MMA(1, 1, At, B1); BAR;
        a1 = a3;
    }
    if (wr == 0) BAR;
    DBG_STAMP(mode == 2 ? 1 : 2);
    const bool bad = __builtin_amdgcn_readfirstlane(ld_agent((const unsigned*)(myws + OFF_TMO))) == call;
    int tz = threadIdx.x; asm volatile("" : "+v"(tz));
    const int l2 = tz & 63;
    if (mode == 2) {
        const __amdgpu_buffer_rsrc_t ro = mk(myws + OFF_DS + (size_t)(qd * 32 + (pm & 1) * 16 + pn) * 262144, 262144);
        const unsigned vo = (unsigned)(wr * 4 * 16384 + wc * 2048 + l2 * 16);
#pragma unroll
        for (int ai = 0; ai < 2; ++ai)
#pragma unroll
            for (int m = 0; m < 4; ++m)
#pragma unroll
                for (int bj = 0; bj < 2; ++bj)
#pragma unroll
                    for (int n = 0; n < 2; ++n)
                        __builtin_amdgcn_raw_buffer_store_b128(__builtin_bit_cast(u32x4, acc[ai][bj][m][n]), ro, vo + (unsigned)((ai * 8 + m) * 16384 + bj * 8192 + n * 1024), 0, SAUX);
    } else if (mode == 1) {
        const __amdgpu_buffer_rsrc_t ro = mk(myws + OFF_PS + (call & 1) * PS_PAR + (size_t)(pm * 16 + pn) * 131072, 131072);
        const unsigned vo = (unsigned)(wr * 4 * 8192 + wc * 1024 + l2 * 16);
#pragma unroll
        for (int ai = 0; ai < 2; ++ai)
#pragma unroll
            for (int m = 0; m < 4; ++m)
#pragma unroll
                for (int bj = 0; bj < 2; ++bj) {
                    const f32x4 v0 = acc[ai][bj][m][0], v1 = acc[ai][bj][m][1];
                    u32x4 o = {pk_bf16(v0[0], v0[1]), pk_bf16(v0[2], v0[3]), pk_bf16(v1[0], v1[1]), pk_bf16(v1[2], v1[3])};
                    if (bad) o = (u32x4){QNAN2, QNAN2, QNAN2, QNAN2};
                    __builtin_amdgcn_raw_buffer_store_b128(o, ro, vo + (unsigned)((ai * 8 + m) * 8192 + bj * 4096), 0, SAUX);
                }
    } else {
        const __amdgpu_buffer_rsrc_t ro = mk(P.out + (size_t)(pm & 1) * 256 * WP + (size_t)g * NB * 2 + pn * 512, 255 * WP + 512);
        const unsigned vo = (unsigned)(wr * 64 + (l2 & 15)) * WP + (unsigned)(wc * 32 + 8 * (l2 >> 4)) * 2u;
#pragma unroll
        for (int ai = 0; ai < 2; ++ai)
#pragma unroll
            for (int m = 0; m < 4; ++m)
#pragma unroll
                for (int bj = 0; bj < 2; ++bj) {
                    const f32x4 v0 = acc[ai][bj][m][0], v1 = acc[ai][bj][m][1];
                    u32x4 o = {pk_bf16(silu(v0[0]), silu(v0[1])), pk_bf16(silu(v0[2]), silu(v0[3])), pk_bf16(silu(v1[0]), silu(v1[1])), pk_bf16(silu(v1[2]), silu(v1[3]))};
                    if (bad) o = (u32x4){QNAN2, QNAN2, QNAN2, QNAN2};
                    __builtin_amdgcn_raw_buffer_store_b128(o, ro, vo + (unsigned)(ai * 8 + m) * 16 * WP + bj * 256, 0, SAUX);
                }
    }
    WAIT_V(0);
    BAR;
    if (tid == 0) {
        if (mode == 1) st_sys(P.ws[pm >> 1] + OFF_FO + ((8 - qd) * 64 + (pm & 1) * 16 + pn) * 4, call);
        if (mode == 2) st_sys(myws + OFF_FO + (qd * 64 + 32 + (pm & 1) * 16 + pn) * 4, call);
    }
    DBG_STAMP(mode == 2 ? 4 : 3);
#undef SA
#undef SB
#undef STAGE
#undef LDA
#undef LDB
#undef TRD
#undef WAIT_B
#undef BF
#undef MMA
}

__global__ __launch_bounds__(512, 2) void fused_gemm_rs(const Params P) {
    extern __shared__ __attribute__((aligned(16))) unsigned char shm[];
    const int b = blockIdx.x, xcd = b & 7, slot = b >> 3;
    unsigned* cnt = (unsigned*)(P.ws[P.rank] + OFF_CNT + (size_t)b * 128);
    const unsigned cntv = ld_agent(cnt);
    DBG_STAMP(0);
    int pw = -1;
    if (slot < CPX) {
        push_role(P, b, cntv);
        if (J1) pw = NPW0 + b;
    } else {
        const unsigned call = __builtin_amdgcn_readfirstlane(cntv) + 1u;
        const int g = P.rank, s = slot - CPX;
        for (int u = (s >= 14 ? 0 : 2); u < (s < 4 ? 4 : 3); ++u) {
            int mode, pm, pn, qd;
            if (u < 2) { const int e = (s - 14) * 8 + xcd; mode = 2; pm = 2 * g + u; pn = e / 7; qd = e % 7 + 1; }
            else if (u == 2) { mode = 1; qd = s % 7 + 1; pm = 2 * ((g + qd) & 7) + (s >= 14); pn = xcd + 8 * ((s / 7) & 1); }
            else { mode = 0; qd = 0; pm = 2 * g + (s >> 1); pn = xcd + 8 * (s & 1); }
            gemm_tile(P, (LAS unsigned char*)shm, mode, pm, pn, qd, call);
        }
        if (s >= 4 && s < (J2 ? 14 : 7)) pw = (s - 4) * 8 + xcd;
    }
    if (pw >= 0) pull_role(P, pw, __builtin_amdgcn_readfirstlane(cntv) + 1u);
    __syncthreads();
    if (threadIdx.x == 0) __hip_atomic_store(cnt, cntv + 1u, __ATOMIC_RELAXED, __HIP_MEMORY_SCOPE_AGENT);
}

void build_schedule(unsigned* sched) {
    unsigned char e[240]; int n = 0;
    auto W = [&](int j0, int j1, int c) { for (int j = j0; j < j1; ++j) e[n++] = (unsigned char)((j << 3) | c); };
    auto X = [&](int u0, int u1, int c) { for (int u = u0; u < u1; ++u) e[n++] = (unsigned char)(128 | (u << 3) | c); };
    for (int c = 0; c < 8; ++c) { X(2, 8, c); W(0, 8, c); if (c & 1) W(8, 16, c >> 1); }
    for (int c = 0; c < 4; ++c) { X(8, 14, c); W(8, 16, 4 + c); X(0, 2, c); }
    for (int c = 4; c < 8; ++c) { X(8, 14, c); X(0, 2, c); }
    for (int i = 0; i < 60; ++i) sched[i] = e[4 * i] | ((unsigned)e[4 * i + 1] << 8) | ((unsigned)e[4 * i + 2] << 16) | ((unsigned)e[4 * i + 3] << 24);
}
}

extern "C" void kernel_launch(int rank, int nranks, ncclComm_t comm, hipStream_t stream,
                              void* const* d_in, const int* in_sizes, int n_in,
                              void* d_out, int out_size, const int* in_dtypes, int out_dtype,
                              void* const* d_out_all, void* const* d_ws_all, size_t ws_bytes) {
    if (nranks != NR || (unsigned)rank >= (unsigned)NR || n_in < 2 || ws_bytes < WS_NEED ||
        in_sizes[0] != GM * KS || in_sizes[1] != KS * GN || out_size != MS * GN) {
        fprintf(stderr, "kernel_launch: unexpected problem shape\n");
        return;
    }
    HIP_CHECK(hipFuncSetAttribute((const void*)fused_gemm_rs, hipFuncAttributeMaxDynamicSharedMemorySize, LDS_BYTES));
    Params P;
    P.x = (const char*)d_in[0]; P.w = (const char*)d_in[1];
    for (int p = 0; p < NR; ++p) P.ws[p] = (char*)d_ws_all[p];
    P.out = (char*)d_out;
    P.rank = rank;
    build_schedule(P.sched);
    hipLaunchKernelGGL(fused_gemm_rs, dim3(GRID), dim3(512), LDS_BYTES, stream, P);
    HIP_CHECK(hipGetLastError());
}
```

```cpp
#include <hip/hip_runtime.h>
#include <rccl/rccl.h>
#include <cstdio>

#define HIP_CHECK(expr) do { hipError_t _e = (expr); if (_e != hipSuccess) { \
    fprintf(stderr, "HIP error %s at %s:%d\n", hipGetErrorString(_e), __FILE__, __LINE__); \
    return; } } while (0)

#define LAS __attribute__((address_space(3)))
typedef short s16x4 __attribute__((ext_vector_type(4)));
typedef short bf16x8 __attribute__((ext_vector_type(8)));
typedef float f32x4 __attribute__((ext_vector_type(4)));
typedef float f32x2 __attribute__((ext_vector_type(2)));
typedef __bf16 bf16x2 __attribute__((ext_vector_type(2)));
typedef unsigned u32x4 __attribute__((ext_vector_type(4)));

namespace {
constexpr int NR = 8;
constexpr int GM = 4096, GK = 4096, GN = 32768;
constexpr int KS = GK / NR;
constexpr int NB = GN / NR;
constexpr int MS = GM / NR;
constexpr unsigned XP = KS * 2;
constexpr unsigned WP = GN * 2;
constexpr int HTB = 128 * 64 * 2;
constexpr int STAGE_BYTES = 8 * HTB;
constexpr int LDS_BYTES = STAGE_BYTES + 64;
constexpr int GRID = 256;
constexpr int CPX = 4;
constexpr int NCOMM = 8 * CPX;
constexpr int NPW0 = 24;
constexpr int NPW = NPW0 + NCOMM;
constexpr unsigned long long DEADLINE = 50000000ull;

constexpr size_t OFF_CNT = 0;
constexpr size_t OFF_TMO = 32768;
constexpr size_t OFF_GDC = OFF_TMO + 256;
constexpr size_t OFF_FX = 65536;
constexpr size_t OFF_FW = OFF_FX + 16384;
constexpr size_t OFF_AV = OFF_FW + 16384;
constexpr size_t OFF_FO = OFF_AV + 8192;
constexpr size_t OFF_Q = OFF_FO + 2048;
constexpr size_t OFF_GD = OFF_Q + 4096;
constexpr size_t OFF_GO = OFF_GD + 1024;
constexpr size_t OFF_DBG = 131072;
constexpr size_t OFF_XS = 1u << 20;
constexpr size_t OFF_WS = OFF_XS + (size_t)NR * GM * XP;
constexpr size_t OFF_PS = OFF_WS + (size_t)KS * WP;
constexpr size_t PS_PAR = (size_t)256 * 131072;
constexpr size_t OFF_DS = OFF_PS + 2 * PS_PAR;
constexpr size_t OFF_OB = OFF_DS + (size_t)256 * 262144;
constexpr size_t WS_NEED = OFF_OB + (size_t)8 * 240 * 32768;
#define NP0 4
constexpr unsigned NBLK = 240 - NP0;
constexpr unsigned NIN = 4 * NBLK;
constexpr unsigned NPULL = 1024, PULL_STEP = NPULL + 2 * NPW;

struct Params {
    const char* x; const char* w;
    char* ws[NR]; char* out;
    int rank;
    unsigned sched[60];
};

__device__ __forceinline__ int lds_byte(int r, int c) { const int st = (r >> 4) * 2 + (c >> 5), rr = r & 15, cc = c & 31, ob = rr * 64 + cc * 2; return st * 1024 + (ob ^ (((ob >> 9) & 1) << 5)); }
__device__ __forceinline__ void stage_rc(int b, int& R, int& C) { const int st = b / 1024, sb = b % 1024, swz = sb ^ (((sb >> 9) & 1) << 5); R = (st >> 1) * 16 + swz / 64; C = (st & 1) * 32 + (swz % 64) / 2; }
__device__ __forceinline__ __amdgpu_buffer_rsrc_t mk(const void* p, unsigned bytes) { return __builtin_amdgcn_make_buffer_rsrc((void*)p, 0, bytes, 0x00020000); }
__device__ __forceinline__ unsigned pk_bf16(float lo, float hi) { const bf16x2 r = __builtin_convertvector((f32x2){lo, hi}, bf16x2); return __builtin_bit_cast(unsigned, r); }
__device__ __forceinline__ float silu(float y) {
    const float c = __builtin_amdgcn_fmed3f(y, -60.0f, 60.0f);
    const float e = __builtin_amdgcn_exp2f(c * -1.44269504088896341f);
    return y * __builtin_amdgcn_rcpf(1.0f + e);
}
__device__ __forceinline__ unsigned ld_agent(const unsigned* p) { return __hip_atomic_load(p, __ATOMIC_RELAXED, __HIP_MEMORY_SCOPE_AGENT); }
__device__ __forceinline__ unsigned ld_sys(const void* p) { return __hip_atomic_load((const unsigned*)p, __ATOMIC_RELAXED, __HIP_MEMORY_SCOPE_SYSTEM); }
__device__ __forceinline__ unsigned long long ld_sys64(const void* p) { return __hip_atomic_load((const unsigned long long*)p, __ATOMIC_RELAXED, __HIP_MEMORY_SCOPE_SYSTEM); }
__device__ __forceinline__ void st_sys(void* p, unsigned v) { __hip_atomic_store((unsigned*)p, v, __ATOMIC_RELAXED, __HIP_MEMORY_SCOPE_SYSTEM); }
__device__ __forceinline__ bool give_up(unsigned* tmo, unsigned call, unsigned long long t0) {
    if (__builtin_amdgcn_readfirstlane(ld_agent(tmo)) != call && __builtin_amdgcn_s_memrealtime() - t0 <= DEADLINE) return false;
    if ((threadIdx.x & 63) == 0) __hip_atomic_store(tmo, call, __ATOMIC_RELAXED, __HIP_MEMORY_SCOPE_AGENT);
    return true;
}
__device__ __forceinline__ int panel_of(int u, int d, int q) {
    if (u < 2) return 2 * d + u;
    const int hi = u >= 8, j = u - (hi ? 7 : 1);
    return 2 * ((d + j + (j >= q)) & 7) + hi;
}

#define DBG_STAMP(k) do { } while (0)
#define DBG_PROG(link, k) do { } while (0)

#define LEAD0 32
#define LEAD1 48
#define PSLEEP 16
#define DS 2
#define DP 1
#define DD 3
#define OAUX 2
constexpr int SAUX = 17;
constexpr unsigned QNAN2 = 0x7fc07fc0u;

struct Half { unsigned isx, src, dst, flag; };
__device__ __forceinline__ Half half_of(const Params& P, unsigned s, int d, int q) {
    const unsigned e = (P.sched[s >> 3] >> (((s >> 1) & 3) * 8)) & 0xffu;
    const unsigned c = e & 7, idx = (e >> 3) & 15, h = s & 1;
    Half r; r.isx = e >> 7;
    if (r.isx) {
        const unsigned pn = panel_of(idx, d, q);
        r.src = (pn * 256 + h * 128) * XP + c * 128; r.dst = (unsigned)OFF_XS + q * GM * XP + r.src;
        r.flag = (unsigned)OFF_FX + ((pn * 64 + c * 8 + q) * 2 + h) * 8;
    } else {
        const unsigned o = c * 64 * WP + idx * 512 + h * 256;
        r.src = o + d * NB * 2; r.dst = (unsigned)OFF_WS + q * NB * 2 + o;
        r.flag = (unsigned)OFF_FW + ((idx * 64 + c * 8 + q) * 2 + h) * 8;
    }
    return r;
}
constexpr unsigned XB = 127 * XP + 128, WB = 63 * WP + 256;
__device__ __forceinline__ bool wait_ge(const char* word, unsigned want, unsigned* tmo, unsigned call) {
    const unsigned long long t0 = __builtin_amdgcn_s_memrealtime();
    for (unsigned polls = 1; __builtin_amdgcn_readfirstlane(ld_sys(word)) < want; ++polls) {
        if ((polls & 15u) == 0u && give_up(tmo, call, t0)) return false;
        __builtin_amdgcn_s_sleep(1);
    }
    return true;
}

__device__ __forceinline__ void in_role(const Params& P, int cw, unsigned call) {
    const int lane = threadIdx.x & 63, hw = __builtin_amdgcn_readfirstlane(threadIdx.x >> 6);
    if (hw >= 7) return;
    const int wid = (hw + cw) % 7;
    const int g = P.rank;
    char* myws = P.ws[g];
    const __amdgpu_buffer_rsrc_t qctr = mk(myws + OFF_Q + (wid + 1) * 256, 4);
    const __amdgpu_buffer_rsrc_t rav = mk(myws + OFF_AV + (wid + 1) * 1024, 1024);
    const char* obox = P.ws[(g + wid + 1) & 7] + OFF_OB + (size_t)(7 - wid) * 240 * 32768;
    const unsigned vx = (lane >> 3) * XP + (lane & 7) * 16, vw = (lane >> 4) * WP + (lane & 15) * 16, vl = lane * 16;
    unsigned* tmo = (unsigned*)(myws + OFF_TMO);
    const unsigned qbase = (call - 1u) * NIN - 2 * NCOMM;
    unsigned s = cw, pflag = 0, tk = qbase + NCOMM + cw;
    bool bad = false, pre = false;
    __amdgpu_buffer_rsrc_t rs = mk(myws, 0);
    u32x4 v[4];
    for (;;) {
        const Half hf = half_of(P, (s < NIN ? s : 0) >> 1, g, wid + 1);
        if (!pre) {
            if (pflag) {
                asm volatile("s_waitcnt vmcnt(0)" ::: "memory");
                if (lane == 0) st_sys(myws + pflag, call);
                pflag = 0;
            }
            if (s >= NIN) break;
            if (!wait_ge(myws + OFF_AV + (wid + 1) * 1024 + (s >> 2) * 4, call, tmo, call)) bad = true;
            rs = mk(obox + (size_t)s * 8192, bad ? 0 : 8192);
#pragma unroll
            for (int k = 0; k < DS; ++k) {
                v[k] = __builtin_amdgcn_raw_buffer_load_b128(rs, vl + k * 1024, 0, SAUX); __builtin_amdgcn_sched_barrier(0);
                __builtin_amdgcn_raw_buffer_store_b128((u32x4){0u, 0u, 0u, 0u}, mk(myws, 0), vl, 0, SAUX); __builtin_amdgcn_sched_barrier(0);
            }
        }
        const unsigned ss = hf.isx ? 8 * XP : 4 * WP;
        const __amdgpu_buffer_rsrc_t rd = mk(myws + hf.dst + (s & 1) * 8 * ss, hf.isx ? 63 * XP + 128 : 31 * WP + 256);
        const unsigned sv = hf.isx ? vx : vw, flag = hf.flag + (s & 1) * 4;
        s = __builtin_amdgcn_readfirstlane(tk) - qbase;
        tk = (unsigned)__builtin_amdgcn_raw_ptr_buffer_atomic_add_i32(1, qctr, lane * 4, 0, 0);
        if (s >= NIN - LEAD0 && lane == 0)
            __hip_atomic_fetch_max((unsigned*)(myws + OFF_GO + (wid + 1) * 128), call * 1024u + (s - (NIN - LEAD0)), __ATOMIC_RELAXED, __HIP_MEMORY_SCOPE_AGENT);
        unsigned av = 0;
#define IN_ITEM(k) do { \
            if (k == 2) av = __builtin_amdgcn_raw_buffer_load_b32(rav, 0, (s >> 2) * 4 & 1023, SAUX); \
            if (k + DS == 8) {              \
                pre = s < NIN && (__builtin_amdgcn_readfirstlane(av) >= call || bad); \
                rs = mk(obox + (size_t)(s < NIN ? s : 0) * 8192, pre && !bad ? 8192 : 0); \
            } \
            asm volatile("" : "+v"(v[k & 3]));        \
            __builtin_amdgcn_sched_barrier(0); \
            if (k == DS && pflag) {         \
                asm volatile("s_waitcnt vmcnt(%0)" :: "i"(2 * DS) : "memory"); \
                if (lane == 0) st_sys(myws + pflag, call); \
                asm volatile("" ::: "memory"); \
            } \
            v[(k + DS) & 3] = __builtin_amdgcn_raw_buffer_load_b128(rs, vl, ((k + DS) & 7) * 1024, SAUX); \
            __builtin_amdgcn_sched_barrier(0); \
            __builtin_amdgcn_raw_buffer_store_b128(v[k & 3], rd, sv, k * ss, SAUX); \
            __builtin_amdgcn_sched_barrier(0); \
            asm volatile("s_nop 1");        \
            __builtin_amdgcn_sched_barrier(0); } while (0)
        IN_ITEM(0); IN_ITEM(1); IN_ITEM(2); IN_ITEM(3); IN_ITEM(4); IN_ITEM(5); IN_ITEM(6); IN_ITEM(7);
#undef IN_ITEM
        pflag = flag;
    }
    DBG_STAMP(1);
}

__device__ __forceinline__ void copy_role(const Params& P, int dd, unsigned n0, unsigned n1, bool push, unsigned call) {
    const int lane = threadIdx.x & 63, wv = __builtin_amdgcn_readfirstlane(threadIdx.x >> 6);
    const int g = P.rank, d = (g + dd) & 7;
    char* myws = P.ws[g]; char* dws = P.ws[d];
    const unsigned gdv = ld_sys(myws + OFF_GD + dd * 128);
    const unsigned vx = (lane >> 3) * XP + (lane & 7) * 16, vw = (lane >> 4) * WP + (lane & 15) * 16;
    const char* const px = P.x; const char* const pwm = P.w;
    u32x4 a[4], b[4];
    Half ha, hb;
    auto fetch = [&](unsigned n, u32x4 (&r)[4], Half& hf) {
        hf = half_of(P, 2 * (n < n1 ? n : 0) + (wv >> 2), d, 8 - dd);
        const __amdgpu_buffer_rsrc_t rs = mk((hf.isx ? px : pwm) + hf.src, n < n1 ? (hf.isx ? XB : WB) : 0);
        const unsigned st = hf.isx ? 8 * XP : 4 * WP;
#pragma unroll
        for (int j = 0; j < 4; ++j) r[j] = __builtin_amdgcn_raw_buffer_load_b128(rs, hf.isx ? vx : vw, ((wv & 3) * 4 + j) * st, 0);
    };
    fetch(n0, a, ha);
    if (__builtin_amdgcn_readfirstlane(gdv) < call - 1u)
        wait_ge(myws + OFF_GD + dd * 128, call - 1u, (unsigned*)(myws + OFF_TMO), call);
    for (unsigned n = n0; n < n1; n += 16) {
        const unsigned st = push ? (ha.isx ? 8 * XP : 4 * WP) : 1024u;
        const __amdgpu_buffer_rsrc_t rd = push ? mk(dws + ha.dst, ha.isx ? XB : WB) : mk(myws + OFF_OB + (size_t)(dd * 240 + n) * 32768 + (wv >> 2) * 16384, 16384);
        const unsigned vo = (push ? (ha.isx ? vx : vw) : lane * 16u) + (wv & 3) * 4 * st;
#pragma unroll
        for (int j = 0; j < 4; ++j) __builtin_amdgcn_raw_buffer_store_b128(a[j], rd, vo + j * st, 0, SAUX);
        fetch(n + 16, b, hb);
        asm volatile("s_waitcnt vmcnt(4)" ::: "memory");
        __builtin_amdgcn_s_barrier();
        if (push) { if ((threadIdx.x & 255) < 2) st_sys(dws + ha.flag + (threadIdx.x & 1) * 4, call); }
        else if (threadIdx.x == 0) st_sys(dws + OFF_AV + ((8 - dd) * 256 + n) * 4, call);
#pragma unroll
        for (int j = 0; j < 4; ++j) a[j] = b[j];
        ha = hb;
    }
    DBG_STAMP(5);
}

#define PORDER 1
__device__ __forceinline__ unsigned ptile(unsigned o) { return PORDER ? (o & 7) | ((o & 8) << 1) | ((o & 16) >> 1) : o; }
__device__ __forceinline__ void pull_role(const Params& P, int pw, unsigned call) {
    const int lane = threadIdx.x & 63, hw = __builtin_amdgcn_readfirstlane(threadIdx.x >> 6);
    if (hw >= 7) return;
    const int wid = (hw + pw) % 7;
    const int g = P.rank;
    char* myws = P.ws[g];
    const int d = (g + wid + 1) & 7;
    const __amdgpu_buffer_rsrc_t qctr = mk(myws + OFF_Q + (8 + wid + 1) * 256, 4);
    const unsigned qbase = (call - 1u) * PULL_STEP;
    unsigned* tmo = (unsigned*)(myws + OFF_TMO);
    unsigned tk0 = (unsigned)__builtin_amdgcn_raw_ptr_buffer_atomic_add_i32(1, qctr, lane * 4, 0, 0);
    unsigned tk = (unsigned)__builtin_amdgcn_raw_ptr_buffer_atomic_add_i32(1, qctr, lane * 4, 0, 0);
    bool bad = false;
    if (pw < NPW0) {
        const unsigned want = call * 1024u + pw * LEAD1 / NPW0;
        const unsigned* go = (const unsigned*)(myws + OFF_GO + (wid + 1) * 128);
        const unsigned long long t0 = __builtin_amdgcn_s_memrealtime();
        for (unsigned polls = 1; __builtin_amdgcn_readfirstlane(ld_agent(go)) < want; ++polls) {
            if ((polls & 15u) == 0u && give_up(tmo, call, t0)) { bad = true; break; }
            __builtin_amdgcn_s_sleep(16);
        }
    }
    DBG_STAMP(5);
    const char* dps = P.ws[d] + OFF_PS + (call & 1) * PS_PAR + (size_t)(2 * g) * 16 * 131072;
    const char* dds = myws + OFF_DS + (size_t)(wid + 1) * 32 * 262144;
    char* ob = P.out + (size_t)d * NB * 2;
    const char* fo = myws + OFF_FO + ((wid + 1) * 64 + lane) * 4;
    const unsigned vl = lane * 16, vs = (lane & 15) * WP + (lane >> 4) * 16;
    const __amdgpu_buffer_rsrc_t rfo = mk(myws + OFF_FO + (wid + 1) * 256, 256);
    unsigned rdy = bad ? ~0u : 0u, fv = 0; bool pre = false;
    u32x4 vp[4]; f32x4 vd[4][2];
    __amdgpu_buffer_rsrc_t rs = mk(myws, 0), rsd = mk(myws, 0);
    unsigned u = __builtin_amdgcn_readfirstlane(tk0) - qbase;
#define LOAD_P(item) vp[item] = __builtin_amdgcn_raw_buffer_load_b128(rs, vl + (item) * 1024, 0, SAUX)
#define LOAD_D(item) do { vd[item][0] = __builtin_bit_cast(f32x4, __builtin_amdgcn_raw_buffer_load_b128(rsd, vl + (item) * 2048, 0, SAUX)); \
        vd[item][1] = __builtin_bit_cast(f32x4, __builtin_amdgcn_raw_buffer_load_b128(rsd, vl + (item) * 2048 + 1024, 0, SAUX)); } while (0)
    while (u < NPULL) {
        const unsigned tile = ptile(u >> 5), part = u & 31;
        if (!pre) {
            const unsigned long long t0 = __builtin_amdgcn_s_memrealtime();
            while (!((rdy >> tile) & 1u)) {
                const unsigned long long m = __ballot(ld_sys(fo) >= call);
                rdy = (unsigned)m & (unsigned)(m >> 32);
                if ((rdy >> tile) & 1u) break;
                if (give_up(tmo, call, t0)) { rdy = ~0u; bad = true; break; }
                __builtin_amdgcn_s_sleep(4);
            }
            __builtin_amdgcn_fence(__ATOMIC_ACQUIRE, "");
            rs = mk(dps + (size_t)(tile * 32 + part) * 4096, bad ? 0 : 4096); rsd = mk(dds + (size_t)(tile * 32 + part) * 8192, 8192);
#pragma unroll
            for (int k = 0; k < DD; ++k) {
                if (k >= DD - DP) LOAD_P(k - (DD - DP));
                LOAD_D(k);
                __builtin_amdgcn_sched_barrier(0);
                __builtin_amdgcn_raw_buffer_store_b128((u32x4){0u, 0u, 0u, 0u}, mk(myws, 0), vl, 0, OAUX);
                __builtin_amdgcn_sched_barrier(0);
            }
        }
        const unsigned un = __builtin_amdgcn_readfirstlane(tk) - qbase;
        {
            const unsigned long long m = __ballot(fv >= call);
            rdy |= (unsigned)m & (unsigned)(m >> 32);
        }
        tk = (unsigned)__builtin_amdgcn_raw_ptr_buffer_atomic_add_i32(1, qctr, lane * 4, 0, 0);
        fv = __builtin_amdgcn_raw_buffer_load_b32(rfo, lane * 4, 0, SAUX);
        const __amdgpu_buffer_rsrc_t rd = mk(ob + (size_t)((tile >> 4) * 256 + (part >> 1) * 16) * WP + (tile & 15) * 512 + (part & 1) * 256, 15 * WP + 256);
        const unsigned tn = ptile((un >> 5) & 31), on = tn * 32 + (un & 31);
        pre = un < NPULL && ((rdy >> tn) & 1u);
#pragma unroll
        for (int k = 0; k < 4; ++k) {
            if (k + DP == 4) rs = mk(dps + (size_t)on * 4096, pre && !bad ? 4096 : 0);
            if (k + DD == 4) rsd = mk(dds + (size_t)on * 8192, pre ? 8192 : 0);
            asm volatile("" : "+v"(vp[k]), "+v"(vd[k][0]), "+v"(vd[k][1]));
            __builtin_amdgcn_sched_barrier(0);
            LOAD_P((k + DP) & 3);
            LOAD_D((k + DD) & 3);
            __builtin_amdgcn_sched_barrier(0);
            const u32x4 p = vp[k]; const f32x4 d0 = vd[k][0], d1 = vd[k][1];
            u32x4 o;
            o.x = pk_bf16(silu(__uint_as_float(p.x << 16) + d0[0]), silu(__uint_as_float(p.x & 0xffff0000u) + d0[1]));
            o.y = pk_bf16(silu(__uint_as_float(p.y << 16) + d0[2]), silu(__uint_as_float(p.y & 0xffff0000u) + d0[3]));
            o.z = pk_bf16(silu(__uint_as_float(p.z << 16) + d1[0]), silu(__uint_as_float(p.z & 0xffff0000u) + d1[1]));
            o.w = pk_bf16(silu(__uint_as_float(p.w << 16) + d1[2]), silu(__uint_as_float(p.w & 0xffff0000u) + d1[3]));
            if (bad) o = (u32x4){QNAN2, QNAN2, QNAN2, QNAN2};
            __builtin_amdgcn_raw_buffer_store_b128(o, rd, vs + k * 64, 0, OAUX);
            __builtin_amdgcn_sched_barrier(0);
        }
        u = un;
    }
#undef LOAD_P
#undef LOAD_D
    DBG_STAMP(6);
}

__device__ __forceinline__ void gemm_tile(const Params& P, LAS unsigned char* lds, int mode, int pm, int pn, int qd, unsigned call) {
    const int tid = threadIdx.x, wid = __builtin_amdgcn_readfirstlane(tid >> 6), lane = tid & 63, wr = wid >> 2, wc = wid & 3, fr = lane & 15, fq = lane >> 4;
    const int g = P.rank;
    char* myws = P.ws[g];
    unsigned voffA[2], voffB[2];
#pragma unroll
    for (int i = 0; i < 2; ++i) {
        int R, C; stage_rc(tid * 16 + i * 8192, R, C); voffA[i] = (unsigned)R * XP + (unsigned)C * 2u;
        const int row = lane >> 3, ch = (lane & 7) ^ (((row >> 1) & 1) << 2);
        voffB[i] = (unsigned)(((wid >> 1) + 4 * i) * 8 + row) * WP + (unsigned)(wid & 1) * 128u + (unsigned)ch * 16u;
    }
    const unsigned ldsw = (unsigned)wid * 1024u;
    const int aoff = lds_byte(wr * 64 + fr, fq * 8);
    const int boff = fq * 2048 + (wc >> 1) * 1024 + (fr >> 2) * 128 + ((4 * (wc & 1) + (fr & 3)) ^ (((fr >> 3) & 1) << 2)) * 16;
    constexpr size_t hA = (size_t)128 * XP, hB = 256;
    const int nt = mode == 0 ? 64 : (mode == 1 ? 56 : 8);
    const char* xl = P.x + (size_t)pm * 256 * XP;
    const char* wl = P.w + (size_t)(mode == 2 ? (g + qd) & 7 : g) * NB * 2 + pn * 512;
    const char* xs = myws + OFF_XS + (size_t)pm * 256 * XP;
    const char* wsb = myws + OFF_WS + pn * 512;
    auto ktile = [&](int t, const char*& a, const char*& b, int& bit) {
        if (t < 8) { a = xl + t * 128; b = wl + (size_t)t * 64 * WP; bit = 0; }
        else { const unsigned u = t - 8, c = mode == 0 ? u / 7u : u / 6u, j = u - c * (mode == 0 ? 7u : 6u) + 1u, q = j + (mode != 0 && j >= (unsigned)qd);
            a = xs + (size_t)q * GM * XP + c * 128; b = wsb + (size_t)c * 64 * WP + (size_t)q * NB * 2; bit = c * 8 + q; }
    };
#define SA(b, h) (((b) * 2 + (h)) * HTB)
#define SB(b, h) ((4 + (b) * 2 + (h)) * HTB)
#define STAGE(bufoff, gbase, voff) do { _Pragma("unroll") for (int _i = 0; _i < 2; ++_i) \
        __builtin_amdgcn_global_load_lds((const unsigned*)((const char*)(gbase) + (voff)[_i]), (LAS unsigned*)(lds + (bufoff) + ldsw + _i * 8192), 16, 0, 0); } while (0)
#define LDA(dst, b, h) do { _Pragma("unroll") for (int m = 0; m < 4; ++m) _Pragma("unroll") for (int k = 0; k < 2; ++k) dst[m][k] = *(const LAS bf16x8*)(lds + SA(b, h) + aoff + m * 2048 + k * 1024); } while (0)
#define TRD(dst, off) asm volatile("ds_read_b64_tr_b16 %0, %1 offset:%2" : "=v"(dst) : "v"(baddr), "i"(off) : "memory")
#define LDB(dst, b, h) do { _Pragma("unroll") for (int n = 0; n < 2; ++n) _Pragma("unroll") for (int k = 0; k < 2; ++k) { \
        TRD(dst[n][k][0], ((b) * 2 + (h)) * HTB + k * 8192 + n * 8); TRD(dst[n][k][1], ((b) * 2 + (h)) * HTB + k * 8192 + 512 + n * 8); } } while (0)
#define WAIT_B(Bt) asm volatile("s_waitcnt lgkmcnt(0)" : "+v"(Bt[0][0][0]), "+v"(Bt[0][0][1]), "+v"(Bt[0][1][0]), "+v"(Bt[0][1][1]), "+v"(Bt[1][0][0]), "+v"(Bt[1][0][1]), "+v"(Bt[1][1][0]), "+v"(Bt[1][1][1]) :: "memory")
#define BF(Bt, n, k) __builtin_shufflevector(Bt[n][k][0], Bt[n][k][1], 0, 1, 2, 3, 4, 5, 6, 7)
#define MMA(ai, bj, At, Bt) do { __builtin_amdgcn_s_setprio(1); _Pragma("unroll") for (int m = 0; m < 4; ++m) _Pragma("unroll") for (int n = 0; n < 2; ++n) _Pragma("unroll") for (int k = 0; k < 2; ++k) \
        acc[ai][bj][m][n] = __builtin_amdgcn_mfma_f32_16x16x32_bf16(BF(Bt, n, k), At[m][k], acc[ai][bj][m][n], 0, 0, 0); __builtin_amdgcn_s_setprio(0); } while (0)
#define WAIT_V(n) asm volatile("s_waitcnt vmcnt(" #n ")" ::: "memory")
#define WAIT_L(n) asm volatile("s_waitcnt lgkmcnt(" #n ")" ::: "memory")
#define BAR __builtin_amdgcn_s_barrier()
#define SCHED __builtin_amdgcn_sched_barrier(0)
    f32x4 acc[2][2][4][2];
#pragma unroll
    for (int a = 0; a < 2; ++a)
#pragma unroll
        for (int b = 0; b < 2; ++b)
#pragma unroll
            for (int m = 0; m < 4; ++m)
#pragma unroll
                for (int n = 0; n < 2; ++n) acc[a][b][m][n] = (f32x4){0.f, 0.f, 0.f, 0.f};
    bf16x8 At[4][2]; s16x4 B0[2][2][2], B1[2][2][2];
    const unsigned baddr = (unsigned)(size_t)lds + 4 * HTB + boff;
    unsigned long long rdy = 1ull;
    LAS unsigned long long* rdy_lds = (LAS unsigned long long*)(lds + STAGE_BYTES);
    STAGE(SB(0, 0), wl, voffB); STAGE(SA(0, 0), xl, voffA); STAGE(SB(0, 1), wl + hB, voffB); STAGE(SA(0, 1), xl + hA, voffA);
    if (wr == 1) BAR;
    WAIT_V(4); BAR;
    STAGE(SB(1, 0), wl + (size_t)64 * WP, voffB); STAGE(SA(1, 0), xl + 128, voffA); STAGE(SB(1, 1), wl + (size_t)64 * WP + hB, voffB);
    WAIT_V(6); BAR;
    const char* a1 = xl + 128;
    for (int t = 0; t < nt; t += 2) {
        const char *a2, *b2, *a3, *b3; int bit2, bit3;
        const int t2 = t + 2 < nt ? t + 2 : 0;
        ktile(t2, a2, b2, bit2); ktile(t2 + 1, a3, b3, bit3);
        const unsigned long long need = (1ull << bit2) | (1ull << bit3);
        if ((rdy & need) != need) {
            if (wid == 0) {
                const __amdgpu_buffer_rsrc_t rfx = mk(myws + OFF_FX + pm * 1024, 1024), rfw = mk(myws + OFF_FW + pn * 1024, 1024);
                const unsigned long long t0 = __builtin_amdgcn_s_memrealtime();
                unsigned long long mask;
                const int l0 = bit2 & ~7;
                unsigned vo = lane >= l0 ? lane * 16 : 4096;
                for (unsigned polls = 1;; ++polls) {
                    const u32x4 vx = __builtin_amdgcn_raw_buffer_load_b128(rfx, vo, 0, SAUX), vw = __builtin_amdgcn_raw_buffer_load_b128(rfw, vo, 0, SAUX);
                    asm volatile("" ::: "memory");
                    const bool ok = min(min(vx.x, vx.y), min(vx.z, vx.w)) >= call && min(min(vw.x, vw.y), min(vw.z, vw.w)) >= call;
                    mask = __ballot(ok) | rdy;
                    if ((mask & need) == need) break;
                    if ((polls & 7u) == 0u && give_up((unsigned*)(myws + OFF_TMO), call, t0)) { mask = ~0ull; break; }
                    if (lane >= l0 + 16) vo = 4096;
                    __builtin_amdgcn_s_sleep(PSLEEP);
                }
                __builtin_amdgcn_fence(__ATOMIC_ACQUIRE, "");
                if (lane == 0) *rdy_lds = mask;
                asm volatile("s_waitcnt vmcnt(0) lgkmcnt(0)" ::: "memory");
            }
            asm volatile("" ::: "memory"); BAR; asm volatile("" ::: "memory");
            const unsigned long long mv = *rdy_lds;
            rdy = ((unsigned long long)(unsigned)__builtin_amdgcn_readfirstlane((unsigned)(mv >> 32)) << 32) | (unsigned)__builtin_amdgcn_readfirstlane((unsigned)mv);
        }
        LDB(B0, 0, 0); SCHED; LDA(At, 0, 0); STAGE(SA(1, 1), a1 + hA, voffA);
        WAIT_L(8); BAR; WAIT_B(B0); MMA(0, 0, At, B0); BAR; SCHED;
        LDB(B1, 0, 1); STAGE(SB(0, 0), b2, voffB);
        BAR; WAIT_B(B1); MMA(0, 1, At, B1); BAR;
        LDA(At, 0, 1); STAGE(SA(0, 0), a2, voffA);
        BAR; WAIT_L(0); MMA(1, 0, At, B0); BAR; SCHED;
        STAGE(SB(0, 1), b2 + hB, voffB);
        WAIT_V(6); BAR; MMA(1, 1, At, B1); BAR;
        LDB(B0, 1, 0); SCHED; LDA(At, 1, 0); STAGE(SA(0, 1), a2 + hA, voffA);
        WAIT_L(8); BAR; WAIT_B(B0); MMA(0, 0, At, B0); BAR; SCHED;
        LDB(B1, 1, 1); STAGE(SB(1, 0), b3, voffB);
        BAR; WAIT_B(B1); MMA(0, 1, At, B1); BAR;
        LDA(At, 1, 1); STAGE(SA(1, 0), a3, voffA);
        BAR; WAIT_L(0); MMA(1, 0, At, B0); BAR; SCHED;
        STAGE(SB(1, 1), b3 + hB, voffB);
        WAIT_V(6); BAR; MMA(1, 1, At, B1); BAR;
        a1 = a3;
    }
    if (wr == 0) BAR;
    DBG_STAMP(mode == 2 ? 1 : 2);
    const bool bad = __builtin_amdgcn_readfirstlane(ld_agent((const unsigned*)(myws + OFF_TMO))) == call;
    int tz = threadIdx.x; asm volatile("" : "+v"(tz));
    const int l2 = tz & 63;
    if (mode == 2) {
        const __amdgpu_buffer_rsrc_t ro = mk(myws + OFF_DS + (size_t)(qd * 32 + (pm & 1) * 16 + pn) * 262144, 262144);
        const unsigned vo = (unsigned)(wr * 4 * 16384 + wc * 2048 + l2 * 16);
#pragma unroll
        for (int ai = 0; ai < 2; ++ai)
#pragma unroll
            for (int m = 0; m < 4; ++m)
#pragma unroll
                for (int bj = 0; bj < 2; ++bj)
#pragma unroll
                    for (int n = 0; n < 2; ++n)
                        __builtin_amdgcn_raw_buffer_store_b128(__builtin_bit_cast(u32x4, acc[ai][bj][m][n]), ro, vo + (unsigned)((ai * 8 + m) * 16384 + bj * 8192 + n * 1024), 0, SAUX);
    } else if (mode == 1) {
        const __amdgpu_buffer_rsrc_t ro = mk(myws + OFF_PS + (call & 1) * PS_PAR + (size_t)(pm * 16 + pn) * 131072, 131072);
        const unsigned vo = (unsigned)(wr * 4 * 8192 + wc * 1024 + l2 * 16);
#pragma unroll
        for (int ai = 0; ai < 2; ++ai)
#pragma unroll
            for (int m = 0; m < 4; ++m)
#pragma unroll
                for (int bj = 0; bj < 2; ++bj) {
                    const f32x4 v0 = acc[ai][bj][m][0], v1 = acc[ai][bj][m][1];
                    u32x4 o = {pk_bf16(v0[0], v0[1]), pk_bf16(v0[2], v0[3]), pk_bf16(v1[0], v1[1]), pk_bf16(v1[2], v1[3])};
                    if (bad) o = (u32x4){QNAN2, QNAN2, QNAN2, QNAN2};
                    __builtin_amdgcn_raw_buffer_store_b128(o, ro, vo + (unsigned)((ai * 8 + m) * 8192 + bj * 4096), 0, SAUX);
                }
    } else {
        const __amdgpu_buffer_rsrc_t ro = mk(P.out + (size_t)(pm & 1) * 256 * WP + (size_t)g * NB * 2 + pn * 512, 255 * WP + 512);
        const unsigned vo = (unsigned)(wr * 64 + (l2 & 15)) * WP + (unsigned)(wc * 32 + 8 * (l2 >> 4)) * 2u;
#pragma unroll
        for (int ai = 0; ai < 2; ++ai)
#pragma unroll
            for (int m = 0; m < 4; ++m)
#pragma unroll
                for (int bj = 0; bj < 2; ++bj) {
                    const f32x4 v0 = acc[ai][bj][m][0], v1 = acc[ai][bj][m][1];
                    u32x4 o = {pk_bf16(silu(v0[0]), silu(v0[1])), pk_bf16(silu(v0[2]), silu(v0[3])), pk_bf16(silu(v1[0]), silu(v1[1])), pk_bf16(silu(v1[2]), silu(v1[3]))};
                    if (bad) o = (u32x4){QNAN2, QNAN2, QNAN2, QNAN2};
                    __builtin_amdgcn_raw_buffer_store_b128(o, ro, vo + (unsigned)(ai * 8 + m) * 16 * WP + bj * 256, 0, SAUX);
                }
    }
    WAIT_V(0);
    BAR;
    if (tid == 0) {
        if (mode == 1) st_sys(P.ws[pm >> 1] + OFF_FO + ((8 - qd) * 64 + (pm & 1) * 16 + pn) * 4, call);
        if (mode == 2) st_sys(myws + OFF_FO + (qd * 64 + 32 + (pm & 1) * 16 + pn) * 4, call);
        else if (__hip_atomic_fetch_add((unsigned*)(myws + OFF_GDC), 1u, __ATOMIC_RELAXED, __HIP_MEMORY_SCOPE_AGENT) == 256u * call - 1u) {
            for (int o = 1; o < 8; ++o) st_sys(P.ws[(g + o) & 7] + OFF_GD + (8 - o) * 128, call);
        }
    }
    DBG_STAMP(mode == 2 ? 4 : 3);
#undef SA
#undef SB
#undef STAGE
#undef LDA
#undef LDB
#undef TRD
#undef WAIT_B
#undef BF
#undef MMA
}

__global__ __launch_bounds__(512, 2) void fused_gemm_rs(const Params P) {
    extern __shared__ __attribute__((aligned(16))) unsigned char shm[];
    const int b = blockIdx.x, xcd = b & 7, slot = b >> 3;
    unsigned* cnt = (unsigned*)(P.ws[P.rank] + OFF_CNT + (size_t)b * 128);
    const unsigned cntv = ld_agent(cnt);
    DBG_STAMP(0);
    int pw = -1;
    if (slot < CPX) {
        in_role(P, b, __builtin_amdgcn_readfirstlane(cntv) + 1u);
        pw = NPW0 + b;
    } else {
        const unsigned call = __builtin_amdgcn_readfirstlane(cntv) + 1u;
        const int g = P.rank, s = slot - CPX;
        const int cl = s / 7, pcl = PORDER ? 2 : 1;
        const bool late = cl != 0 && cl != pcl;
        const int e = ((late && cl == 3 ? 7 : 0) + s % 7) * 8 + xcd;
        if (cl != 0) copy_role(P, e % 7 + 1, (late ? 0 : NBLK) + e / 7, late ? NBLK : 240, !late, call);
        for (int u = (late ? 0 : 2); u < (s < 4 ? 4 : 3); ++u) {
            int mode, pm, pn, qd;
            if (u < 2) { mode = 2; pm = 2 * g + u; pn = e / 7; qd = e % 7 + 1; }
            else if (u == 2) { mode = 1; qd = s % 7 + 1; pm = 2 * ((g + qd) & 7) + (cl >> 1); pn = xcd + 8 * (cl & 1); }
            else { mode = 0; qd = 0; pm = 2 * g + (s >> 1); pn = xcd + 8 * (s & 1); }
            gemm_tile(P, (LAS unsigned char*)shm, mode, pm, pn, qd, call);
        }
        if (s >= 4 && s < 7) pw = (s - 4) * 8 + xcd;
    }
    if (pw >= 0) pull_role(P, pw, __builtin_amdgcn_readfirstlane(cntv) + 1u);
    __syncthreads();
    if (threadIdx.x == 0) __hip_atomic_store(cnt, cntv + 1u, __ATOMIC_RELAXED, __HIP_MEMORY_SCOPE_AGENT);
}

#define SCHED_V 4
void build_schedule(unsigned* sched) {
    unsigned char e[240]; int n = 0;
    auto W = [&](int j0, int j1, int c) { for (int j = j0; j < j1; ++j) e[n++] = (unsigned char)((j << 3) | c); };
    auto X = [&](int u0, int u1, int c) { for (int u = u0; u < u1; ++u) e[n++] = (unsigned char)(128 | (u << 3) | c); };
    for (int c = 0; c < 8; ++c) { X(2, 8, c); W(0, 8, c); if (c == 4) X(8, 14, 0); if (c == 6) X(8, 14, 1); }
    for (int c = 0; c < 6; ++c) { X(8, 14, c + 2); W(8, 16, c); X(0, 2, c); }
    W(8, 16, 6); W(8, 16, 7); X(0, 2, 6); X(0, 2, 7);
    for (int i = 0; i < 60; ++i) sched[i] = e[4 * i] | ((unsigned)e[4 * i + 1] << 8) | ((unsigned)e[4 * i + 2] << 16) | ((unsigned)e[4 * i + 3] << 24);
}
}

extern "C" void kernel_launch(int rank, int nranks, ncclComm_t comm, hipStream_t stream,
                              void* const* d_in, const int* in_sizes, int n_in,
                              void* d_out, int out_size, const int* in_dtypes, int out_dtype,
                              void* const* d_out_all, void* const* d_ws_all, size_t ws_bytes) {
    if (nranks != NR || (unsigned)rank >= (unsigned)NR || n_in < 2 || ws_bytes < WS_NEED ||
        in_sizes[0] != GM * KS || in_sizes[1] != KS * GN || out_size != MS * GN) {
        fprintf(stderr, "kernel_launch: unexpected problem shape\n");
        return;
    }
    HIP_CHECK(hipFuncSetAttribute((const void*)fused_gemm_rs, hipFuncAttributeMaxDynamicSharedMemorySize, LDS_BYTES));
    Params P;
    P.x = (const char*)d_in[0]; P.w = (const char*)d_in[1];
    for (int p = 0; p < NR; ++p) P.ws[p] = (char*)d_ws_all[p];
    P.out = (char*)d_out;
    P.rank = rank;
    build_schedule(P.sched);
    hipLaunchKernelGGL(fused_gemm_rs, dim3(GRID), dim3(512), LDS_BYTES, stream, P);
    HIP_CHECK(hipGetLastError());
}
```

```cpp
#include <hip/hip_runtime.h>
#include <rccl/rccl.h>
#include <cstdio>

#define HIP_CHECK(expr) do { hipError_t _e = (expr); if (_e != hipSuccess) { \
    fprintf(stderr, "HIP error %s at %s:%d\n", hipGetErrorString(_e), __FILE__, __LINE__); \
    return; } } while (0)

#define LAS __attribute__((address_space(3)))
typedef short s16x4 __attribute__((ext_vector_type(4)));
typedef short bf16x8 __attribute__((ext_vector_type(8)));
typedef float f32x4 __attribute__((ext_vector_type(4)));
typedef float f32x2 __attribute__((ext_vector_type(2)));
typedef __bf16 bf16x2 __attribute__((ext_vector_type(2)));
typedef unsigned u32x4 __attribute__((ext_vector_type(4)));

namespace {
constexpr int NR = 8;
constexpr int GM = 4096, GK = 4096, GN = 32768;
constexpr int KS = GK / NR;
constexpr int NB = GN / NR;
constexpr int MS = GM / NR;
constexpr unsigned XP = KS * 2;
constexpr unsigned WP = GN * 2;
constexpr int HTB = 128 * 64 * 2;
#define BSHIFT 1
constexpr int KBP = 4096 + 16 * BSHIFT;
constexpr int HTBB = 4 * KBP;
constexpr int STAGE_BYTES = 4 * HTB + 4 * HTBB;
constexpr int LDS_BYTES = STAGE_BYTES + 64;
constexpr int GRID = 256;
constexpr int CPX = 4;
constexpr int NCOMM = 8 * CPX;
constexpr int NPW0 = 24;
constexpr int NPW = NPW0 + NCOMM;
constexpr unsigned long long DEADLINE = 50000000ull;

constexpr size_t OFF_CNT = 0;
constexpr size_t OFF_TMO = 32768;
constexpr size_t OFF_GDC = OFF_TMO + 256;
constexpr size_t OFF_FX = 65536;
constexpr size_t OFF_FW = OFF_FX + 16384;
constexpr size_t OFF_AV = OFF_FW + 16384;
constexpr size_t OFF_FO = OFF_AV + 8192;
constexpr size_t OFF_Q = OFF_FO + 2048;
constexpr size_t OFF_GD = OFF_Q + 4096;
constexpr size_t OFF_GO = OFF_GD + 1024;
constexpr size_t OFF_AV2 = 524288;
constexpr size_t OFF_DBG = 131072;
constexpr size_t OFF_XS = 1u << 20;
constexpr size_t OFF_WS = OFF_XS + (size_t)NR * GM * XP;
constexpr size_t OFF_PS = OFF_WS + (size_t)KS * WP;
constexpr size_t PS_PAR = (size_t)256 * 131072;
constexpr size_t OFF_DS = OFF_PS + 2 * PS_PAR;
constexpr size_t OFF_OB = OFF_DS + (size_t)256 * 262144;
constexpr size_t WS_NEED = OFF_OB + (size_t)8 * 240 * 32768;
#define NP0 4
constexpr unsigned NBLK = 240 - NP0;
constexpr unsigned NIN = 4 * NBLK;
constexpr unsigned NPULL = 1024;

struct Params {
    const char* x; const char* w;
    char* ws[NR];
    char* out;
    int rank;
    unsigned sched[60];
};

__device__ __forceinline__ int lds_byte(int r, int c) { const int st = (r >> 4) * 2 + (c >> 5), rr = r & 15, cc = c & 31, ob = rr * 64 + cc * 2; return st * 1024 + (ob ^ (((ob >> 9) & 1) << 5)); }
__device__ __forceinline__ void stage_rc(int b, int& R, int& C) { const int st = b / 1024, sb = b % 1024, swz = sb ^ (((sb >> 9) & 1) << 5); R = (st >> 1) * 16 + swz / 64; C = (st & 1) * 32 + (swz % 64) / 2; }
__device__ __forceinline__ __amdgpu_buffer_rsrc_t mk(const void* p, unsigned bytes) { return __builtin_amdgcn_make_buffer_rsrc((void*)p, 0, bytes, 0x00020000); }
__device__ __forceinline__ unsigned pk_bf16(float lo, float hi) { const bf16x2 r = __builtin_convertvector((f32x2){lo, hi}, bf16x2); return __builtin_bit_cast(unsigned, r); }
__device__ __forceinline__ float silu(float y) {
    const float c = __builtin_amdgcn_fmed3f(y, -60.0f, 60.0f);
    const float e = __builtin_amdgcn_exp2f(c * -1.44269504088896341f);
    return y * __builtin_amdgcn_rcpf(1.0f + e);
}
__device__ __forceinline__ unsigned ld_agent(const unsigned* p) { return __hip_atomic_load(p, __ATOMIC_RELAXED, __HIP_MEMORY_SCOPE_AGENT); }
__device__ __forceinline__ unsigned ld_sys(const void* p) { return __hip_atomic_load((const unsigned*)p, __ATOMIC_RELAXED, __HIP_MEMORY_SCOPE_SYSTEM); }
__device__ __forceinline__ unsigned long long ld_sys64(const void* p) { return __hip_atomic_load((const unsigned long long*)p, __ATOMIC_RELAXED, __HIP_MEMORY_SCOPE_SYSTEM); }
__device__ __forceinline__ void st_sys(void* p, unsigned v) { __hip_atomic_store((unsigned*)p, v, __ATOMIC_RELAXED, __HIP_MEMORY_SCOPE_SYSTEM); }
__device__ __forceinline__ bool give_up(unsigned* tmo, unsigned call, unsigned long long t0) {
    if (__builtin_amdgcn_readfirstlane(ld_agent(tmo)) != call && __builtin_amdgcn_s_memrealtime() - t0 <= DEADLINE) return false;
    if ((threadIdx.x & 63) == 0) __hip_atomic_store(tmo, call, __ATOMIC_RELAXED, __HIP_MEMORY_SCOPE_AGENT);
    return true;
}
__device__ __forceinline__ int panel_of(int u, int d, int q) {
    if (u < 2) return 2 * d + u;
    const int hi = u >= 8, j = u - (hi ? 7 : 1);
    return 2 * ((d + j + (j >= q)) & 7) + hi;
}

#define DBG_STAMP(k) do { } while (0)
#define DBG_PROG(link, k) do { } while (0)

#define LEAD0 32
#define LEAD1 48
#define AVREMOTE 1
#define WARM 1
#define FAHEAD 3
#define PAUX 17
#define CPAUSE 0
#define PSLEEP 16
#define DS 2
#define DP 1
#define DD 3
#define OAUX 18
#define NTMASK 25
constexpr int A_OBL = 17 | ((NTMASK >> 0) & 1) * 2, A_XST = 17 | ((NTMASK >> 1) & 1) * 2, A_OBS = 17 | ((NTMASK >> 2) & 1) * 2, A_PL = 17 | ((NTMASK >> 3) & 1) * 2;
constexpr int A_DL = 17 | ((NTMASK >> 4) & 1) * 2, A_DS = 17 | ((NTMASK >> 5) & 1) * 2, A_PS = 17 | ((NTMASK >> 6) & 1) * 2, A_LS = 17 | ((NTMASK >> 7) & 1) * 2;
constexpr int SAUX = 17;
constexpr unsigned QNAN2 = 0x7fc07fc0u;

struct Half { unsigned isx, src, dst, flag; };
__device__ __forceinline__ Half half_of(const Params& P, unsigned s, int d, int q) {
    const unsigned e = (P.sched[s >> 3] >> (((s >> 1) & 3) * 8)) & 0xffu;
    const unsigned c = e & 7, idx = (e >> 3) & 15, h = s & 1;
    Half r; r.isx = e >> 7;
    if (r.isx) {
        const unsigned pn = panel_of(idx, d, q);
        r.src = (pn * 256 + h * 128) * XP + c * 128; r.dst = (unsigned)OFF_XS + q * GM * XP + r.src;
        r.flag = (unsigned)OFF_FX + ((pn * 64 + c * 8 + q) * 2 + h) * 8;
    } else {
        const unsigned o = c * 64 * WP + idx * 512 + h * 256;
        r.src = o + d * NB * 2; r.dst = (unsigned)OFF_WS + q * NB * 2 + o;
        r.flag = (unsigned)OFF_FW + ((idx * 64 + c * 8 + q) * 2 + h) * 8;
    }
    return r;
}
constexpr unsigned XB = 127 * XP + 128, WB = 63 * WP + 256;
__device__ __forceinline__ bool wait_ge(const char* word, unsigned want, unsigned* tmo, unsigned call) {
    const unsigned long long t0 = __builtin_amdgcn_s_memrealtime();
    for (unsigned polls = 1; __builtin_amdgcn_readfirstlane(ld_sys(word)) < want; ++polls) {
        if ((polls & 15u) == 0u && give_up(tmo, call, t0)) return false;
        __builtin_amdgcn_s_sleep(1);
    }
    return true;
}

__device__ __forceinline__ unsigned in_role(const Params& P, int cw, unsigned call) {
    const int lane = threadIdx.x & 63, hw = __builtin_amdgcn_readfirstlane(threadIdx.x >> 6);
    if (hw >= 7) return 0;
    const int wid = (hw + cw) % 7;
    const int g = P.rank;
    char* myws = P.ws[0];
    const __amdgpu_buffer_rsrc_t qctr = mk(myws + OFF_Q + (wid + 1) * 256, 4);
    const __amdgpu_buffer_rsrc_t rav = mk(myws + OFF_AV + (wid + 1) * 1024, 1024);
    const char* obox = P.ws[wid + 1] + OFF_OB + (size_t)(7 - wid) * 240 * 32768;
    const unsigned vx = (lane >> 3) * XP + (lane & 7) * 16, vw = (lane >> 4) * WP + (lane & 15) * 16, vl = lane * 16;
    unsigned* tmo = (unsigned*)(myws + OFF_TMO);
    const unsigned qbase = (call - 1u) * NIN - 2 * NCOMM;
    unsigned s = cw, pflag = 0, tk = qbase + NCOMM + cw;
    bool bad = false, pre = false;
    __amdgpu_buffer_rsrc_t rs = mk(myws, 0);
    u32x4 v[4];
    {
        const unsigned w0 = ld_sys(obox + (size_t)s * 8192), w1 = ld_sys(myws + half_of(P, s >> 1, g, wid + 1).dst);
        asm volatile("" :: "v"(w0), "v"(w1));
    }
    for (;;) {
        const Half hf = half_of(P, (s < NIN ? s : 0) >> 1, g, wid + 1);
        if (!pre) {
            if (s >= NIN) break;
            if (pflag) {
                asm volatile("s_waitcnt vmcnt(0)" ::: "memory");
                if (lane == 0) st_sys(myws + pflag, call);
                pflag = 0;
            }
            if (!wait_ge(P.ws[wid + 1] + OFF_AV2 + ((7 - wid) * 256 + (s >> 2)) * 128, call, tmo, call)) bad = true;
            rs = mk(obox + (size_t)s * 8192, bad ? 0 : 8192);
#pragma unroll
            for (int k = 0; k < DS; ++k) {
                v[k] = __builtin_amdgcn_raw_buffer_load_b128(rs, vl + k * 1024, 0, A_OBL); __builtin_amdgcn_sched_barrier(0);
                __builtin_amdgcn_raw_buffer_store_b128((u32x4){0u, 0u, 0u, 0u}, mk(myws, 0), vl, 0, SAUX); __builtin_amdgcn_sched_barrier(0);
            }
        }
        const unsigned ss = hf.isx ? 8 * XP : 4 * WP;
        const __amdgpu_buffer_rsrc_t rd = mk(myws + hf.dst + (s & 1) * 8 * ss, hf.isx ? 63 * XP + 128 : 31 * WP + 256);
        const unsigned sv = hf.isx ? vx : vw, flag = hf.flag + (s & 1) * 4;
        s = __builtin_amdgcn_readfirstlane(tk) - qbase;
        tk = (unsigned)__builtin_amdgcn_raw_ptr_buffer_atomic_add_i32(1, qctr, lane * 4, 0, 0);
        if (s >= NIN - LEAD0 && lane == 0)
            __hip_atomic_fetch_max((unsigned*)(myws + OFF_GO + (wid + 1) * 128), call * 1024u + (s - (NIN - LEAD0)), __ATOMIC_RELAXED, __HIP_MEMORY_SCOPE_AGENT);
        unsigned av = 0;
#define IN_ITEM(k) do { \
            if (k == 2) av = __builtin_amdgcn_raw_buffer_load_b32(rav, 0, (s >> 2) * 4 & 1023, SAUX); \
            if (k + DS == 8) {              \
                pre = s < NIN && (__builtin_amdgcn_readfirstlane(av) >= call || bad); \
                rs = mk(obox + (size_t)(s < NIN ? s : 0) * 8192, pre && !bad ? 8192 : 0); \
            } \
            asm volatile("" : "+v"(v[k & 3]));        \
            __builtin_amdgcn_sched_barrier(0); \
            if (k == DS && pflag) {         \
                asm volatile("s_waitcnt vmcnt(%0)" :: "i"(2 * DS) : "memory"); \
                if (lane == 0) st_sys(myws + pflag, call); \
                asm volatile("" ::: "memory"); \
            } \
            v[(k + DS) & 3] = __builtin_amdgcn_raw_buffer_load_b128(rs, vl, ((k + DS) & 7) * 1024, A_OBL); \
            __builtin_amdgcn_sched_barrier(0); \
            __builtin_amdgcn_raw_buffer_store_b128(v[k & 3], rd, sv, k * ss, A_XST); \
            __builtin_amdgcn_sched_barrier(0); \
            asm volatile("s_nop 1");        \
            __builtin_amdgcn_sched_barrier(0); } while (0)
        IN_ITEM(0); IN_ITEM(1); IN_ITEM(2); IN_ITEM(3); IN_ITEM(4); IN_ITEM(5); IN_ITEM(6); IN_ITEM(7);
#undef IN_ITEM
        pflag = flag;
    }
    DBG_STAMP(1);
    return pflag;
}

__device__ __forceinline__ void copy_role(const Params& P, int dd, unsigned n0, unsigned n1, bool push, unsigned cntv) {
    const int lane = threadIdx.x & 63, wv = __builtin_amdgcn_readfirstlane(threadIdx.x >> 6);
    const int g = P.rank, d = (g + dd) & 7;
    char* myws = P.ws[0]; char* dws = P.ws[dd];
    const unsigned gdv = ld_agent((const unsigned*)(myws + OFF_GD + dd * 128));
    const unsigned vx = (lane >> 3) * XP + (lane & 7) * 16, vw = (lane >> 4) * WP + (lane & 15) * 16;
    const char* const px = P.x; const char* const pwm = P.w;
    u32x4 a[4], b[4];
    Half ha, hb;
    auto fetch = [&](unsigned n, u32x4 (&r)[4], Half& hf) {
        hf = half_of(P, 2 * (n < n1 ? n : 0) + (wv >> 2), d, 8 - dd);
        const __amdgpu_buffer_rsrc_t rs = mk((hf.isx ? px : pwm) + hf.src, n < n1 ? (hf.isx ? XB : WB) : 0);
        const unsigned st = hf.isx ? 8 * XP : 4 * WP;
#pragma unroll
        for (int j = 0; j < 4; ++j) r[j] = __builtin_amdgcn_raw_buffer_load_b128(rs, hf.isx ? vx : vw, ((wv & 3) * 4 + j) * st, 0);
    };
    fetch(n0, a, ha);
    __builtin_amdgcn_sched_barrier(0);
    const unsigned call = __builtin_amdgcn_readfirstlane(cntv) + 1u;
    if (__builtin_amdgcn_readfirstlane(gdv) < call - 1u)
        wait_ge(myws + OFF_GD + dd * 128, call - 1u, (unsigned*)(myws + OFF_TMO), call);
    for (unsigned n = n0; n < n1; n += 16) {
        const unsigned st = push ? (ha.isx ? 8 * XP : 4 * WP) : 1024u;
        const __amdgpu_buffer_rsrc_t rd = push ? mk(dws + ha.dst, ha.isx ? XB : WB) : mk(myws + OFF_OB + (size_t)(dd * 240 + n) * 32768 + (wv >> 2) * 16384, 16384);
        const unsigned vo = (push ? (ha.isx ? vx : vw) : lane * 16u) + (wv & 3) * 4 * st;
#pragma unroll
        for (int j = 0; j < 4; ++j) __builtin_amdgcn_raw_buffer_store_b128(a[j], rd, vo + j * st, 0, A_OBS);
        __builtin_amdgcn_sched_barrier(0);
        fetch(n + 16, b, hb);
        asm volatile("s_waitcnt vmcnt(4)" ::: "memory");
        __builtin_amdgcn_s_barrier();
        if (push) { if ((threadIdx.x & 255) < 2) st_sys(dws + ha.flag + (threadIdx.x & 1) * 4, call); }
        else if (threadIdx.x < 2) st_sys(threadIdx.x ? dws + OFF_AV + ((8 - dd) * 256 + n) * 4 : myws + OFF_AV2 + (dd * 256 + n) * 128, call);
        if (n == n0 && !push) { for (int z = 0; z < CPAUSE; ++z) __builtin_amdgcn_s_sleep(16); }
#pragma unroll
        for (int j = 0; j < 4; ++j) a[j] = b[j];
        ha = hb;
    }
    DBG_STAMP(5);
}

#define PORDER 1
__device__ __forceinline__ unsigned ptile(unsigned o) { return PORDER ? (o & 7) | ((o & 8) << 1) | ((o & 16) >> 1) : o; }
__device__ __forceinline__ void pull_role(const Params& P, int pw, unsigned call, unsigned pend) {
    const int lane = threadIdx.x & 63, hw = __builtin_amdgcn_readfirstlane(threadIdx.x >> 6);
    if (hw >= 7) return;
    const int wid = (hw + pw) % 7;
    const int g = P.rank;
    char* myws = P.ws[0];
    const int d = (g + wid + 1) & 7;
    const __amdgpu_buffer_rsrc_t qctr = mk(myws + OFF_Q + (8 + wid + 1) * 256, 4);
    const unsigned qbase = (call - 1u) * NPULL - 2 * NPW;
    unsigned* tmo = (unsigned*)(myws + OFF_TMO);
    const char* fo = myws + OFF_FO + ((wid + 1) * 64 + lane) * 4;
    unsigned fv = ld_sys(fo), tk = qbase + NPW + pw, u = pw;
    asm volatile("s_waitcnt vmcnt(0)" ::: "memory");
    if (pend && lane == 0) st_sys(myws + pend, call);
    bool bad = false;
    if (pw < NPW0) {
        const unsigned want = call * 1024u + pw * LEAD1 / NPW0;
        const unsigned* go = (const unsigned*)(myws + OFF_GO + (wid + 1) * 128);
        const unsigned long long t0 = __builtin_amdgcn_s_memrealtime();
        for (unsigned polls = 1; __builtin_amdgcn_readfirstlane(ld_agent(go)) < want; ++polls) {
            if ((polls & 15u) == 0u && give_up(tmo, call, t0)) { bad = true; break; }
            __builtin_amdgcn_s_sleep(16);
        }
    }
    DBG_STAMP(5);
    const char* dps = P.ws[wid + 1] + OFF_PS + (call & 1) * PS_PAR + (size_t)(2 * g) * 16 * 131072;
    const char* dds = myws + OFF_DS + (size_t)(wid + 1) * 32 * 262144;
    char* ob = P.out + (size_t)d * NB * 2;
    const unsigned vl = lane * 16, vs = (lane & 15) * WP + (lane >> 4) * 16;
    const __amdgpu_buffer_rsrc_t rfo = mk(myws + OFF_FO + (wid + 1) * 256, 256);
    const unsigned long long m0 = __ballot(fv >= call);
    unsigned rdy = bad ? ~0u : (unsigned)m0 & (unsigned)(m0 >> 32); bool pre = false;
    u32x4 vp[4]; f32x4 vd[4][2];
    __amdgpu_buffer_rsrc_t rs = mk(myws, 0), rsd = mk(myws, 0);
#define LOAD_P(item) vp[item] = __builtin_amdgcn_raw_buffer_load_b128(rs, vl + (item) * 1024, 0, A_PL)
#define LOAD_D(item) do { vd[item][0] = __builtin_bit_cast(f32x4, __builtin_amdgcn_raw_buffer_load_b128(rsd, vl + (item) * 2048, 0, A_DL)); \
        vd[item][1] = __builtin_bit_cast(f32x4, __builtin_amdgcn_raw_buffer_load_b128(rsd, vl + (item) * 2048 + 1024, 0, A_DL)); } while (0)
    while (u < NPULL) {
        const unsigned tile = ptile(u >> 5), part = u & 31;
        if (!pre) {
            const unsigned long long t0 = __builtin_amdgcn_s_memrealtime();
            while (!((rdy >> tile) & 1u)) {
                const unsigned long long m = __ballot(ld_sys(fo) >= call);
                rdy = (unsigned)m & (unsigned)(m >> 32);
                if ((rdy >> tile) & 1u) break;
                if (give_up(tmo, call, t0)) { rdy = ~0u; bad = true; break; }
                __builtin_amdgcn_s_sleep(4);
            }
            rs = mk(dps + (size_t)(tile * 32 + part) * 4096, bad ? 0 : 4096); rsd = mk(dds + (size_t)(tile * 32 + part) * 8192, 8192);
#pragma unroll
            for (int k = 0; k < DD; ++k) {
                if (k >= DD - DP) LOAD_P(k - (DD - DP));
                LOAD_D(k);
                __builtin_amdgcn_sched_barrier(0);
                __builtin_amdgcn_raw_buffer_store_b128((u32x4){0u, 0u, 0u, 0u}, mk(myws, 0), vl, 0, OAUX);
                __builtin_amdgcn_sched_barrier(0);
            }
        }
        const unsigned un = __builtin_amdgcn_readfirstlane(tk) - qbase;
        {
            const unsigned long long m = __ballot(fv >= call);
            rdy |= (unsigned)m & (unsigned)(m >> 32);
        }
        tk = (unsigned)__builtin_amdgcn_raw_ptr_buffer_atomic_add_i32(1, qctr, lane * 4, 0, 0);
        {
            const unsigned oa = (un >> 5) + FAHEAD;
            fv = __builtin_amdgcn_raw_buffer_load_b32(rfo, ((rdy >> ptile(oa < 31 ? oa : 31)) & (rdy >> ptile(oa < 32 ? oa - 1 : 31)) & 1u) ? 256 : lane * 4, 0, SAUX);
        }
        const __amdgpu_buffer_rsrc_t rd = mk(ob + (size_t)((tile >> 4) * 256 + (part >> 1) * 16) * WP + (tile & 15) * 512 + (part & 1) * 256, 15 * WP + 256);
        const unsigned tn = ptile((un >> 5) & 31), on = tn * 32 + (un & 31);
        pre = un < NPULL && ((rdy >> tn) & 1u);
#pragma unroll
        for (int k = 0; k < 4; ++k) {
            if (k + DP == 4) rs = mk(dps + (size_t)on * 4096, pre && !bad ? 4096 : 0);
            if (k + DD == 4) rsd = mk(dds + (size_t)on * 8192, pre ? 8192 : 0);
            asm volatile("" : "+v"(vp[k]), "+v"(vd[k][0]), "+v"(vd[k][1]));
            __builtin_amdgcn_sched_barrier(0);
            LOAD_P((k + DP) & 3);
            LOAD_D((k + DD) & 3);
            __builtin_amdgcn_sched_barrier(0);
            const u32x4 p = vp[k]; const f32x4 d0 = vd[k][0], d1 = vd[k][1];
            u32x4 o;
            o.x = pk_bf16(silu(__uint_as_float(p.x << 16) + d0[0]), silu(__uint_as_float(p.x & 0xffff0000u) + d0[1]));
            o.y = pk_bf16(silu(__uint_as_float(p.y << 16) + d0[2]), silu(__uint_as_float(p.y & 0xffff0000u) + d0[3]));
            o.z = pk_bf16(silu(__uint_as_float(p.z << 16) + d1[0]), silu(__uint_as_float(p.z & 0xffff0000u) + d1[1]));
            o.w = pk_bf16(silu(__uint_as_float(p.w << 16) + d1[2]), silu(__uint_as_float(p.w & 0xffff0000u) + d1[3]));
            if (bad) o = (u32x4){QNAN2, QNAN2, QNAN2, QNAN2};
            __builtin_amdgcn_raw_buffer_store_b128(o, rd, vs + k * 64, 0, OAUX);
            __builtin_amdgcn_sched_barrier(0);
        }
        u = un;
    }
#undef LOAD_P
#undef LOAD_D
    DBG_STAMP(6);
}

__device__ __forceinline__ void gemm_tile(const Params& P, LAS unsigned char* lds, int mode, int pm, int pn, int qd, unsigned call) {
    const int tid = threadIdx.x, wid = __builtin_amdgcn_readfirstlane(tid >> 6), lane = tid & 63, wr = wid >> 2, wc = wid & 3, fr = lane & 15, fq = lane >> 4;
    const int g = P.rank;
    char* myws = P.ws[0];
    unsigned voffA[2], voffB[2];
#pragma unroll
    for (int i = 0; i < 2; ++i) {
        int R, C; stage_rc(tid * 16 + i * 8192, R, C); voffA[i] = (unsigned)R * XP + (unsigned)C * 2u;
        const int row = lane >> 3, ch = (lane & 7) ^ (((row >> 1) & 1) << 2);
        voffB[i] = (unsigned)(((wid >> 1) + 4 * i) * 8 + row) * WP + (unsigned)(wid & 1) * 128u + (unsigned)ch * 16u;
    }
    const unsigned ldsw = (unsigned)wid * 1024u, ldswb = (unsigned)((wid >> 2) * KBP + ((wid >> 1) & 1) * (2048 + 8 * BSHIFT) + (wid & 1) * 1024);
    const int aoff = lds_byte(wr * 64 + fr, fq * 8);
    const int boff = (fq >> 1) * KBP + (fq & 1) * (2048 + 8 * BSHIFT) + (wc >> 1) * 1024 + (fr >> 2) * 128 + ((4 * (wc & 1) + (fr & 3)) ^ (((fr >> 3) & 1) << 2)) * 16;
    constexpr size_t hA = (size_t)128 * XP, hB = 256;
    const int nt = mode == 0 ? 64 : (mode == 1 ? 56 : 8);
    const char* xl = P.x + (size_t)pm * 256 * XP;
    const char* wl = P.w + (size_t)(mode == 2 ? (g + qd) & 7 : g) * NB * 2 + pn * 512;
    const char* xs = myws + OFF_XS + (size_t)pm * 256 * XP;
    const char* wsb = myws + OFF_WS + pn * 512;
    auto ktile = [&](unsigned id, const char*& a, const char*& b) {
        const unsigned c = id >> 3, q = id & 7;
        a = (q ? xs : xl) + (q * (GM * XP) + c * 128); b = (q ? wsb : wl) + (c * (64 * WP) + q * (NB * 2));
    };
#define SA(b, h) (((b) * 2 + (h)) * HTB)
#define SB(b, h) (4 * HTB + ((b) * 2 + (h)) * HTBB)
#define STAGEB(bufoff, gbase) do { _Pragma("unroll") for (int _i = 0; _i < 2; ++_i) \
        __builtin_amdgcn_global_load_lds((const unsigned*)((const char*)(gbase) + voffB[_i]), (LAS unsigned*)(lds + (bufoff) + ldswb + _i * 2 * KBP), 16, 0, 0); } while (0)
#define STAGE(bufoff, gbase, voff) do { _Pragma("unroll") for (int _i = 0; _i < 2; ++_i) \
        __builtin_amdgcn_global_load_lds((const unsigned*)((const char*)(gbase) + (voff)[_i]), (LAS unsigned*)(lds + (bufoff) + ldsw + _i * 8192), 16, 0, 0); } while (0)
#define LDA(dst, b, h) do { _Pragma("unroll") for (int m = 0; m < 4; ++m) _Pragma("unroll") for (int k = 0; k < 2; ++k) dst[m][k] = *(const LAS bf16x8*)(lds + SA(b, h) + aoff + m * 2048 + k * 1024); } while (0)
#define TRD(dst, off) asm volatile("ds_read_b64_tr_b16 %0, %1 offset:%2" : "=v"(dst) : "v"(baddr), "i"(off) : "memory")
#define LDB(dst, b, h) do { _Pragma("unroll") for (int n = 0; n < 2; ++n) _Pragma("unroll") for (int k = 0; k < 2; ++k) { \
        TRD(dst[n][k][0], ((b) * 2 + (h)) * HTBB + k * 2 * KBP + n * 8); TRD(dst[n][k][1], ((b) * 2 + (h)) * HTBB + k * 2 * KBP + 512 + n * 8); } } while (0)
#define WAIT_B(Bt) asm volatile("s_waitcnt lgkmcnt(0)" : "+v"(Bt[0][0][0]), "+v"(Bt[0][0][1]), "+v"(Bt[0][1][0]), "+v"(Bt[0][1][1]), "+v"(Bt[1][0][0]), "+v"(Bt[1][0][1]), "+v"(Bt[1][1][0]), "+v"(Bt[1][1][1]) :: "memory")
#define BF(Bt, n, k) __builtin_shufflevector(Bt[n][k][0], Bt[n][k][1], 0, 1, 2, 3, 4, 5, 6, 7)
#define MMA(ai, bj, At, Bt) do { __builtin_amdgcn_s_setprio(1); _Pragma("unroll") for (int m = 0; m < 4; ++m) _Pragma("unroll") for (int n = 0; n < 2; ++n) _Pragma("unroll") for (int k = 0; k < 2; ++k) \
        acc[ai][bj][m][n] = __builtin_amdgcn_mfma_f32_16x16x32_bf16(BF(Bt, n, k), At[m][k], acc[ai][bj][m][n], 0, 0, 0); __builtin_amdgcn_s_setprio(0); } while (0)
#define WAIT_V(n) asm volatile("s_waitcnt vmcnt(" #n ")" ::: "memory")
#define WAIT_L(n) asm volatile("s_waitcnt lgkmcnt(" #n ")" ::: "memory")
#define BAR __builtin_amdgcn_s_barrier()
#define SCHED __builtin_amdgcn_sched_barrier(0)
    f32x4 acc[2][2][4][2];
#pragma unroll
    for (int a = 0; a < 2; ++a)
#pragma unroll
        for (int b = 0; b < 2; ++b)
#pragma unroll
            for (int m = 0; m < 4; ++m)
#pragma unroll
                for (int n = 0; n < 2; ++n) acc[a][b][m][n] = (f32x4){0.f, 0.f, 0.f, 0.f};
    bf16x8 At[4][2]; s16x4 B0[2][2][2], B1[2][2][2];
    const unsigned baddr = (unsigned)(size_t)lds + 4 * HTB + boff;
    constexpr unsigned long long LOCAL = 0x0101010101010101ull;
    unsigned long long rdy = LOCAL;
    unsigned long long todo = (mode == 0 ? ~0ull : (mode == 1 ? ~(LOCAL << qd) : LOCAL)) & ~0x101ull;
    LAS unsigned long long* rdy_lds = (LAS unsigned long long*)(lds + STAGE_BYTES);
    STAGEB(SB(0, 0), wl); STAGE(SA(0, 0), xl, voffA); STAGEB(SB(0, 1), wl + hB); STAGE(SA(0, 1), xl + hA, voffA);
    if (wr == 1) BAR;
    WAIT_V(4); BAR;
    STAGEB(SB(1, 0), wl + (size_t)64 * WP); STAGE(SA(1, 0), xl + 128, voffA); STAGEB(SB(1, 1), wl + (size_t)64 * WP + hB);
    WAIT_V(6); BAR;
    const char* a1 = xl + 128;
    for (int t = 0; t < nt; t += 2) {
        if (todo != 0ull && __builtin_popcountll(todo & rdy) < 2) {
            if (wid == 0) {
                const __amdgpu_buffer_rsrc_t rfx = mk(myws + OFF_FX + pm * 1024, 1024), rfw = mk(myws + OFF_FW + pn * 1024, 1024);
                const unsigned long long t0 = __builtin_amdgcn_s_memrealtime();
                unsigned long long mask = rdy;
                const unsigned long long pend = todo & ~rdy, pend2 = pend & (pend - 1ull);
                const int l1 = (__builtin_ctzll(pend2 ? pend2 : pend) | 7) + 8;
                unsigned vo = (pend >> lane) & 1ull ? lane * 16 : 4096;
                for (unsigned polls = 1;; ++polls) {
                    const u32x4 vx = __builtin_amdgcn_raw_buffer_load_b128(rfx, vo, 0, PAUX), vw = __builtin_amdgcn_raw_buffer_load_b128(rfw, vo, 0, PAUX);
                    asm volatile("" ::: "memory");
                    const bool ok = min(min(vx.x, vx.y), min(vx.z, vx.w)) >= call && min(min(vw.x, vw.y), min(vw.z, vw.w)) >= call;
                    mask |= __ballot(ok);
                    if (__builtin_popcountll(mask & todo) >= 2) break;
                    if ((polls & 7u) == 0u && give_up((unsigned*)(myws + OFF_TMO), call, t0)) { mask = ~0ull; break; }
                    if (lane > l1) vo = 4096;
                    __builtin_amdgcn_s_sleep(PSLEEP);
                }
                __builtin_amdgcn_fence(__ATOMIC_ACQUIRE, "");
                if (lane == 0) *rdy_lds = mask;
                asm volatile("s_waitcnt vmcnt(0) lgkmcnt(0)" ::: "memory");
            }
            asm volatile("" ::: "memory"); BAR; asm volatile("" ::: "memory");
            const unsigned long long mv = *rdy_lds;
            rdy = ((unsigned long long)(unsigned)__builtin_amdgcn_readfirstlane((unsigned)(mv >> 32)) << 32) | (unsigned)__builtin_amdgcn_readfirstlane((unsigned)mv);
        }
        LDB(B0, 0, 0); SCHED; LDA(At, 0, 0); STAGE(SA(1, 1), a1 + hA, voffA); SCHED;
        unsigned long long avail = todo ? todo & rdy : 0x101ull;
        const unsigned id2 = __builtin_ctzll(avail); avail &= avail - 1ull;
        const unsigned id3 = __builtin_ctzll(avail);
        todo &= ~((1ull << id2) | (1ull << id3));
        const char *a2, *b2, *a3, *b3;
        ktile(id2, a2, b2); ktile(id3, a3, b3);
        WAIT_L(8); BAR; WAIT_B(B0); MMA(0, 0, At, B0); BAR; SCHED;
        LDB(B1, 0, 1); STAGEB(SB(0, 0), b2);
        BAR; WAIT_B(B1); MMA(0, 1, At, B1); BAR;
        LDA(At, 0, 1); STAGE(SA(0, 0), a2, voffA);
        BAR; WAIT_L(0); MMA(1, 0, At, B0); BAR; SCHED;
        STAGEB(SB(0, 1), b2 + hB);
        WAIT_V(6); BAR; MMA(1, 1, At, B1); BAR;
        LDB(B0, 1, 0); SCHED; LDA(At, 1, 0); STAGE(SA(0, 1), a2 + hA, voffA);
        WAIT_L(8); BAR; WAIT_B(B0); MMA(0, 0, At, B0); BAR; SCHED;
        LDB(B1, 1, 1); STAGEB(SB(1, 0), b3);
        BAR; WAIT_B(B1); MMA(0, 1, At, B1); BAR;
        LDA(At, 1, 1); STAGE(SA(1, 0), a3, voffA);
        BAR; WAIT_L(0); MMA(1, 0, At, B0); BAR; SCHED;
        STAGEB(SB(1, 1), b3 + hB);
        WAIT_V(6); BAR; MMA(1, 1, At, B1); BAR;
        a1 = a3;
    }
    if (wr == 0) BAR;
    DBG_STAMP(mode == 2 ? 1 : 2);
    const bool bad = __builtin_amdgcn_readfirstlane(ld_agent((const unsigned*)(myws + OFF_TMO))) == call;
    int tz = threadIdx.x; asm volatile("" : "+v"(tz));
    const int l2 = tz & 63;
    if (mode == 2) {
        const __amdgpu_buffer_rsrc_t ro = mk(myws + OFF_DS + (size_t)(qd * 32 + (pm & 1) * 16 + pn) * 262144, 262144);
        const unsigned vo = (unsigned)(wr * 4 * 16384 + wc * 2048 + l2 * 16);
#pragma unroll
        for (int ai = 0; ai < 2; ++ai)
#pragma unroll
            for (int m = 0; m < 4; ++m)
#pragma unroll
                for (int bj = 0; bj < 2; ++bj)
#pragma unroll
                    for (int n = 0; n < 2; ++n)
                        __builtin_amdgcn_raw_buffer_store_b128(__builtin_bit_cast(u32x4, acc[ai][bj][m][n]), ro, vo + (unsigned)((ai * 8 + m) * 16384 + bj * 8192 + n * 1024), 0, A_DS);
    } else if (mode == 1) {
        const __amdgpu_buffer_rsrc_t ro = mk(myws + OFF_PS + (call & 1) * PS_PAR + (size_t)(pm * 16 + pn) * 131072, 131072);
        const unsigned vo = (unsigned)(wr * 4 * 8192 + wc * 1024 + l2 * 16);
#pragma unroll
        for (int ai = 0; ai < 2; ++ai)
#pragma unroll
            for (int m = 0; m < 4; ++m)
#pragma unroll
                for (int bj = 0; bj < 2; ++bj) {
                    const f32x4 v0 = acc[ai][bj][m][0], v1 = acc[ai][bj][m][1];
                    u32x4 o = {pk_bf16(v0[0], v0[1]), pk_bf16(v0[2], v0[3]), pk_bf16(v1[0], v1[1]), pk_bf16(v1[2], v1[3])};
                    if (bad) o = (u32x4){QNAN2, QNAN2, QNAN2, QNAN2};
                    __builtin_amdgcn_raw_buffer_store_b128(o, ro, vo + (unsigned)((ai * 8 + m) * 8192 + bj * 4096), 0, A_PS);
                }
    } else {
        const __amdgpu_buffer_rsrc_t ro = mk(P.out + (size_t)(pm & 1) * 256 * WP + (size_t)g * NB * 2 + pn * 512, 255 * WP + 512);
        const unsigned vo = (unsigned)(wr * 64 + (l2 & 15)) * WP + (unsigned)(wc * 32 + 8 * (l2 >> 4)) * 2u;
#pragma unroll
        for (int ai = 0; ai < 2; ++ai)
#pragma unroll
            for (int m = 0; m < 4; ++m)
#pragma unroll
                for (int bj = 0; bj < 2; ++bj) {
                    const f32x4 v0 = acc[ai][bj][m][0], v1 = acc[ai][bj][m][1];
                    u32x4 o = {pk_bf16(silu(v0[0]), silu(v0[1])), pk_bf16(silu(v0[2]), silu(v0[3])), pk_bf16(silu(v1[0]), silu(v1[1])), pk_bf16(silu(v1[2]), silu(v1[3]))};
                    if (bad) o = (u32x4){QNAN2, QNAN2, QNAN2, QNAN2};
                    __builtin_amdgcn_raw_buffer_store_b128(o, ro, vo + (unsigned)(ai * 8 + m) * 16 * WP + bj * 256, 0, A_LS);
                }
    }
    WAIT_V(0);
    BAR;
    if (tid == 0) {
        if (mode == 1) st_sys(P.ws[qd] + OFF_FO + ((8 - qd) * 64 + (pm & 1) * 16 + pn) * 4, call);
        if (mode == 2) st_sys(myws + OFF_FO + (qd * 64 + 32 + (pm & 1) * 16 + pn) * 4, call);
        else if (__hip_atomic_fetch_add((unsigned*)(myws + OFF_GDC), 1u, __ATOMIC_RELAXED, __HIP_MEMORY_SCOPE_AGENT) == 256u * call - 1u) {
            for (int o = 1; o < 8; ++o) st_sys(P.ws[o] + OFF_GD + (8 - o) * 128, call);
        }
    }
    DBG_STAMP(mode == 2 ? 4 : 3);
#undef SA
#undef SB
#undef STAGE
#undef STAGEB
#undef LDA
#undef LDB
#undef TRD
#undef WAIT_B
#undef BF
#undef MMA
}

__global__ __launch_bounds__(512, 2) void fused_gemm_rs(const Params P) {
    extern __shared__ __attribute__((aligned(16))) unsigned char shm[];
    const int b = blockIdx.x, xcd = b & 7, slot = b >> 3;
    asm volatile("" :: "s"(P.ws[0]), "s"(P.rank), "s"(P.sched[10]), "s"(P.sched[26]), "s"(P.sched[42]), "s"(P.sched[58]));
    unsigned* cnt = (unsigned*)(P.ws[0] + OFF_CNT + (size_t)b * 128);
    const unsigned cntv = ld_agent(cnt);
    DBG_STAMP(0);
    unsigned pend = 0;
    int pw = -1;
    if (slot < CPX) {
        pend = in_role(P, b, __builtin_amdgcn_readfirstlane(cntv) + 1u);
        pw = NPW0 + b;
    } else {
        const int g = P.rank, s = slot - CPX;
        const int cl = s / 7, pcl = PORDER ? 2 : 1;
        const bool late = cl != 0 && cl != pcl;
        const int e = ((late && cl == 3 ? 7 : 0) + s % 7) * 8 + xcd;
        if (cl != 0) copy_role(P, e % 7 + 1, (late ? 0 : NBLK) + e / 7, late ? NBLK : 240, !late, cntv);
        const unsigned call = __builtin_amdgcn_readfirstlane(cntv) + 1u;
        for (int u = (late ? 0 : 2); u < (s < 4 ? 4 : 3); ++u) {
            int mode, pm, pn, qd;
            if (u < 2) { mode = 2; pm = 2 * g + u; pn = e / 7; qd = e % 7 + 1; }
            else if (u == 2) { mode = 1; qd = s % 7 + 1; pm = 2 * ((g + qd) & 7) + (cl >> 1); pn = xcd + 8 * (cl & 1); }
            else { mode = 0; qd = 0; pm = 2 * g + (s >> 1); pn = xcd + 8 * (s & 1); }
            gemm_tile(P, (LAS unsigned char*)shm, mode, pm, pn, qd, call);
        }
        if (s >= 4 && s < 7) pw = (s - 4) * 8 + xcd;
    }
    if (pw >= 0) pull_role(P, pw, __builtin_amdgcn_readfirstlane(cntv) + 1u, pend);
    __syncthreads();
    if (threadIdx.x == 0) __hip_atomic_store(cnt, cntv + 1u, __ATOMIC_RELAXED, __HIP_MEMORY_SCOPE_AGENT);
}

#define SCHED_V 4
void build_schedule(unsigned* sched) {
    unsigned char e[240]; int n = 0;
    auto W = [&](int j0, int j1, int c) { for (int j = j0; j < j1; ++j) e[n++] = (unsigned char)((j << 3) | c); };
    auto X = [&](int u0, int u1, int c) { for (int u = u0; u < u1; ++u) e[n++] = (unsigned char)(128 | (u << 3) | c); };
    for (int c = 0; c < 8; ++c) { X(2, 8, c); W(0, 8, c); if (c == 4) X(8, 14, 0); if (c == 6) X(8, 14, 1); }
    for (int c = 0; c < 6; ++c) { X(8, 14, c + 2); W(8, 16, c); X(0, 2, c); }
    W(8, 16, 6); W(8, 16, 7); X(0, 2, 6); X(0, 2, 7);
    for (int i = 0; i < 60; ++i) sched[i] = e[4 * i] | ((unsigned)e[4 * i + 1] << 8) | ((unsigned)e[4 * i + 2] << 16) | ((unsigned)e[4 * i + 3] << 24);
}
}

extern "C" void kernel_launch(int rank, int nranks, ncclComm_t comm, hipStream_t stream,
                              void* const* d_in, const int* in_sizes, int n_in,
                              void* d_out, int out_size, const int* in_dtypes, int out_dtype,
                              void* const* d_out_all, void* const* d_ws_all, size_t ws_bytes) {
    if (nranks != NR || (unsigned)rank >= (unsigned)NR || n_in < 2 || ws_bytes < WS_NEED ||
        in_sizes[0] != GM * KS || in_sizes[1] != KS * GN || out_size != MS * GN) {
        fprintf(stderr, "kernel_launch: unexpected problem shape\n");
        return;
    }
    HIP_CHECK(hipFuncSetAttribute((const void*)fused_gemm_rs, hipFuncAttributeMaxDynamicSharedMemorySize, LDS_BYTES));
    Params P;
    P.x = (const char*)d_in[0]; P.w = (const char*)d_in[1];
    for (int p = 0; p < NR; ++p) P.ws[p] = (char*)d_ws_all[(rank + p) & 7];
    P.out = (char*)d_out;
    P.rank = rank;
    build_schedule(P.sched);
    hipLaunchKernelGGL(fused_gemm_rs, dim3(GRID), dim3(512), LDS_BYTES, stream, P);
    HIP_CHECK(hipGetLastError());
}
```

```cpp
#include <hip/hip_runtime.h>
#include <rccl/rccl.h>
#include <cstdio>

#define HIP_CHECK(expr) do { hipError_t _e = (expr); if (_e != hipSuccess) { \
    fprintf(stderr, "HIP error %s at %s:%d\n", hipGetErrorString(_e), __FILE__, __LINE__); \
    return; } } while (0)

#define LAS __attribute__((address_space(3)))
typedef short s16x4 __attribute__((ext_vector_type(4)));
typedef short bf16x8 __attribute__((ext_vector_type(8)));
typedef float f32x4 __attribute__((ext_vector_type(4)));
typedef float f32x2 __attribute__((ext_vector_type(2)));
typedef __bf16 bf16x2 __attribute__((ext_vector_type(2)));
typedef unsigned u32x4 __attribute__((ext_vector_type(4)));

namespace {
constexpr int NR = 8;
constexpr int GM = 4096, GK = 4096, GN = 32768;
constexpr int KS = GK / NR;
constexpr int NB = GN / NR;
constexpr int MS = GM / NR;
constexpr unsigned XP = KS * 2;
constexpr unsigned WP = GN * 2;
constexpr int HTB = 128 * 64 * 2;
#define BSHIFT 1
constexpr int KBP = 4096 + 16 * BSHIFT;
constexpr int HTBB = 4 * KBP;
constexpr int STAGE_BYTES = 4 * HTB + 4 * HTBB;
constexpr int LDS_BYTES = STAGE_BYTES + 64;
constexpr int GRID = 256;
constexpr int CPX = 4;
constexpr int NCOMM = 8 * CPX;
constexpr int NPW0 = 24;
constexpr int NPW = NPW0 + NCOMM;
constexpr unsigned long long DEADLINE = 50000000ull;

constexpr size_t OFF_CNT = 0;
constexpr size_t OFF_TMO = 32768;
constexpr size_t OFF_GDC = OFF_TMO + 256;
constexpr size_t OFF_FX = 65536;
constexpr size_t OFF_FW = OFF_FX + 16384;
constexpr size_t OFF_AV = OFF_FW + 16384;
constexpr size_t OFF_FO = OFF_AV + 8192;
constexpr size_t OFF_Q = OFF_FO + 2048;
constexpr size_t OFF_GD = OFF_Q + 4096;
constexpr size_t OFF_GO = OFF_GD + 1024;
constexpr size_t OFF_AV2 = 524288;
constexpr size_t OFF_AV3 = OFF_AV2 + 262144;
constexpr size_t OFF_DBG = 131072;
constexpr size_t OFF_XS = 1u << 20;
constexpr size_t OFF_WS = OFF_XS + (size_t)NR * GM * XP;
constexpr size_t OFF_PS = OFF_WS + (size_t)KS * WP;
constexpr size_t PS_PAR = (size_t)256 * 131072;
constexpr size_t OFF_DS = OFF_PS + 2 * PS_PAR;
constexpr size_t OFF_OB = OFF_DS + (size_t)256 * 262144;
constexpr size_t WS_NEED = OFF_OB + (size_t)8 * 240 * 32768;
#define NP0 2
constexpr unsigned NBLK = 240 - NP0;
constexpr unsigned NIN = 4 * NBLK;
constexpr unsigned NPULL = 1024;

struct Params {
    const char* x; const char* w;
    char* ws[NR];
    char* out;
    int rank;
    unsigned sched[60];
};

__device__ __forceinline__ int lds_byte(int r, int c) { const int st = (r >> 4) * 2 + (c >> 5), rr = r & 15, cc = c & 31, ob = rr * 64 + cc * 2; return st * 1024 + (ob ^ (((ob >> 9) & 1) << 5)); }
__device__ __forceinline__ void stage_rc(int b, int& R, int& C) { const int st = b / 1024, sb = b % 1024, swz = sb ^ (((sb >> 9) & 1) << 5); R = (st >> 1) * 16 + swz / 64; C = (st & 1) * 32 + (swz % 64) / 2; }
__device__ __forceinline__ __amdgpu_buffer_rsrc_t mk(const void* p, unsigned bytes) { return __builtin_amdgcn_make_buffer_rsrc((void*)p, 0, bytes, 0x00020000); }
__device__ __forceinline__ unsigned pk_bf16(float lo, float hi) { const bf16x2 r = __builtin_convertvector((f32x2){lo, hi}, bf16x2); return __builtin_bit_cast(unsigned, r); }
__device__ __forceinline__ float silu(float y) {
    const float c = __builtin_amdgcn_fmed3f(y, -60.0f, 60.0f);
    const float e = __builtin_amdgcn_exp2f(c * -1.44269504088896341f);
    return y * __builtin_amdgcn_rcpf(1.0f + e);
}
__device__ __forceinline__ unsigned ld_agent(const unsigned* p) { return __hip_atomic_load(p, __ATOMIC_RELAXED, __HIP_MEMORY_SCOPE_AGENT); }
__device__ __forceinline__ unsigned ld_sys(const void* p) { return __hip_atomic_load((const unsigned*)p, __ATOMIC_RELAXED, __HIP_MEMORY_SCOPE_SYSTEM); }
__device__ __forceinline__ unsigned long long ld_sys64(const void* p) { return __hip_atomic_load((const unsigned long long*)p, __ATOMIC_RELAXED, __HIP_MEMORY_SCOPE_SYSTEM); }
__device__ __forceinline__ void st_sys(void* p, unsigned v) { __hip_atomic_store((unsigned*)p, v, __ATOMIC_RELAXED, __HIP_MEMORY_SCOPE_SYSTEM); }
__device__ __forceinline__ bool give_up(unsigned* tmo, unsigned call, unsigned long long t0) {
    if (__builtin_amdgcn_readfirstlane(ld_agent(tmo)) != call && __builtin_amdgcn_s_memrealtime() - t0 <= DEADLINE) return false;
    if ((threadIdx.x & 63) == 0) __hip_atomic_store(tmo, call, __ATOMIC_RELAXED, __HIP_MEMORY_SCOPE_AGENT);
    return true;
}
__device__ __forceinline__ int panel_of(int u, int d, int q) {
    if (u < 2) return 2 * d + u;
    const int hi = u >= 8, j = u - (hi ? 7 : 1);
    return 2 * ((d + j + (j >= q)) & 7) + hi;
}

#define DBG_STAMP(k) do { } while (0)
#define DBG_PROG(link, k) do { } while (0)

#define LEAD0 32
#define LEAD1 48
#define AVREMOTE 1
#define WARM 1
#define FAHEAD 3
#define PAUX 17
#define CPAUSE 0
#define PACE 0
#define LASTALL 0
#define QUICK 1
#define NPA 4
#define AVK 4
#define Q3R 1
#define AVR 160
#define PSLEEP 16
#define DS 2
#define DP 1
#define DD 3
#define OAUX 18
#define NTMASK 25
constexpr int A_OBL = 17 | ((NTMASK >> 0) & 1) * 2, A_XST = 17 | ((NTMASK >> 1) & 1) * 2, A_OBS = 17 | ((NTMASK >> 2) & 1) * 2, A_PL = 17 | ((NTMASK >> 3) & 1) * 2;
constexpr int A_DL = 17 | ((NTMASK >> 4) & 1) * 2, A_DS = 17 | ((NTMASK >> 5) & 1) * 2, A_PS = 17 | ((NTMASK >> 6) & 1) * 2, A_LS = 17 | ((NTMASK >> 7) & 1) * 2;
constexpr int SAUX = 17;
constexpr unsigned QNAN2 = 0x7fc07fc0u;

struct Half { unsigned isx, src, dst, flag; };
__device__ __forceinline__ Half half_of(const Params& P, unsigned s, int d, int q) {
    const unsigned e = (P.sched[s >> 3] >> (((s >> 1) & 3) * 8)) & 0xffu;
    const unsigned c = e & 7, idx = (e >> 3) & 15, h = s & 1;
    Half r; r.isx = e >> 7;
    if (r.isx) {
        const unsigned pn = panel_of(idx, d, q);
        r.src = (pn * 256 + h * 128) * XP + c * 128; r.dst = (unsigned)OFF_XS + q * GM * XP + r.src;
        r.flag = (unsigned)OFF_FX + ((pn * 64 + c * 8 + q) * 2 + h) * 8;
    } else {
        const unsigned o = c * 64 * WP + idx * 512 + h * 256;
        r.src = o + d * NB * 2; r.dst = (unsigned)OFF_WS + q * NB * 2 + o;
        r.flag = (unsigned)OFF_FW + ((idx * 64 + c * 8 + q) * 2 + h) * 8;
    }
    return r;
}
constexpr unsigned XB = 127 * XP + 128, WB = 63 * WP + 256;
__device__ __forceinline__ bool wait_ge(const char* word, unsigned want, unsigned* tmo, unsigned call) {
    const unsigned long long t0 = __builtin_amdgcn_s_memrealtime();
    for (unsigned polls = 1; __builtin_amdgcn_readfirstlane(ld_sys(word)) < want; ++polls) {
        if ((polls & 15u) == 0u && give_up(tmo, call, t0)) return false;
        __builtin_amdgcn_s_sleep(1);
    }
    return true;
}

__device__ __forceinline__ unsigned in_role(const Params& P, int cw, unsigned call) {
    const int lane = threadIdx.x & 63, hw = __builtin_amdgcn_readfirstlane(threadIdx.x >> 6);
    if (hw >= 7) return 0;
    const int wid = (hw + cw) % 7;
    const int g = P.rank;
    char* myws = P.ws[0];
    const __amdgpu_buffer_rsrc_t qctr = mk(myws + OFF_Q + (wid + 1) * 256, 4);
    const __amdgpu_buffer_rsrc_t rav = mk(myws + OFF_AV + (wid + 1) * 1024, 1024);
    const __amdgpu_buffer_rsrc_t rav2 = mk(P.ws[wid + 1] + OFF_AV2 + (7 - wid) * 32768, 32768);
    const char* obox = P.ws[wid + 1] + OFF_OB + (size_t)(7 - wid) * 240 * 32768;
    const unsigned vx = (lane >> 3) * XP + (lane & 7) * 16, vw = (lane >> 4) * WP + (lane & 15) * 16, vl = lane * 16;
    unsigned* tmo = (unsigned*)(myws + OFF_TMO);
    const unsigned qbase = (call - 1u) * NIN - 2 * NCOMM;
    unsigned s = cw, pflag = 0, tk = qbase + NCOMM + cw;
    bool bad = false, pre = false;
    __amdgpu_buffer_rsrc_t rs = mk(myws, 0);
    u32x4 v[4];
    for (;;) {
        const Half hf = half_of(P, (s < NIN ? s : 0) >> 1, g, wid + 1);
        if (!pre) {
            if (s >= NIN) break;
            if (pflag) {
                asm volatile("s_waitcnt vmcnt(0)" ::: "memory");
                if (lane == 0) st_sys(myws + pflag, call);
                pflag = 0;
            }
            if (!wait_ge(QUICK && s < NCOMM ? (Q3R ? P.ws[wid + 1] + OFF_AV3 + ((7 - wid) * 32 + s) * 128 : myws + OFF_AV3 + ((wid + 1) * 32 + s) * 128) : P.ws[wid + 1] + OFF_AV2 + ((7 - wid) * 256 + (s >> 2)) * 128, call, tmo, call)) bad = true;
            rs = mk(obox + (size_t)s * 8192, bad ? 0 : 8192);
#pragma unroll
            for (int k = 0; k < DS; ++k) {
                v[k] = __builtin_amdgcn_raw_buffer_load_b128(rs, vl + k * 1024, 0, A_OBL); __builtin_amdgcn_sched_barrier(0);
                __builtin_amdgcn_raw_buffer_store_b128((u32x4){0u, 0u, 0u, 0u}, mk(myws, 0), vl, 0, SAUX); __builtin_amdgcn_sched_barrier(0);
            }
        }
        const unsigned ss = hf.isx ? 8 * XP : 4 * WP;
        const __amdgpu_buffer_rsrc_t rd = mk(myws + hf.dst + (s & 1) * 8 * ss, hf.isx ? 63 * XP + 128 : 31 * WP + 256);
        const unsigned sv = hf.isx ? vx : vw, flag = hf.flag + (s & 1) * 4;
        s = __builtin_amdgcn_readfirstlane(tk) - qbase;
        tk = (unsigned)__builtin_amdgcn_raw_ptr_buffer_atomic_add_i32(1, qctr, lane * 4, 0, 0);
        if (s >= NIN - LEAD0 && lane == 0)
            __hip_atomic_fetch_max((unsigned*)(myws + OFF_GO + (wid + 1) * 128), call * 1024u + (s - (NIN - LEAD0)), __ATOMIC_RELAXED, __HIP_MEMORY_SCOPE_AGENT);
        unsigned av = 0;
#define IN_ITEM(k) do { \
            if (k == AVK) av = s < AVR ? __builtin_amdgcn_raw_buffer_load_b32(rav2, 0, (s >> 2) * 128, SAUX) : __builtin_amdgcn_raw_buffer_load_b32(rav, 0, (s >> 2) * 4 & 1023, SAUX); \
            if (k + DS == 8) {              \
                pre = s < NIN && (__builtin_amdgcn_readfirstlane(av) >= call || bad); \
                rs = mk(obox + (size_t)(s < NIN ? s : 0) * 8192, pre && !bad ? 8192 : 0); \
            } \
            asm volatile("" : "+v"(v[k & 3]));        \
            __builtin_amdgcn_sched_barrier(0); \
            if (k == DS && pflag) {         \
                asm volatile("s_waitcnt vmcnt(%0)" :: "i"(2 * DS) : "memory"); \
                if (lane == 0) st_sys(myws + pflag, call); \
                asm volatile("" ::: "memory"); \
            } \
            v[(k + DS) & 3] = __builtin_amdgcn_raw_buffer_load_b128(rs, vl, ((k + DS) & 7) * 1024, A_OBL); \
            __builtin_amdgcn_sched_barrier(0); \
            __builtin_amdgcn_raw_buffer_store_b128(v[k & 3], rd, sv, k * ss, A_XST); \
            __builtin_amdgcn_sched_barrier(0); \
            asm volatile("s_nop 1");        \
            __builtin_amdgcn_sched_barrier(0); } while (0)
        IN_ITEM(0); IN_ITEM(1); IN_ITEM(2); IN_ITEM(3); IN_ITEM(4); IN_ITEM(5); IN_ITEM(6); IN_ITEM(7);
#undef IN_ITEM
        pflag = flag;
    }
    DBG_STAMP(1);
    return pflag;
}

__device__ __forceinline__ void copy_role(const Params& P, int dd, unsigned n0, unsigned n1, bool push, unsigned cntv) {
    const int lane = threadIdx.x & 63, wv = __builtin_amdgcn_readfirstlane(threadIdx.x >> 6);
    const int g = P.rank, d = (g + dd) & 7;
    char* myws = P.ws[0]; char* dws = P.ws[dd];
    const unsigned gdv = ld_agent((const unsigned*)(myws + OFF_GD + dd * 128));
    const unsigned vx = (lane >> 3) * XP + (lane & 7) * 16, vw = (lane >> 4) * WP + (lane & 15) * 16;
    const char* const px = P.x; const char* const pwm = P.w;
    u32x4 a[4], b[4];
    Half ha, hb;
    auto fetch = [&](unsigned n, u32x4 (&r)[4], Half& hf) {
        hf = half_of(P, 2 * (n < n1 ? n : 0) + (wv >> 2), d, 8 - dd);
        const __amdgpu_buffer_rsrc_t rs = mk((hf.isx ? px : pwm) + hf.src, n < n1 ? (hf.isx ? XB : WB) : 0);
        const unsigned st = hf.isx ? 8 * XP : 4 * WP;
#pragma unroll
        for (int j = 0; j < 4; ++j) r[j] = __builtin_amdgcn_raw_buffer_load_b128(rs, hf.isx ? vx : vw, ((wv & 3) * 4 + j) * st, 0);
    };
    fetch(n0, a, ha);
    __builtin_amdgcn_sched_barrier(0);
    const unsigned call = __builtin_amdgcn_readfirstlane(cntv) + 1u;
    if (__builtin_amdgcn_readfirstlane(gdv) < call - 1u)
        wait_ge(myws + OFF_GD + dd * 128, call - 1u, (unsigned*)(myws + OFF_TMO), call);
    for (unsigned n = n0; n < n1; n += 16) {
        const unsigned st = push ? (ha.isx ? 8 * XP : 4 * WP) : 1024u;
        const __amdgpu_buffer_rsrc_t rd = push ? mk(dws + ha.dst, ha.isx ? XB : WB) : mk(myws + OFF_OB + (size_t)(dd * 240 + n) * 32768 + (wv >> 2) * 16384, 16384);
        const unsigned vo = (push ? (ha.isx ? vx : vw) : lane * 16u) + (wv & 3) * 4 * st;
#pragma unroll
        for (int j = 0; j < 4; ++j) {
            __builtin_amdgcn_raw_buffer_store_b128(a[j], rd, vo + j * st, 0, A_OBS);
            if (PACE && push) asm volatile("s_waitcnt vmcnt(0)" ::: "memory");
        }
        __builtin_amdgcn_sched_barrier(0);
        fetch(n + 16, b, hb);
        asm volatile("s_waitcnt vmcnt(4)" ::: "memory");
        __builtin_amdgcn_s_barrier();
        if (push) { if ((threadIdx.x & 255) < 2) st_sys(dws + ha.flag + (threadIdx.x & 1) * 4, call); }
        else if (threadIdx.x < 2) st_sys(threadIdx.x ? dws + OFF_AV + ((8 - dd) * 256 + n) * 4 : myws + OFF_AV2 + (dd * 256 + n) * 128, call);
        if (n == n0 && !push) { for (int z = 0; z < CPAUSE; ++z) __builtin_amdgcn_s_sleep(16); }
#pragma unroll
        for (int j = 0; j < 4; ++j) a[j] = b[j];
        ha = hb;
    }
    DBG_STAMP(5);
}

__device__ __forceinline__ void quick_role(const Params& P, int dd, unsigned i, unsigned cntv) {
    const int lane = threadIdx.x & 63, wv = __builtin_amdgcn_readfirstlane(threadIdx.x >> 6);
    const int d = (P.rank + dd) & 7;
    char* myws = P.ws[0]; char* dws = P.ws[dd];
    const unsigned gdv = ld_agent((const unsigned*)(myws + OFF_GD + dd * 128));
    const unsigned vx = (lane >> 3) * XP + (lane & 7) * 16, vw = (lane >> 4) * WP + (lane & 15) * 16;
    const char* const px = P.x; const char* const pwm = P.w;
    const bool push = i < 4 * NP0;
    const Half h0 = half_of(P, i >> 1, d, 8 - dd), h1 = half_of(P, (push ? NIN + i : 0) >> 1, d, 8 - dd);
    const unsigned o0 = (h0.isx ? vx : vw) + ((i & 1) * 8 + wv) * (h0.isx ? 8 * XP : 4 * WP), o1 = (h1.isx ? vx : vw) + ((i & 1) * 8 + wv) * (h1.isx ? 8 * XP : 4 * WP);
    const u32x4 a = __builtin_amdgcn_raw_buffer_load_b128(mk((h0.isx ? px : pwm) + h0.src, h0.isx ? XB : WB), o0, 0, 0);
    const u32x4 b = __builtin_amdgcn_raw_buffer_load_b128(mk((h1.isx ? px : pwm) + h1.src, push ? (h1.isx ? XB : WB) : 0), o1, 0, 0);
    __builtin_amdgcn_sched_barrier(0);
    const unsigned call = __builtin_amdgcn_readfirstlane(cntv) + 1u;
    if (__builtin_amdgcn_readfirstlane(gdv) < call - 1u)
        wait_ge(myws + OFF_GD + dd * 128, call - 1u, (unsigned*)(myws + OFF_TMO), call);
    __builtin_amdgcn_raw_buffer_store_b128(a, mk(myws + OFF_OB + (size_t)dd * 240 * 32768 + i * 8192, 8192), wv * 1024 + lane * 16, 0, A_OBS);
    __builtin_amdgcn_sched_barrier(0);
    __builtin_amdgcn_raw_buffer_store_b128(b, mk(dws + h1.dst, i < NPA ? (h1.isx ? XB : WB) : 0), o1, 0, SAUX);
    asm volatile("s_waitcnt vmcnt(1)" ::: "memory");
    __builtin_amdgcn_s_barrier();
    if (threadIdx.x == 0) st_sys(Q3R ? myws + OFF_AV3 + (dd * 32 + i) * 128 : dws + OFF_AV3 + ((8 - dd) * 32 + i) * 128, call);
    if (push) {
        __builtin_amdgcn_raw_buffer_store_b128(b, mk(dws + h1.dst, i >= NPA ? (h1.isx ? XB : WB) : 0), o1, 0, SAUX);
        asm volatile("s_waitcnt vmcnt(0)" ::: "memory");
        __builtin_amdgcn_s_barrier();
        if (threadIdx.x == 0) st_sys(dws + h1.flag + (i & 1) * 4, call);
    }
    DBG_STAMP(6);
}

#define PORDER 1
__device__ __forceinline__ unsigned ptile(unsigned o) { return PORDER ? (o & 7) | ((o & 8) << 1) | ((o & 16) >> 1) : o; }
__device__ __forceinline__ void pull_role(const Params& P, int pw, unsigned call, unsigned pend) {
    const int lane = threadIdx.x & 63, hw = __builtin_amdgcn_readfirstlane(threadIdx.x >> 6);
    if (hw >= 7) return;
    const int wid = (hw + pw) % 7;
    const int g = P.rank;
    char* myws = P.ws[0];
    const int d = (g + wid + 1) & 7;
    const __amdgpu_buffer_rsrc_t qctr = mk(myws + OFF_Q + (8 + wid + 1) * 256, 4);
    const unsigned qbase = (call - 1u) * NPULL - 2 * NPW;
    unsigned* tmo = (unsigned*)(myws + OFF_TMO);
    const char* fo = myws + OFF_FO + ((wid + 1) * 64 + lane) * 4;
    unsigned fv = ld_sys(fo), tk = qbase + NPW + pw, u = pw;
    asm volatile("s_waitcnt vmcnt(0)" ::: "memory");
    if (pend && lane == 0) st_sys(myws + pend, call);
    bool bad = false;
    if (pw < NPW0) {
        const unsigned want = call * 1024u + pw * LEAD1 / NPW0;
        const unsigned* go = (const unsigned*)(myws + OFF_GO + (wid + 1) * 128);
        const unsigned long long t0 = __builtin_amdgcn_s_memrealtime();
        for (unsigned polls = 1; __builtin_amdgcn_readfirstlane(ld_agent(go)) < want; ++polls) {
            if ((polls & 15u) == 0u && give_up(tmo, call, t0)) { bad = true; break; }
            __builtin_amdgcn_s_sleep(16);
        }
    }
    DBG_STAMP(5);
    const char* dps = P.ws[wid + 1] + OFF_PS + (call & 1) * PS_PAR + (size_t)(2 * g) * 16 * 131072;
    const char* dds = myws + OFF_DS + (size_t)(wid + 1) * 32 * 262144;
    char* ob = P.out + (size_t)d * NB * 2;
    const unsigned vl = lane * 16, vs = (lane & 15) * WP + (lane >> 4) * 16;
    const __amdgpu_buffer_rsrc_t rfo = mk(myws + OFF_FO + (wid + 1) * 256, 256);
    const unsigned long long m0 = __ballot(fv >= call);
    unsigned rdy = bad ? ~0u : (unsigned)m0 & (unsigned)(m0 >> 32); bool pre = false;
    u32x4 vp[4]; f32x4 vd[4][2];
    __amdgpu_buffer_rsrc_t rs = mk(myws, 0), rsd = mk(myws, 0);
#define LOAD_P(item) vp[item] = __builtin_amdgcn_raw_buffer_load_b128(rs, vl + (item) * 1024, 0, A_PL)
#define LOAD_D(item) do { vd[item][0] = __builtin_bit_cast(f32x4, __builtin_amdgcn_raw_buffer_load_b128(rsd, vl + (item) * 2048, 0, A_DL)); \
        vd[item][1] = __builtin_bit_cast(f32x4, __builtin_amdgcn_raw_buffer_load_b128(rsd, vl + (item) * 2048 + 1024, 0, A_DL)); } while (0)
#define FINISH(k) do { const u32x4 p = vp[k]; const f32x4 d0 = vd[k][0], d1 = vd[k][1]; u32x4 o; \
            o.x = pk_bf16(silu(__uint_as_float(p.x << 16) + d0[0]), silu(__uint_as_float(p.x & 0xffff0000u) + d0[1])); \
            o.y = pk_bf16(silu(__uint_as_float(p.y << 16) + d0[2]), silu(__uint_as_float(p.y & 0xffff0000u) + d0[3])); \
            o.z = pk_bf16(silu(__uint_as_float(p.z << 16) + d1[0]), silu(__uint_as_float(p.z & 0xffff0000u) + d1[1])); \
            o.w = pk_bf16(silu(__uint_as_float(p.w << 16) + d1[2]), silu(__uint_as_float(p.w & 0xffff0000u) + d1[3])); \
            if (bad) o = (u32x4){QNAN2, QNAN2, QNAN2, QNAN2}; \
            __builtin_amdgcn_raw_buffer_store_b128(o, rd, vs + k * 64, 0, OAUX); } while (0)
    while (u < NPULL) {
        const unsigned tile = ptile(u >> 5), part = u & 31;
        if (!pre) {
            const unsigned long long t0 = __builtin_amdgcn_s_memrealtime();
            while (!((rdy >> tile) & 1u)) {
                const unsigned long long m = __ballot(ld_sys(fo) >= call);
                rdy = (unsigned)m & (unsigned)(m >> 32);
                if ((rdy >> tile) & 1u) break;
                if (give_up(tmo, call, t0)) { rdy = ~0u; bad = true; break; }
                __builtin_amdgcn_s_sleep(4);
            }
            rs = mk(dps + (size_t)(tile * 32 + part) * 4096, bad ? 0 : 4096); rsd = mk(dds + (size_t)(tile * 32 + part) * 8192, 8192);
#pragma unroll
            for (int k = 0; k < DD; ++k) {
                if (k >= DD - DP) LOAD_P(k - (DD - DP));
                LOAD_D(k);
                __builtin_amdgcn_sched_barrier(0);
                __builtin_amdgcn_raw_buffer_store_b128((u32x4){0u, 0u, 0u, 0u}, mk(myws, 0), vl, 0, OAUX);
                __builtin_amdgcn_sched_barrier(0);
            }
        }
        const unsigned un = __builtin_amdgcn_readfirstlane(tk) - qbase;
        {
            const unsigned long long m = __ballot(fv >= call);
            rdy |= (unsigned)m & (unsigned)(m >> 32);
        }
        tk = (unsigned)__builtin_amdgcn_raw_ptr_buffer_atomic_add_i32(1, qctr, lane * 4, 0, 0);
        {
            const unsigned oa = (un >> 5) + FAHEAD;
            fv = __builtin_amdgcn_raw_buffer_load_b32(rfo, ((rdy >> ptile(oa < 31 ? oa : 31)) & (rdy >> ptile(oa < 32 ? oa - 1 : 31)) & 1u) ? 256 : lane * 4, 0, SAUX);
        }
        const __amdgpu_buffer_rsrc_t rd = mk(ob + (size_t)((tile >> 4) * 256 + (part >> 1) * 16) * WP + (tile & 15) * 512 + (part & 1) * 256, 15 * WP + 256);
        const unsigned tn = ptile((un >> 5) & 31), on = tn * 32 + (un & 31);
        pre = un < NPULL && ((rdy >> tn) & 1u);
        if (LASTALL && un >= NPULL) {
#pragma unroll
            for (int k = DD; k < 4; ++k) LOAD_D(k);
#pragma unroll
            for (int k = DP; k < 4; ++k) LOAD_P(k);
            __builtin_amdgcn_sched_barrier(0);
#pragma unroll
            for (int k = 0; k < 4; ++k) { FINISH(k); }
            break;
        }
#pragma unroll
        for (int k = 0; k < 4; ++k) {
            if (k + DP == 4) rs = mk(dps + (size_t)on * 4096, pre && !bad ? 4096 : 0);
            if (k + DD == 4) rsd = mk(dds + (size_t)on * 8192, pre ? 8192 : 0);
            asm volatile("" : "+v"(vp[k]), "+v"(vd[k][0]), "+v"(vd[k][1]));
            __builtin_amdgcn_sched_barrier(0);
            LOAD_P((k + DP) & 3);
            LOAD_D((k + DD) & 3);
            __builtin_amdgcn_sched_barrier(0);
            FINISH(k);
            __builtin_amdgcn_sched_barrier(0);
        }
        u = un;
    }
#undef LOAD_P
#undef FINISH
#undef LOAD_D
    DBG_STAMP(6);
}

__device__ __forceinline__ void gemm_tile(const Params& P, LAS unsigned char* lds, int mode, int pm, int pn, int qd, unsigned call) {
    const int tid = threadIdx.x, wid = __builtin_amdgcn_readfirstlane(tid >> 6), lane = tid & 63, wr = wid >> 2, wc = wid & 3, fr = lane & 15, fq = lane >> 4;
    const int g = P.rank;
    char* myws = P.ws[0];
    unsigned voffA[2], voffB[2];
#pragma unroll
    for (int i = 0; i < 2; ++i) {
        int R, C; stage_rc(tid * 16 + i * 8192, R, C); voffA[i] = (unsigned)R * XP + (unsigned)C * 2u;
        const int row = lane >> 3, ch = (lane & 7) ^ (((row >> 1) & 1) << 2);
        voffB[i] = (unsigned)(((wid >> 1) + 4 * i) * 8 + row) * WP + (unsigned)(wid & 1) * 128u + (unsigned)ch * 16u;
    }
    const unsigned ldsw = (unsigned)wid * 1024u, ldswb = (unsigned)((wid >> 2) * KBP + ((wid >> 1) & 1) * (2048 + 8 * BSHIFT) + (wid & 1) * 1024);
    const int aoff = lds_byte(wr * 64 + fr, fq * 8);
    const int boff = (fq >> 1) * KBP + (fq & 1) * (2048 + 8 * BSHIFT) + (wc >> 1) * 1024 + (fr >> 2) * 128 + ((4 * (wc & 1) + (fr & 3)) ^ (((fr >> 3) & 1) << 2)) * 16;
    constexpr size_t hA = (size_t)128 * XP, hB = 256;
    const int nt = mode == 0 ? 64 : (mode == 1 ? 56 : 8);
    const char* xl = P.x + (size_t)pm * 256 * XP;
    const char* wl = P.w + (size_t)(mode == 2 ? (g + qd) & 7 : g) * NB * 2 + pn * 512;
    const char* xs = myws + OFF_XS + (size_t)pm * 256 * XP;
    const char* wsb = myws + OFF_WS + pn * 512;
    auto ktile = [&](unsigned id, const char*& a, const char*& b) {
        const unsigned c = id >> 3, q = id & 7;
        a = (q ? xs : xl) + (q * (GM * XP) + c * 128); b = (q ? wsb : wl) + (c * (64 * WP) + q * (NB * 2));
    };
#define SA(b, h) (((b) * 2 + (h)) * HTB)
#define SB(b, h) (4 * HTB + ((b) * 2 + (h)) * HTBB)
#define STAGEB(bufoff, gbase) do { _Pragma("unroll") for (int _i = 0; _i < 2; ++_i) \
        __builtin_amdgcn_global_load_lds((const unsigned*)((const char*)(gbase) + voffB[_i]), (LAS unsigned*)(lds + (bufoff) + ldswb + _i * 2 * KBP), 16, 0, 0); } while (0)
#define STAGE(bufoff, gbase, voff) do { _Pragma("unroll") for (int _i = 0; _i < 2; ++_i) \
        __builtin_amdgcn_global_load_lds((const unsigned*)((const char*)(gbase) + (voff)[_i]), (LAS unsigned*)(lds + (bufoff) + ldsw + _i * 8192), 16, 0, 0); } while (0)
#define LDA(dst, b, h) do { _Pragma("unroll") for (int m = 0; m < 4; ++m) _Pragma("unroll") for (int k = 0; k < 2; ++k) dst[m][k] = *(const LAS bf16x8*)(lds + SA(b, h) + aoff + m * 2048 + k * 1024); } while (0)
#define TRD(dst, off) asm volatile("ds_read_b64_tr_b16 %0, %1 offset:%2" : "=v"(dst) : "v"(baddr), "i"(off) : "memory")
#define LDB(dst, b, h) do { _Pragma("unroll") for (int n = 0; n < 2; ++n) _Pragma("unroll") for (int k = 0; k < 2; ++k) { \
        TRD(dst[n][k][0], ((b) * 2 + (h)) * HTBB + k * 2 * KBP + n * 8); TRD(dst[n][k][1], ((b) * 2 + (h)) * HTBB + k * 2 * KBP + 512 + n * 8); } } while (0)
#define WAIT_B(Bt) asm volatile("s_waitcnt lgkmcnt(0)" : "+v"(Bt[0][0][0]), "+v"(Bt[0][0][1]), "+v"(Bt[0][1][0]), "+v"(Bt[0][1][1]), "+v"(Bt[1][0][0]), "+v"(Bt[1][0][1]), "+v"(Bt[1][1][0]), "+v"(Bt[1][1][1]) :: "memory")
#define BF(Bt, n, k) __builtin_shufflevector(Bt[n][k][0], Bt[n][k][1], 0, 1, 2, 3, 4, 5, 6, 7)
#define MMA(ai, bj, At, Bt) do { __builtin_amdgcn_s_setprio(1); _Pragma("unroll") for (int m = 0; m < 4; ++m) _Pragma("unroll") for (int n = 0; n < 2; ++n) _Pragma("unroll") for (int k = 0; k < 2; ++k) \
        acc[ai][bj][m][n] = __builtin_amdgcn_mfma_f32_16x16x32_bf16(BF(Bt, n, k), At[m][k], acc[ai][bj][m][n], 0, 0, 0); __builtin_amdgcn_s_setprio(0); } while (0)
#define WAIT_V(n) asm volatile("s_waitcnt vmcnt(" #n ")" ::: "memory")
#define WAIT_L(n) asm volatile("s_waitcnt lgkmcnt(" #n ")" ::: "memory")
#define BAR __builtin_amdgcn_s_barrier()
#define SCHED __builtin_amdgcn_sched_barrier(0)
    f32x4 acc[2][2][4][2];
#pragma unroll
    for (int a = 0; a < 2; ++a)
#pragma unroll
        for (int b = 0; b < 2; ++b)
#pragma unroll
            for (int m = 0; m < 4; ++m)
#pragma unroll
                for (int n = 0; n < 2; ++n) acc[a][b][m][n] = (f32x4){0.f, 0.f, 0.f, 0.f};
    bf16x8 At[4][2]; s16x4 B0[2][2][2], B1[2][2][2];
    const unsigned baddr = (unsigned)(size_t)lds + 4 * HTB + boff;
    constexpr unsigned long long LOCAL = 0x0101010101010101ull;
    unsigned long long rdy = LOCAL;
    unsigned long long todo = (mode == 0 ? ~0ull : (mode == 1 ? ~(LOCAL << qd) : LOCAL)) & ~0x101ull;
    LAS unsigned long long* rdy_lds = (LAS unsigned long long*)(lds + STAGE_BYTES);
    STAGEB(SB(0, 0), wl); STAGE(SA(0, 0), xl, voffA); STAGEB(SB(0, 1), wl + hB); STAGE(SA(0, 1), xl + hA, voffA);
    if (wr == 1) BAR;
    WAIT_V(4); BAR;
    STAGEB(SB(1, 0), wl + (size_t)64 * WP); STAGE(SA(1, 0), xl + 128, voffA); STAGEB(SB(1, 1), wl + (size_t)64 * WP + hB);
    WAIT_V(6); BAR;
    const char* a1 = xl + 128;
    for (int t = 0; t < nt; t += 2) {
        if (todo != 0ull && __builtin_popcountll(todo & rdy) < 2) {
            if (wid == 0) {
                const __amdgpu_buffer_rsrc_t rfx = mk(myws + OFF_FX + pm * 1024, 1024), rfw = mk(myws + OFF_FW + pn * 1024, 1024);
                const unsigned long long t0 = __builtin_amdgcn_s_memrealtime();
                unsigned long long mask = rdy;
                const unsigned long long pend = todo & ~rdy, pend2 = pend & (pend - 1ull);
                const int l1 = (__builtin_ctzll(pend2 ? pend2 : pend) | 7) + 8;
                unsigned vo = (pend >> lane) & 1ull ? lane * 16 : 4096;
                for (unsigned polls = 1;; ++polls) {
                    const u32x4 vx = __builtin_amdgcn_raw_buffer_load_b128(rfx, vo, 0, PAUX), vw = __builtin_amdgcn_raw_buffer_load_b128(rfw, vo, 0, PAUX);
                    asm volatile("" ::: "memory");
                    const bool ok = min(min(vx.x, vx.y), min(vx.z, vx.w)) >= call && min(min(vw.x, vw.y), min(vw.z, vw.w)) >= call;
                    mask |= __ballot(ok);
                    if (__builtin_popcountll(mask & todo) >= 2) break;
                    if ((polls & 7u) == 0u && give_up((unsigned*)(myws + OFF_TMO), call, t0)) { mask = ~0ull; break; }
                    if (lane > l1) vo = 4096;
                    __builtin_amdgcn_s_sleep(PSLEEP);
                }
                __builtin_amdgcn_fence(__ATOMIC_ACQUIRE, "");
                if (lane == 0) *rdy_lds = mask;
                asm volatile("s_waitcnt vmcnt(0) lgkmcnt(0)" ::: "memory");
            }
            asm volatile("" ::: "memory"); BAR; asm volatile("" ::: "memory");
            const unsigned long long mv = *rdy_lds;
            rdy = ((unsigned long long)(unsigned)__builtin_amdgcn_readfirstlane((unsigned)(mv >> 32)) << 32) | (unsigned)__builtin_amdgcn_readfirstlane((unsigned)mv);
        }
        LDB(B0, 0, 0); SCHED; LDA(At, 0, 0); STAGE(SA(1, 1), a1 + hA, voffA); SCHED;
        unsigned long long avail = todo ? todo & rdy : 0x101ull;
        const unsigned id2 = __builtin_ctzll(avail); avail &= avail - 1ull;
        const unsigned id3 = __builtin_ctzll(avail);
        todo &= ~((1ull << id2) | (1ull << id3));
        const char *a2, *b2, *a3, *b3;
        ktile(id2, a2, b2); ktile(id3, a3, b3);
        WAIT_L(8); BAR; WAIT_B(B0); MMA(0, 0, At, B0); BAR; SCHED;
        LDB(B1, 0, 1); STAGEB(SB(0, 0), b2);
        BAR; WAIT_B(B1); MMA(0, 1, At, B1); BAR;
        LDA(At, 0, 1); STAGE(SA(0, 0), a2, voffA);
        BAR; WAIT_L(0); MMA(1, 0, At, B0); BAR; SCHED;
        STAGEB(SB(0, 1), b2 + hB);
        WAIT_V(6); BAR; MMA(1, 1, At, B1); BAR;
        LDB(B0, 1, 0); SCHED; LDA(At, 1, 0); STAGE(SA(0, 1), a2 + hA, voffA);
        WAIT_L(8); BAR; WAIT_B(B0); MMA(0, 0, At, B0); BAR; SCHED;
        LDB(B1, 1, 1); STAGEB(SB(1, 0), b3);
        BAR; WAIT_B(B1); MMA(0, 1, At, B1); BAR;
        LDA(At, 1, 1); STAGE(SA(1, 0), a3, voffA);
        BAR; WAIT_L(0); MMA(1, 0, At, B0); BAR; SCHED;
        STAGEB(SB(1, 1), b3 + hB);
        WAIT_V(6); BAR; MMA(1, 1, At, B1); BAR;
        a1 = a3;
    }
    if (wr == 0) BAR;
    DBG_STAMP(mode == 2 ? 1 : 2);
    const bool bad = __builtin_amdgcn_readfirstlane(ld_agent((const unsigned*)(myws + OFF_TMO))) == call;
    int tz = threadIdx.x; asm volatile("" : "+v"(tz));
    const int l2 = tz & 63;
    if (mode == 2) {
        const __amdgpu_buffer_rsrc_t ro = mk(myws + OFF_DS + (size_t)(qd * 32 + (pm & 1) * 16 + pn) * 262144, 262144);
        const unsigned vo = (unsigned)(wr * 4 * 16384 + wc * 2048 + l2 * 16);
#pragma unroll
        for (int ai = 0; ai < 2; ++ai)
#pragma unroll
            for (int m = 0; m < 4; ++m)
#pragma unroll
                for (int bj = 0; bj < 2; ++bj)
#pragma unroll
                    for (int n = 0; n < 2; ++n)
                        __builtin_amdgcn_raw_buffer_store_b128(__builtin_bit_cast(u32x4, acc[ai][bj][m][n]), ro, vo + (unsigned)((ai * 8 + m) * 16384 + bj * 8192 + n * 1024), 0, A_DS);
    } else if (mode == 1) {
        const __amdgpu_buffer_rsrc_t ro = mk(myws + OFF_PS + (call & 1) * PS_PAR + (size_t)(pm * 16 + pn) * 131072, 131072);
        const unsigned vo = (unsigned)(wr * 4 * 8192 + wc * 1024 + l2 * 16);
#pragma unroll
        for (int ai = 0; ai < 2; ++ai)
#pragma unroll
            for (int m = 0; m < 4; ++m)
#pragma unroll
                for (int bj = 0; bj < 2; ++bj) {
                    const f32x4 v0 = acc[ai][bj][m][0], v1 = acc[ai][bj][m][1];
                    u32x4 o = {pk_bf16(v0[0], v0[1]), pk_bf16(v0[2], v0[3]), pk_bf16(v1[0], v1[1]), pk_bf16(v1[2], v1[3])};
                    if (bad) o = (u32x4){QNAN2, QNAN2, QNAN2, QNAN2};
                    __builtin_amdgcn_raw_buffer_store_b128(o, ro, vo + (unsigned)((ai * 8 + m) * 8192 + bj * 4096), 0, A_PS);
                }
    } else {
        const __amdgpu_buffer_rsrc_t ro = mk(P.out + (size_t)(pm & 1) * 256 * WP + (size_t)g * NB * 2 + pn * 512, 255 * WP + 512);
        const unsigned vo = (unsigned)(wr * 64 + (l2 & 15)) * WP + (unsigned)(wc * 32 + 8 * (l2 >> 4)) * 2u;
#pragma unroll
        for (int ai = 0; ai < 2; ++ai)
#pragma unroll
            for (int m = 0; m < 4; ++m)
#pragma unroll
                for (int bj = 0; bj < 2; ++bj) {
                    const f32x4 v0 = acc[ai][bj][m][0], v1 = acc[ai][bj][m][1];
                    u32x4 o = {pk_bf16(silu(v0[0]), silu(v0[1])), pk_bf16(silu(v0[2]), silu(v0[3])), pk_bf16(silu(v1[0]), silu(v1[1])), pk_bf16(silu(v1[2]), silu(v1[3]))};
                    if (bad) o = (u32x4){QNAN2, QNAN2, QNAN2, QNAN2};
                    __builtin_amdgcn_raw_buffer_store_b128(o, ro, vo + (unsigned)(ai * 8 + m) * 16 * WP + bj * 256, 0, A_LS);
                }
    }
    WAIT_V(0);
    BAR;
    if (tid == 0) {
        if (mode == 1) st_sys(P.ws[qd] + OFF_FO + ((8 - qd) * 64 + (pm & 1) * 16 + pn) * 4, call);
        if (mode == 2) st_sys(myws + OFF_FO + (qd * 64 + 32 + (pm & 1) * 16 + pn) * 4, call);
        else if (__hip_atomic_fetch_add((unsigned*)(myws + OFF_GDC), 1u, __ATOMIC_RELAXED, __HIP_MEMORY_SCOPE_AGENT) == 256u * call - 1u) {
            for (int o = 1; o < 8; ++o) st_sys(P.ws[o] + OFF_GD + (8 - o) * 128, call);
        }
    }
    DBG_STAMP(mode == 2 ? 4 : 3);
#undef SA
#undef SB
#undef STAGE
#undef STAGEB
#undef LDA
#undef LDB
#undef TRD
#undef WAIT_B
#undef BF
#undef MMA
}

__global__ __launch_bounds__(512, 2) void fused_gemm_rs(const Params P) {
    extern __shared__ __attribute__((aligned(16))) unsigned char shm[];
    const int b = blockIdx.x, xcd = b & 7, slot = b >> 3;
    asm volatile("" :: "s"(P.ws[0]), "s"(P.rank), "s"(P.sched[10]), "s"(P.sched[26]), "s"(P.sched[42]), "s"(P.sched[58]));
    unsigned* cnt = (unsigned*)(P.ws[0] + OFF_CNT + (size_t)b * 128);
    const unsigned cntv = ld_agent(cnt);
    DBG_STAMP(0);
    unsigned pend = 0;
    int pw = -1;
    if (slot < CPX) {
        pend = in_role(P, b, __builtin_amdgcn_readfirstlane(cntv) + 1u);
        pw = NPW0 + b;
    } else {
        const int g = P.rank, s = slot - CPX;
        const int cl = s / 7, pcl = PORDER ? 2 : 1;
        const bool late = cl != 0 && cl != pcl;
        const int e = ((late && cl == 3 ? 7 : 0) + s % 7) * 8 + xcd;
        static_assert(NP0 <= 4, "pushed by the two early classes");
        { const int e2 = (late ? 112 : (cl == 0 ? 56 : 0)) + e; quick_role(P, e2 % 7 + 1, e2 / 7, cntv); }
        if (late) copy_role(P, e % 7 + 1, NCOMM / 4 + e / 7, NBLK, false, cntv);
        const unsigned call = __builtin_amdgcn_readfirstlane(cntv) + 1u;
        for (int u = (late ? 0 : 2); u < (s < 4 ? 4 : 3); ++u) {
            int mode, pm, pn, qd;
            if (u < 2) { mode = 2; pm = 2 * g + u; pn = e / 7; qd = e % 7 + 1; }
            else if (u == 2) { mode = 1; qd = s % 7 + 1; pm = 2 * ((g + qd) & 7) + (cl >> 1); pn = xcd + 8 * (cl & 1); }
            else { mode = 0; qd = 0; pm = 2 * g + (s >> 1); pn = xcd + 8 * (s & 1); }
            gemm_tile(P, (LAS unsigned char*)shm, mode, pm, pn, qd, call);
        }
        if (s >= 4 && s < 7) pw = (s - 4) * 8 + xcd;
    }
    if (pw >= 0) pull_role(P, pw, __builtin_amdgcn_readfirstlane(cntv) + 1u, pend);
    __syncthreads();
    if (threadIdx.x == 0) __hip_atomic_store(cnt, cntv + 1u, __ATOMIC_RELAXED, __HIP_MEMORY_SCOPE_AGENT);
}

#define SCHED_V 4
void build_schedule(unsigned* sched) {
    unsigned char e[240]; int n = 0;
    auto W = [&](int j0, int j1, int c) { for (int j = j0; j < j1; ++j) e[n++] = (unsigned char)((j << 3) | c); };
    auto X = [&](int u0, int u1, int c) { for (int u = u0; u < u1; ++u) e[n++] = (unsigned char)(128 | (u << 3) | c); };
    for (int c = 0; c < 8; ++c) { X(2, 8, c); W(0, 8, c); if (c == 4) X(8, 14, 0); if (c == 6) X(8, 14, 1); }
    for (int c = 0; c < 6; ++c) { X(8, 14, c + 2); W(8, 16, c); X(0, 2, c); }
    W(8, 16, 6); W(8, 16, 7); X(0, 2, 6); X(0, 2, 7);
    for (int i = 0; i < 60; ++i) sched[i] = e[4 * i] | ((unsigned)e[4 * i + 1] << 8) | ((unsigned)e[4 * i + 2] << 16) | ((unsigned)e[4 * i + 3] << 24);
}
}

extern "C" void kernel_launch(int rank, int nranks, ncclComm_t comm, hipStream_t stream,
                              void* const* d_in, const int* in_sizes, int n_in,
                              void* d_out, int out_size, const int* in_dtypes, int out_dtype,
                              void* const* d_out_all, void* const* d_ws_all, size_t ws_bytes) {
    if (nranks != NR || (unsigned)rank >= (unsigned)NR || n_in < 2 || ws_bytes < WS_NEED ||
        in_sizes[0] != GM * KS || in_sizes[1] != KS * GN || out_size != MS * GN) {
        fprintf(stderr, "kernel_launch: unexpected problem shape\n");
        return;
    }
    HIP_CHECK(hipFuncSetAttribute((const void*)fused_gemm_rs, hipFuncAttributeMaxDynamicSharedMemorySize, LDS_BYTES));
    Params P;
    P.x = (const char*)d_in[0]; P.w = (const char*)d_in[1];
    for (int p = 0; p < NR; ++p) P.ws[p] = (char*)d_ws_all[(rank + p) & 7];
    P.out = (char*)d_out;
    P.rank = rank;
    build_schedule(P.sched);
    hipLaunchKernelGGL(fused_gemm_rs, dim3(GRID), dim3(512), LDS_BYTES, stream, P);
    HIP_CHECK(hipGetLastError());
}
```

```cpp
#include <hip/hip_runtime.h>
#include <rccl/rccl.h>
#include <cstdio>

#define HIP_CHECK(expr) do { hipError_t _e = (expr); if (_e != hipSuccess) { \
    fprintf(stderr, "HIP error %s at %s:%d\n", hipGetErrorString(_e), __FILE__, __LINE__); \
    return; } } while (0)

#define LAS __attribute__((address_space(3)))
typedef short s16x4 __attribute__((ext_vector_type(4)));
typedef short bf16x8 __attribute__((ext_vector_type(8)));
typedef float f32x4 __attribute__((ext_vector_type(4)));
typedef float f32x2 __attribute__((ext_vector_type(2)));
typedef __bf16 bf16x2 __attribute__((ext_vector_type(2)));
typedef unsigned u32x4 __attribute__((ext_vector_type(4)));

namespace {
constexpr int NR = 8;
constexpr int GM = 4096, GK = 4096, GN = 32768;
constexpr int KS = GK / NR;
constexpr int NB = GN / NR;
constexpr int MS = GM / NR;
constexpr unsigned XP = KS * 2;
constexpr unsigned WP = GN * 2;
constexpr int HTB = 128 * 64 * 2;
#define BSHIFT 1
constexpr int KBP = 4096 + 16 * BSHIFT;
constexpr int HTBB = 4 * KBP;
constexpr int STAGE_BYTES = 4 * HTB + 4 * HTBB;
constexpr int LDS_BYTES = STAGE_BYTES + 64;
constexpr int GRID = 256;
constexpr int CPX = 4;
constexpr int NCOMM = 8 * CPX;
constexpr int NPW0 = 24;
#define XPW 1
constexpr int NPW = NPW0 + NCOMM + 8 * XPW;
constexpr unsigned long long DEADLINE = 50000000ull;

constexpr size_t OFF_CNT = 0;
constexpr size_t OFF_TMO = 32768;
constexpr size_t OFF_GDC = OFF_TMO + 256;
constexpr size_t OFF_FX = 65536;
constexpr size_t OFF_FW = OFF_FX + 16384;
constexpr size_t OFF_AV = OFF_FW + 16384;
constexpr size_t OFF_FO = OFF_AV + 8192;
constexpr size_t OFF_Q = OFF_FO + 2048;
constexpr size_t OFF_GD = OFF_Q + 4096;
constexpr size_t OFF_GO = OFF_GD + 1024;
constexpr size_t OFF_AV2 = 524288;
constexpr size_t OFF_AV3 = OFF_AV2 + 262144;
constexpr size_t OFF_DBG = 131072;
constexpr size_t OFF_XS = 1u << 20;
constexpr size_t OFF_WS = OFF_XS + (size_t)NR * GM * XP;
constexpr size_t OFF_PS = OFF_WS + (size_t)KS * WP;
constexpr size_t PS_PAR = (size_t)256 * 131072;
constexpr size_t OFF_DS = OFF_PS + 2 * PS_PAR;
constexpr size_t OFF_OB = OFF_DS + (size_t)256 * 262144;
constexpr size_t WS_NEED = OFF_OB + (size_t)8 * 240 * 32768;
#define NP0 2
#define NPL 6
constexpr unsigned NPOOL = 7 * 4 * NPL, POOL_STEP = NPOOL + 7 * NCOMM;
constexpr unsigned NBLK = 240 - NP0 - NPL;
constexpr unsigned NIN = 4 * NBLK;
constexpr unsigned NPULL = 1024;

struct Params {
    const char* x; const char* w;
    char* ws[NR];
    char* out;
    int rank;
    unsigned sched[60];
};

__device__ __forceinline__ int lds_byte(int r, int c) { const int st = (r >> 4) * 2 + (c >> 5), rr = r & 15, cc = c & 31, ob = rr * 64 + cc * 2; return st * 1024 + (ob ^ (((ob >> 9) & 1) << 5)); }
__device__ __forceinline__ void stage_rc(int b, int& R, int& C) { const int st = b / 1024, sb = b % 1024, swz = sb ^ (((sb >> 9) & 1) << 5); R = (st >> 1) * 16 + swz / 64; C = (st & 1) * 32 + (swz % 64) / 2; }
__device__ __forceinline__ __amdgpu_buffer_rsrc_t mk(const void* p, unsigned bytes) { return __builtin_amdgcn_make_buffer_rsrc((void*)p, 0, bytes, 0x00020000); }
__device__ __forceinline__ unsigned pk_bf16(float lo, float hi) { const bf16x2 r = __builtin_convertvector((f32x2){lo, hi}, bf16x2); return __builtin_bit_cast(unsigned, r); }
__device__ __forceinline__ float silu(float y) {
    const float c = __builtin_amdgcn_fmed3f(y, -60.0f, 60.0f);
    const float e = __builtin_amdgcn_exp2f(c * -1.44269504088896341f);
    return y * __builtin_amdgcn_rcpf(1.0f + e);
}
__device__ __forceinline__ unsigned ld_agent(const unsigned* p) { return __hip_atomic_load(p, __ATOMIC_RELAXED, __HIP_MEMORY_SCOPE_AGENT); }
__device__ __forceinline__ unsigned ld_sys(const void* p) { return __hip_atomic_load((const unsigned*)p, __ATOMIC_RELAXED, __HIP_MEMORY_SCOPE_SYSTEM); }
__device__ __forceinline__ unsigned long long ld_sys64(const void* p) { return __hip_atomic_load((const unsigned long long*)p, __ATOMIC_RELAXED, __HIP_MEMORY_SCOPE_SYSTEM); }
__device__ __forceinline__ void st_sys(void* p, unsigned v) { __hip_atomic_store((unsigned*)p, v, __ATOMIC_RELAXED, __HIP_MEMORY_SCOPE_SYSTEM); }
__device__ __forceinline__ bool give_up(unsigned* tmo, unsigned call, unsigned long long t0) {
    if (__builtin_amdgcn_readfirstlane(ld_agent(tmo)) != call && __builtin_amdgcn_s_memrealtime() - t0 <= DEADLINE) return false;
    if ((threadIdx.x & 63) == 0) __hip_atomic_store(tmo, call, __ATOMIC_RELAXED, __HIP_MEMORY_SCOPE_AGENT);
    return true;
}
__device__ __forceinline__ int panel_of(int u, int d, int q) {
    if (u < 2) return 2 * d + u;
    const int hi = u >= 8, j = u - (hi ? 7 : 1);
    return 2 * ((d + j + (j >= q)) & 7) + hi;
}

#define DBG_STAMP(k) do { } while (0)
#define DBG_PROG(link, k) do { } while (0)

#define LEAD0 0
#define LEAD1 31
#define AVREMOTE 1
#define WARM 1
#define FAHEAD 3
#define PAUX 17
#define CPAUSE 0
#define PACE 0
#define LASTALL 0
#define QUICK 1
#define NPA 4
#define AVK 5
#define Q3R 1
#define AVR 160
#define PSLEEP 16
#define DS 2
#define DP 1
#define DD 3
#define OAUX 18
#define NTMASK 25
constexpr int A_OBL = 17 | ((NTMASK >> 0) & 1) * 2, A_XST = 17 | ((NTMASK >> 1) & 1) * 2, A_OBS = 17 | ((NTMASK >> 2) & 1) * 2, A_PL = 17 | ((NTMASK >> 3) & 1) * 2;
constexpr int A_DL = 17 | ((NTMASK >> 4) & 1) * 2, A_DS = 17 | ((NTMASK >> 5) & 1) * 2, A_PS = 17 | ((NTMASK >> 6) & 1) * 2, A_LS = 17 | ((NTMASK >> 7) & 1) * 2;
constexpr int SAUX = 17;
constexpr unsigned QNAN2 = 0x7fc07fc0u;

struct Half { unsigned isx, src, dst, flag; };
__device__ __forceinline__ Half half_of(const Params& P, unsigned s, int d, int q) {
    const unsigned e = (P.sched[s >> 3] >> (((s >> 1) & 3) * 8)) & 0xffu;
    const unsigned c = e & 7, idx = (e >> 3) & 15, h = s & 1;
    Half r; r.isx = e >> 7;
    if (r.isx) {
        const unsigned pn = panel_of(idx, d, q);
        r.src = (pn * 256 + h * 128) * XP + c * 128; r.dst = (unsigned)OFF_XS + q * GM * XP + r.src;
        r.flag = (unsigned)OFF_FX + ((pn * 64 + c * 8 + q) * 2 + h) * 8;
    } else {
        const unsigned o = c * 64 * WP + idx * 512 + h * 256;
        r.src = o + d * NB * 2; r.dst = (unsigned)OFF_WS + q * NB * 2 + o;
        r.flag = (unsigned)OFF_FW + ((idx * 64 + c * 8 + q) * 2 + h) * 8;
    }
    return r;
}
constexpr unsigned XB = 127 * XP + 128, WB = 63 * WP + 256;
__device__ __forceinline__ bool wait_ge(const char* word, unsigned want, unsigned* tmo, unsigned call) {
    const unsigned long long t0 = __builtin_amdgcn_s_memrealtime();
    for (unsigned polls = 1; __builtin_amdgcn_readfirstlane(ld_sys(word)) < want; ++polls) {
        if ((polls & 15u) == 0u && give_up(tmo, call, t0)) return false;
        __builtin_amdgcn_s_sleep(1);
    }
    return true;
}

__device__ __forceinline__ unsigned in_role(const Params& P, int cw, unsigned call) {
    const int lane = threadIdx.x & 63, hw = __builtin_amdgcn_readfirstlane(threadIdx.x >> 6);
    if (hw >= 7) return 0;
    const int wid = (hw + cw) % 7;
    const int g = P.rank;
    char* myws = P.ws[0];
    const __amdgpu_buffer_rsrc_t qctr = mk(myws + OFF_Q + (wid + 1) * 256, 4);
    const unsigned vx = (lane >> 3) * XP + (lane & 7) * 16, vw = (lane >> 4) * WP + (lane & 15) * 16, vl = lane * 16;
    unsigned* tmo = (unsigned*)(myws + OFF_TMO);
    const unsigned qbase = (call - 1u) * NIN - 2 * NCOMM, pbase = (call - 1u) * POOL_STEP;
    const __amdgpu_buffer_rsrc_t pctr = mk(myws + OFF_Q, 4);
    const char* ews = P.ws[wid + 1];
    unsigned s = cw, pflag = 0, tk = qbase + NCOMM + cw;
    bool bad = false, pre = false;
    {
        const char* gd = myws + OFF_GD + ((lane & 7) ? lane & 7 : 1) * 128;
        if (!__all(ld_agent((const unsigned*)gd) >= call - 1u)) {
            const unsigned long long t0 = __builtin_amdgcn_s_memrealtime();
            while (!__all(ld_sys(gd) >= call - 1u)) { if (give_up(tmo, call, t0)) { bad = true; break; } __builtin_amdgcn_s_sleep(8); }
        }
    }
    __amdgpu_buffer_rsrc_t rs = mk(myws, 0);
    unsigned lv = vl, ls = 1024;
    u32x4 v[4];
    auto locate = [&](unsigned s, Half& hf, unsigned& sdat, unsigned& sflag) {
        const bool pool = s >= NIN;
        const unsigned t = pool && s != ~0u ? s - NIN : 0u, j = t / 7u, o = t - j * 7u + 1u, qe = (o - (wid + 1)) & 7u;
        const unsigned sq = pool ? NIN + j : s;
        hf = half_of(P, sq >> 1, g, pool ? o : wid + 1);
        const unsigned ss = hf.isx ? 8 * XP : 4 * WP;
        hf.dst += (sq & 1) * 8 * ss; hf.flag += (sq & 1) * 4;
        sdat = pool ? (unsigned)OFF_XS + qe * (GM * XP) + hf.src + (sq & 1) * 8 * ss : (unsigned)OFF_OB + ((7 - wid) * 960 + s) * 8192;
        sflag = pool ? hf.flag + qe * 16 - o * 16 : (s < NCOMM ? (unsigned)OFF_AV3 + ((7 - wid) * 32 + s) * 128 : (unsigned)OFF_AV2 + ((7 - wid) * 256 + (s >> 2)) * 128);
        asm volatile("" : "+s"(hf.dst), "+s"(hf.flag), "+s"(hf.isx), "+s"(sdat), "+s"(sflag));
    };
    Half hf; unsigned sdat, sflag;
    locate(s, hf, sdat, sflag);
    for (;;) {
        hf.dst = __builtin_amdgcn_readfirstlane(hf.dst); hf.flag = __builtin_amdgcn_readfirstlane(hf.flag); hf.isx = __builtin_amdgcn_readfirstlane(hf.isx);
        sdat = __builtin_amdgcn_readfirstlane(sdat); sflag = __builtin_amdgcn_readfirstlane(sflag);
        if (!pre) {
            if (s == ~0u) break;
            if (pflag) {
                asm volatile("s_waitcnt vmcnt(0)" ::: "memory");
                if (lane == 0) st_sys(myws + pflag, call);
                pflag = 0;
            }
            if (!wait_ge(ews + sflag, call, tmo, call)) bad = true;
            rs = mk(ews + sdat, bad ? 0 : 63 * XP + 128);
#pragma unroll
            for (int k = 0; k < DS; ++k) {
                v[k] = __builtin_amdgcn_raw_buffer_load_b128(rs, lv, k * ls, A_OBL); __builtin_amdgcn_sched_barrier(0);
                __builtin_amdgcn_raw_buffer_store_b128((u32x4){0u, 0u, 0u, 0u}, mk(myws, 0), vl, 0, SAUX); __builtin_amdgcn_sched_barrier(0);
            }
        }
        const unsigned ss = hf.isx ? 8 * XP : 4 * WP;
        const __amdgpu_buffer_rsrc_t rd = mk(myws + hf.dst, hf.isx ? 63 * XP + 128 : 31 * WP + 256);
        const unsigned sv = hf.isx ? vx : vw, flag = hf.flag;
        const bool priv = s < NIN;
        if (priv) s = __builtin_amdgcn_readfirstlane(tk) - qbase;
        tk = (unsigned)__builtin_amdgcn_raw_ptr_buffer_atomic_add_i32(1, qctr, priv ? lane * 4 : 4, 0, 0);
        const unsigned tkp = (unsigned)__builtin_amdgcn_raw_ptr_buffer_atomic_add_i32(1, pctr, s >= NIN ? lane * 4 : 4, 0, 0);
        if (priv && s >= NIN - LEAD0 && lane == 0)
            __hip_atomic_fetch_max((unsigned*)(myws + OFF_GO + (wid + 1) * 128), call * 1024u + (s - (NIN - LEAD0)), __ATOMIC_RELAXED, __HIP_MEMORY_SCOPE_AGENT);
        unsigned av = 0;
#define IN_ITEM(k) do { \
            if (k == AVK) { \
                if (s >= NIN) { const unsigned t = __builtin_amdgcn_readfirstlane(tkp) - pbase; s = t < NPOOL ? NIN + t : ~0u; } \
                locate(s, hf, sdat, sflag); \
                const bool loc = s >= AVR && s < NIN;       \
                av = __builtin_amdgcn_raw_buffer_load_b32(mk(loc ? myws + OFF_AV + (wid + 1) * 1024 + (s >> 2) * 4 : ews + sflag, s != ~0u ? 4 : 0), 0, 0, SAUX); \
            } \
            if (k + DS == 8) {              \
                pre = s != ~0u && (__builtin_amdgcn_readfirstlane(av) >= call || bad); \
                rs = mk(ews + sdat, pre && !bad ? 63 * XP + 128 : 0); \
                lv = s >= NIN ? vx : vl; ls = s >= NIN ? 8 * XP : 1024; \
            } \
            asm volatile("" : "+v"(v[k & 3]));        \
            __builtin_amdgcn_sched_barrier(0); \
            if (k == DS && pflag) {         \
                asm volatile("s_waitcnt vmcnt(%0)" :: "i"(2 * DS) : "memory"); \
                if (lane == 0) st_sys(myws + pflag, call); \
                asm volatile("" ::: "memory"); \
            } \
            v[(k + DS) & 3] = __builtin_amdgcn_raw_buffer_load_b128(rs, lv, ((k + DS) & 7) * ls, A_OBL); \
            __builtin_amdgcn_sched_barrier(0); \
            __builtin_amdgcn_raw_buffer_store_b128(v[k & 3], rd, sv, k * ss, A_XST); \
            __builtin_amdgcn_sched_barrier(0); \
            asm volatile("s_nop 1");        \
            __builtin_amdgcn_sched_barrier(0); } while (0)
        IN_ITEM(0); IN_ITEM(1); IN_ITEM(2); IN_ITEM(3); IN_ITEM(4); IN_ITEM(5); IN_ITEM(6); IN_ITEM(7);
#undef IN_ITEM
        pflag = flag;
    }
    DBG_STAMP(1);
    return pflag;
}

__device__ __forceinline__ void copy_role(const Params& P, int dd, unsigned n0, unsigned call) {
    const int lane = threadIdx.x & 63, wv = __builtin_amdgcn_readfirstlane(threadIdx.x >> 6);
    const int g = P.rank, d = (g + dd) & 7;
    char* myws = P.ws[0]; char* dws = P.ws[dd];
    const unsigned vx = (lane >> 3) * XP + (lane & 7) * 16, vw = (lane >> 4) * WP + (lane & 15) * 16;
    const char* const px = P.x; const char* const pwm = P.w;
    constexpr unsigned n1 = NBLK + NPL;
    u32x4 a[4], b[4];
    Half ha, hb;
    auto fetch = [&](unsigned n, u32x4 (&r)[4], Half& hf) {
        hf = half_of(P, 2 * (n < n1 ? n : 0) + (wv >> 2), d, 8 - dd);
        const __amdgpu_buffer_rsrc_t rs = mk((hf.isx ? px : pwm) + hf.src, n < n1 ? (hf.isx ? XB : WB) : 0);
        const unsigned st = hf.isx ? 8 * XP : 4 * WP;
#pragma unroll
        for (int j = 0; j < 4; ++j) r[j] = __builtin_amdgcn_raw_buffer_load_b128(rs, hf.isx ? vx : vw, ((wv & 3) * 4 + j) * st, 0);
    };
    fetch(n0, a, ha);
    for (unsigned n = n0; n < n1; n += 16) {
        const bool pool = n >= NBLK;
        const unsigned st = pool ? 8 * XP : 1024u;
        const __amdgpu_buffer_rsrc_t rd = pool ? mk(myws + OFF_XS + ha.src, XB) : mk(myws + OFF_OB + (size_t)(dd * 240 + n) * 32768 + (wv >> 2) * 16384, 16384);
        const unsigned vo = (pool ? vx : lane * 16u) + (wv & 3) * 4 * st;
#pragma unroll
        for (int j = 0; j < 4; ++j) __builtin_amdgcn_raw_buffer_store_b128(a[j], rd, vo + j * st, 0, A_OBS);
        __builtin_amdgcn_sched_barrier(0);
        fetch(n + 16, b, hb);
        asm volatile("s_waitcnt vmcnt(4)" ::: "memory");
        __builtin_amdgcn_s_barrier();
        if (pool) { if ((threadIdx.x & 255) < 2) st_sys(myws + ha.flag - (8 - dd) * 16 + (threadIdx.x & 1) * 4, call); }
        else if (threadIdx.x < 2) st_sys(threadIdx.x ? dws + OFF_AV + ((8 - dd) * 256 + n) * 4 : myws + OFF_AV2 + (dd * 256 + n) * 128, call);
#pragma unroll
        for (int j = 0; j < 4; ++j) a[j] = b[j];
        ha = hb;
    }
    DBG_STAMP(5);
}

__device__ __forceinline__ void quick_role(const Params& P, int dd, unsigned i, unsigned cntv) {
    const int lane = threadIdx.x & 63, wv = __builtin_amdgcn_readfirstlane(threadIdx.x >> 6);
    const int d = (P.rank + dd) & 7;
    char* myws = P.ws[0]; char* dws = P.ws[dd];
    const unsigned gdv = ld_agent((const unsigned*)(myws + OFF_GD + dd * 128));
    const unsigned vx = (lane >> 3) * XP + (lane & 7) * 16, vw = (lane >> 4) * WP + (lane & 15) * 16;
    const char* const px = P.x; const char* const pwm = P.w;
    const bool push = i < 4 * NP0;
    const Half h0 = half_of(P, i >> 1, d, 8 - dd), h1 = half_of(P, (push ? 4 * (240 - NP0) + i : 0) >> 1, d, 8 - dd);
    const unsigned o0 = (h0.isx ? vx : vw) + ((i & 1) * 8 + wv) * (h0.isx ? 8 * XP : 4 * WP), o1 = (h1.isx ? vx : vw) + ((i & 1) * 8 + wv) * (h1.isx ? 8 * XP : 4 * WP);
    const u32x4 a = __builtin_amdgcn_raw_buffer_load_b128(mk((h0.isx ? px : pwm) + h0.src, h0.isx ? XB : WB), o0, 0, 0);
    const u32x4 b = __builtin_amdgcn_raw_buffer_load_b128(mk((h1.isx ? px : pwm) + h1.src, push ? (h1.isx ? XB : WB) : 0), o1, 0, 0);
    __builtin_amdgcn_sched_barrier(0);
    const unsigned call = __builtin_amdgcn_readfirstlane(cntv) + 1u;
    if (__builtin_amdgcn_readfirstlane(gdv) < call - 1u)
        wait_ge(myws + OFF_GD + dd * 128, call - 1u, (unsigned*)(myws + OFF_TMO), call);
    __builtin_amdgcn_raw_buffer_store_b128(a, mk(myws + OFF_OB + (size_t)dd * 240 * 32768 + i * 8192, 8192), wv * 1024 + lane * 16, 0, A_OBS);
    __builtin_amdgcn_sched_barrier(0);
    __builtin_amdgcn_raw_buffer_store_b128(b, mk(dws + h1.dst, i < NPA ? (h1.isx ? XB : WB) : 0), o1, 0, SAUX);
    asm volatile("s_waitcnt vmcnt(1)" ::: "memory");
    __builtin_amdgcn_s_barrier();
    if (threadIdx.x == 0) st_sys(myws + OFF_AV3 + (dd * 32 + i) * 128, call);
    if (push) {
        __builtin_amdgcn_raw_buffer_store_b128(b, mk(dws + h1.dst, i >= NPA ? (h1.isx ? XB : WB) : 0), o1, 0, SAUX);
        asm volatile("s_waitcnt vmcnt(0)" ::: "memory");
        __builtin_amdgcn_s_barrier();
        if (threadIdx.x == 0) st_sys(dws + h1.flag + (i & 1) * 4, call);
    }
    DBG_STAMP(6);
}

#define PORDER 1
__device__ __forceinline__ unsigned ptile(unsigned o) { return PORDER ? (o & 7) | ((o & 8) << 1) | ((o & 16) >> 1) : o; }
__device__ __forceinline__ void pull_role(const Params& P, int pw, unsigned call, unsigned pend) {
    const int lane = threadIdx.x & 63, hw = __builtin_amdgcn_readfirstlane(threadIdx.x >> 6);
    if (hw >= 7) return;
    const int wid = (hw + pw) % 7;
    const int g = P.rank;
    char* myws = P.ws[0];
    const int d = (g + wid + 1) & 7;
    const __amdgpu_buffer_rsrc_t qctr = mk(myws + OFF_Q + (8 + wid + 1) * 256, 4);
    const unsigned qbase = (call - 1u) * NPULL - 2 * NPW;
    unsigned* tmo = (unsigned*)(myws + OFF_TMO);
    const char* fo = myws + OFF_FO + ((wid + 1) * 64 + lane) * 4;
    unsigned fv = ld_sys(fo), tk = qbase + NPW + pw, u = pw;
    asm volatile("s_waitcnt vmcnt(0)" ::: "memory");
    if (pend && lane == 0) st_sys(myws + pend, call);
    bool bad = false;
    if (pw < NPW0) {
        const unsigned want = call * 1024u + pw * LEAD1 / NPW0;
        const unsigned* go = (const unsigned*)(myws + OFF_GO + (wid + 1) * 128);
        const unsigned long long t0 = __builtin_amdgcn_s_memrealtime();
        for (unsigned polls = 1; __builtin_amdgcn_readfirstlane(ld_agent(go)) < want; ++polls) {
            if ((polls & 15u) == 0u && give_up(tmo, call, t0)) { bad = true; break; }
            __builtin_amdgcn_s_sleep(16);
        }
    }
    DBG_STAMP(5);
    const char* dps = P.ws[wid + 1] + OFF_PS + (call & 1) * PS_PAR + (size_t)(2 * g) * 16 * 131072;
    const char* dds = myws + OFF_DS + (size_t)(wid + 1) * 32 * 262144;
    char* ob = P.out + (size_t)d * NB * 2;
    const unsigned vl = lane * 16, vs = (lane & 15) * WP + (lane >> 4) * 16;
    const __amdgpu_buffer_rsrc_t rfo = mk(myws + OFF_FO + (wid + 1) * 256, 256);
    const unsigned long long m0 = __ballot(fv >= call);
    unsigned rdy = bad ? ~0u : (unsigned)m0 & (unsigned)(m0 >> 32); bool pre = false;
    u32x4 vp[4]; f32x4 vd[4][2];
    __amdgpu_buffer_rsrc_t rs = mk(myws, 0), rsd = mk(myws, 0);
#define LOAD_P(item) vp[item] = __builtin_amdgcn_raw_buffer_load_b128(rs, vl + (item) * 1024, 0, A_PL)
#define LOAD_D(item) do { vd[item][0] = __builtin_bit_cast(f32x4, __builtin_amdgcn_raw_buffer_load_b128(rsd, vl + (item) * 2048, 0, A_DL)); \
        vd[item][1] = __builtin_bit_cast(f32x4, __builtin_amdgcn_raw_buffer_load_b128(rsd, vl + (item) * 2048 + 1024, 0, A_DL)); } while (0)
#define FINISH(k) do { const u32x4 p = vp[k]; const f32x4 d0 = vd[k][0], d1 = vd[k][1]; u32x4 o; \
            o.x = pk_bf16(silu(__uint_as_float(p.x << 16) + d0[0]), silu(__uint_as_float(p.x & 0xffff0000u) + d0[1])); \
            o.y = pk_bf16(silu(__uint_as_float(p.y << 16) + d0[2]), silu(__uint_as_float(p.y & 0xffff0000u) + d0[3])); \
            o.z = pk_bf16(silu(__uint_as_float(p.z << 16) + d1[0]), silu(__uint_as_float(p.z & 0xffff0000u) + d1[1])); \
            o.w = pk_bf16(silu(__uint_as_float(p.w << 16) + d1[2]), silu(__uint_as_float(p.w & 0xffff0000u) + d1[3])); \
            if (bad) o = (u32x4){QNAN2, QNAN2, QNAN2, QNAN2}; \
            __builtin_amdgcn_raw_buffer_store_b128(o, rd, vs + k * 64, 0, OAUX); } while (0)
    while (u < NPULL) {
        const unsigned tile = ptile(u >> 5), part = u & 31;
        if (!pre) {
            const unsigned long long t0 = __builtin_amdgcn_s_memrealtime();
            while (!((rdy >> tile) & 1u)) {
                const unsigned long long m = __ballot(ld_sys(fo) >= call);
                rdy = (unsigned)m & (unsigned)(m >> 32);
                if ((rdy >> tile) & 1u) break;
                if (give_up(tmo, call, t0)) { rdy = ~0u; bad = true; break; }
                __builtin_amdgcn_s_sleep(4);
            }
            rs = mk(dps + (size_t)(tile * 32 + part) * 4096, bad ? 0 : 4096); rsd = mk(dds + (size_t)(tile * 32 + part) * 8192, 8192);
#pragma unroll
            for (int k = 0; k < DD; ++k) {
                if (k >= DD - DP) LOAD_P(k - (DD - DP));
                LOAD_D(k);
                __builtin_amdgcn_sched_barrier(0);
                __builtin_amdgcn_raw_buffer_store_b128((u32x4){0u, 0u, 0u, 0u}, mk(myws, 0), vl, 0, OAUX);
                __builtin_amdgcn_sched_barrier(0);
            }
        }
        const unsigned un = __builtin_amdgcn_readfirstlane(tk) - qbase;
        {
            const unsigned long long m = __ballot(fv >= call);
            rdy |= (unsigned)m & (unsigned)(m >> 32);
        }
        tk = (unsigned)__builtin_amdgcn_raw_ptr_buffer_atomic_add_i32(1, qctr, lane * 4, 0, 0);
        {
            const unsigned oa = (un >> 5) + FAHEAD;
            fv = __builtin_amdgcn_raw_buffer_load_b32(rfo, ((rdy >> ptile(oa < 31 ? oa : 31)) & (rdy >> ptile(oa < 32 ? oa - 1 : 31)) & 1u) ? 256 : lane * 4, 0, SAUX);
        }
        const __amdgpu_buffer_rsrc_t rd = mk(ob + (size_t)((tile >> 4) * 256 + (part >> 1) * 16) * WP + (tile & 15) * 512 + (part & 1) * 256, 15 * WP + 256);
        const unsigned tn = ptile((un >> 5) & 31), on = tn * 32 + (un & 31);
        pre = un < NPULL && ((rdy >> tn) & 1u);
        if (LASTALL && un >= NPULL) {
#pragma unroll
            for (int k = DD; k < 4; ++k) LOAD_D(k);
#pragma unroll
            for (int k = DP; k < 4; ++k) LOAD_P(k);
            __builtin_amdgcn_sched_barrier(0);
#pragma unroll
            for (int k = 0; k < 4; ++k) { FINISH(k); }
            break;
        }
#pragma unroll
        for (int k = 0; k < 4; ++k) {
            if (k + DP == 4) rs = mk(dps + (size_t)on * 4096, pre && !bad ? 4096 : 0);
            if (k + DD == 4) rsd = mk(dds + (size_t)on * 8192, pre ? 8192 : 0);
            asm volatile("" : "+v"(vp[k]), "+v"(vd[k][0]), "+v"(vd[k][1]));
            __builtin_amdgcn_sched_barrier(0);
            LOAD_P((k + DP) & 3);
            LOAD_D((k + DD) & 3);
            __builtin_amdgcn_sched_barrier(0);
            FINISH(k);
            __builtin_amdgcn_sched_barrier(0);
        }
        u = un;
    }
#undef LOAD_P
#undef FINISH
#undef LOAD_D
    DBG_STAMP(6);
}

__device__ __forceinline__ void gemm_tile(const Params& P, LAS unsigned char* lds, int mode, int pm, int pn, int qd, unsigned call) {
    const int tid = threadIdx.x, wid = __builtin_amdgcn_readfirstlane(tid >> 6), lane = tid & 63, wr = wid >> 2, wc = wid & 3, fr = lane & 15, fq = lane >> 4;
    const int g = P.rank;
    char* myws = P.ws[0];
    unsigned voffA[2], voffB[2];
#pragma unroll
    for (int i = 0; i < 2; ++i) {
        int R, C; stage_rc(tid * 16 + i * 8192, R, C); voffA[i] = (unsigned)R * XP + (unsigned)C * 2u;
        const int row = lane >> 3, ch = (lane & 7) ^ (((row >> 1) & 1) << 2);
        voffB[i] = (unsigned)(((wid >> 1) + 4 * i) * 8 + row) * WP + (unsigned)(wid & 1) * 128u + (unsigned)ch * 16u;
    }
    const unsigned ldsw = (unsigned)wid * 1024u, ldswb = (unsigned)((wid >> 2) * KBP + ((wid >> 1) & 1) * (2048 + 8 * BSHIFT) + (wid & 1) * 1024);
    const int aoff = lds_byte(wr * 64 + fr, fq * 8);
    const int boff = (fq >> 1) * KBP + (fq & 1) * (2048 + 8 * BSHIFT) + (wc >> 1) * 1024 + (fr >> 2) * 128 + ((4 * (wc & 1) + (fr & 3)) ^ (((fr >> 3) & 1) << 2)) * 16;
    constexpr size_t hA = (size_t)128 * XP, hB = 256;
    const int nt = mode == 0 ? 64 : (mode == 1 ? 56 : 8);
    const char* xl = P.x + (size_t)pm * 256 * XP;
    const char* wl = P.w + (size_t)(mode == 2 ? (g + qd) & 7 : g) * NB * 2 + pn * 512;
    const char* xs = myws + OFF_XS + (size_t)pm * 256 * XP;
    const char* wsb = myws + OFF_WS + pn * 512;
    auto ktile = [&](unsigned id, const char*& a, const char*& b) {
        const unsigned c = id >> 3, q = id & 7;
        a = (q ? xs : xl) + (q * (GM * XP) + c * 128); b = (q ? wsb : wl) + (c * (64 * WP) + q * (NB * 2));
    };
#define SA(b, h) (((b) * 2 + (h)) * HTB)
#define SB(b, h) (4 * HTB + ((b) * 2 + (h)) * HTBB)
#define STAGEB(bufoff, gbase) do { _Pragma("unroll") for (int _i = 0; _i < 2; ++_i) \
        __builtin_amdgcn_global_load_lds((const unsigned*)((const char*)(gbase) + voffB[_i]), (LAS unsigned*)(lds + (bufoff) + ldswb + _i * 2 * KBP), 16, 0, 0); } while (0)
#define STAGE(bufoff, gbase, voff) do { _Pragma("unroll") for (int _i = 0; _i < 2; ++_i) \
        __builtin_amdgcn_global_load_lds((const unsigned*)((const char*)(gbase) + (voff)[_i]), (LAS unsigned*)(lds + (bufoff) + ldsw + _i * 8192), 16, 0, 0); } while (0)
#define LDA(dst, b, h) do { _Pragma("unroll") for (int m = 0; m < 4; ++m) _Pragma("unroll") for (int k = 0; k < 2; ++k) dst[m][k] = *(const LAS bf16x8*)(lds + SA(b, h) + aoff + m * 2048 + k * 1024); } while (0)
#define TRD(dst, off) asm volatile("ds_read_b64_tr_b16 %0, %1 offset:%2" : "=v"(dst) : "v"(baddr), "i"(off) : "memory")
#define LDB(dst, b, h) do { _Pragma("unroll") for (int n = 0; n < 2; ++n) _Pragma("unroll") for (int k = 0; k < 2; ++k) { \
        TRD(dst[n][k][0], ((b) * 2 + (h)) * HTBB + k * 2 * KBP + n * 8); TRD(dst[n][k][1], ((b) * 2 + (h)) * HTBB + k * 2 * KBP + 512 + n * 8); } } while (0)
#define WAIT_B(Bt) asm volatile("s_waitcnt lgkmcnt(0)" : "+v"(Bt[0][0][0]), "+v"(Bt[0][0][1]), "+v"(Bt[0][1][0]), "+v"(Bt[0][1][1]), "+v"(Bt[1][0][0]), "+v"(Bt[1][0][1]), "+v"(Bt[1][1][0]), "+v"(Bt[1][1][1]) :: "memory")
#define BF(Bt, n, k) __builtin_shufflevector(Bt[n][k][0], Bt[n][k][1], 0, 1, 2, 3, 4, 5, 6, 7)
#define MMA(ai, bj, At, Bt) do { __builtin_amdgcn_s_setprio(1); _Pragma("unroll") for (int m = 0; m < 4; ++m) _Pragma("unroll") for (int n = 0; n < 2; ++n) _Pragma("unroll") for (int k = 0; k < 2; ++k) \
        acc[ai][bj][m][n] = __builtin_amdgcn_mfma_f32_16x16x32_bf16(BF(Bt, n, k), At[m][k], acc[ai][bj][m][n], 0, 0, 0); __builtin_amdgcn_s_setprio(0); } while (0)
#define WAIT_V(n) asm volatile("s_waitcnt vmcnt(" #n ")" ::: "memory")
#define WAIT_L(n) asm volatile("s_waitcnt lgkmcnt(" #n ")" ::: "memory")
#define BAR __builtin_amdgcn_s_barrier()
#define SCHED __builtin_amdgcn_sched_barrier(0)
    f32x4 acc[2][2][4][2];
#pragma unroll
    for (int a = 0; a < 2; ++a)
#pragma unroll
        for (int b = 0; b < 2; ++b)
#pragma unroll
            for (int m = 0; m < 4; ++m)
#pragma unroll
                for (int n = 0; n < 2; ++n) acc[a][b][m][n] = (f32x4){0.f, 0.f, 0.f, 0.f};
    bf16x8 At[4][2]; s16x4 B0[2][2][2], B1[2][2][2];
    const unsigned baddr = (unsigned)(size_t)lds + 4 * HTB + boff;
    constexpr unsigned long long LOCAL = 0x0101010101010101ull;
    unsigned long long rdy = LOCAL;
    unsigned long long todo = (mode == 0 ? ~0ull : (mode == 1 ? ~(LOCAL << qd) : LOCAL)) & ~0x101ull;
    LAS unsigned long long* rdy_lds = (LAS unsigned long long*)(lds + STAGE_BYTES);
    STAGEB(SB(0, 0), wl); STAGE(SA(0, 0), xl, voffA); STAGEB(SB(0, 1), wl + hB); STAGE(SA(0, 1), xl + hA, voffA);
    if (wr == 1) BAR;
    WAIT_V(4); BAR;
    STAGEB(SB(1, 0), wl + (size_t)64 * WP); STAGE(SA(1, 0), xl + 128, voffA); STAGEB(SB(1, 1), wl + (size_t)64 * WP + hB);
    WAIT_V(6); BAR;
    const char* a1 = xl + 128;
    for (int t = 0; t < nt; t += 2) {
        if (todo != 0ull && __builtin_popcountll(todo & rdy) < 2) {
            if (wid == 0) {
                const __amdgpu_buffer_rsrc_t rfx = mk(myws + OFF_FX + pm * 1024, 1024), rfw = mk(myws + OFF_FW + pn * 1024, 1024);
                const unsigned long long t0 = __builtin_amdgcn_s_memrealtime();
                unsigned long long mask = rdy;
                const unsigned long long pend = todo & ~rdy, pend2 = pend & (pend - 1ull);
                const int l1 = (__builtin_ctzll(pend2 ? pend2 : pend) | 7) + 8;
                unsigned vo = (pend >> lane) & 1ull ? lane * 16 : 4096;
                for (unsigned polls = 1;; ++polls) {
                    const u32x4 vx = __builtin_amdgcn_raw_buffer_load_b128(rfx, vo, 0, PAUX), vw = __builtin_amdgcn_raw_buffer_load_b128(rfw, vo, 0, PAUX);
                    asm volatile("" ::: "memory");
                    const bool ok = min(min(vx.x, vx.y), min(vx.z, vx.w)) >= call && min(min(vw.x, vw.y), min(vw.z, vw.w)) >= call;
                    mask |= __ballot(ok);
                    if (__builtin_popcountll(mask & todo) >= 2) break;
                    if ((polls & 7u) == 0u && give_up((unsigned*)(myws + OFF_TMO), call, t0)) { mask = ~0ull; break; }
                    if (lane > l1) vo = 4096;
                    __builtin_amdgcn_s_sleep(PSLEEP);
                }
                __builtin_amdgcn_fence(__ATOMIC_ACQUIRE, "");
                if (lane == 0) *rdy_lds = mask;
                asm volatile("s_waitcnt vmcnt(0) lgkmcnt(0)" ::: "memory");
            }
            asm volatile("" ::: "memory"); BAR; asm volatile("" ::: "memory");
            const unsigned long long mv = *rdy_lds;
            rdy = ((unsigned long long)(unsigned)__builtin_amdgcn_readfirstlane((unsigned)(mv >> 32)) << 32) | (unsigned)__builtin_amdgcn_readfirstlane((unsigned)mv);
        }
        LDB(B0, 0, 0); SCHED; LDA(At, 0, 0); STAGE(SA(1, 1), a1 + hA, voffA); SCHED;
        unsigned long long avail = todo ? todo & rdy : 0x101ull;
        const unsigned id2 = __builtin_ctzll(avail); avail &= avail - 1ull;
        const unsigned id3 = __builtin_ctzll(avail);
        todo &= ~((1ull << id2) | (1ull << id3));
        const char *a2, *b2, *a3, *b3;
        ktile(id2, a2, b2); ktile(id3, a3, b3);
        WAIT_L(8); BAR; WAIT_B(B0); MMA(0, 0, At, B0); BAR; SCHED;
        LDB(B1, 0, 1); STAGEB(SB(0, 0), b2);
        BAR; WAIT_B(B1); MMA(0, 1, At, B1); BAR;
        LDA(At, 0, 1); STAGE(SA(0, 0), a2, voffA);
        BAR; WAIT_L(0); MMA(1, 0, At, B0); BAR; SCHED;
        STAGEB(SB(0, 1), b2 + hB);
        WAIT_V(6); BAR; MMA(1, 1, At, B1); BAR;
        LDB(B0, 1, 0); SCHED; LDA(At, 1, 0); STAGE(SA(0, 1), a2 + hA, voffA);
        WAIT_L(8); BAR; WAIT_B(B0); MMA(0, 0, At, B0); BAR; SCHED;
        LDB(B1, 1, 1); STAGEB(SB(1, 0), b3);
        BAR; WAIT_B(B1); MMA(0, 1, At, B1); BAR;
        LDA(At, 1, 1); STAGE(SA(1, 0), a3, voffA);
        BAR; WAIT_L(0); MMA(1, 0, At, B0); BAR; SCHED;
        STAGEB(SB(1, 1), b3 + hB);
        WAIT_V(6); BAR; MMA(1, 1, At, B1); BAR;
        a1 = a3;
    }
    if (wr == 0) BAR;
    DBG_STAMP(mode == 2 ? 1 : 2);
    const bool bad = __builtin_amdgcn_readfirstlane(ld_agent((const unsigned*)(myws + OFF_TMO))) == call;
    int tz = threadIdx.x; asm volatile("" : "+v"(tz));
    const int l2 = tz & 63;
    if (mode == 2) {
        const __amdgpu_buffer_rsrc_t ro = mk(myws + OFF_DS + (size_t)(qd * 32 + (pm & 1) * 16 + pn) * 262144, 262144);
        const unsigned vo = (unsigned)(wr * 4 * 16384 + wc * 2048 + l2 * 16);
#pragma unroll
        for (int ai = 0; ai < 2; ++ai)
#pragma unroll
            for (int m = 0; m < 4; ++m)
#pragma unroll
                for (int bj = 0; bj < 2; ++bj)
#pragma unroll
                    for (int n = 0; n < 2; ++n)
                        __builtin_amdgcn_raw_buffer_store_b128(__builtin_bit_cast(u32x4, acc[ai][bj][m][n]), ro, vo + (unsigned)((ai * 8 + m) * 16384 + bj * 8192 + n * 1024), 0, A_DS);
    } else if (mode == 1) {
        const __amdgpu_buffer_rsrc_t ro = mk(myws + OFF_PS + (call & 1) * PS_PAR + (size_t)(pm * 16 + pn) * 131072, 131072);
        const unsigned vo = (unsigned)(wr * 4 * 8192 + wc * 1024 + l2 * 16);
#pragma unroll
        for (int ai = 0; ai < 2; ++ai)
#pragma unroll
            for (int m = 0; m < 4; ++m)
#pragma unroll
                for (int bj = 0; bj < 2; ++bj) {
                    const f32x4 v0 = acc[ai][bj][m][0], v1 = acc[ai][bj][m][1];
                    u32x4 o = {pk_bf16(v0[0], v0[1]), pk_bf16(v0[2], v0[3]), pk_bf16(v1[0], v1[1]), pk_bf16(v1[2], v1[3])};
                    if (bad) o = (u32x4){QNAN2, QNAN2, QNAN2, QNAN2};
                    __builtin_amdgcn_raw_buffer_store_b128(o, ro, vo + (unsigned)((ai * 8 + m) * 8192 + bj * 4096), 0, A_PS);
                }
    } else {
        const __amdgpu_buffer_rsrc_t ro = mk(P.out + (size_t)(pm & 1) * 256 * WP + (size_t)g * NB * 2 + pn * 512, 255 * WP + 512);
        const unsigned vo = (unsigned)(wr * 64 + (l2 & 15)) * WP + (unsigned)(wc * 32 + 8 * (l2 >> 4)) * 2u;
#pragma unroll
        for (int ai = 0; ai < 2; ++ai)
#pragma unroll
            for (int m = 0; m < 4; ++m)
#pragma unroll
                for (int bj = 0; bj < 2; ++bj) {
                    const f32x4 v0 = acc[ai][bj][m][0], v1 = acc[ai][bj][m][1];
                    u32x4 o = {pk_bf16(silu(v0[0]), silu(v0[1])), pk_bf16(silu(v0[2]), silu(v0[3])), pk_bf16(silu(v1[0]), silu(v1[1])), pk_bf16(silu(v1[2]), silu(v1[3]))};
                    if (bad) o = (u32x4){QNAN2, QNAN2, QNAN2, QNAN2};
                    __builtin_amdgcn_raw_buffer_store_b128(o, ro, vo + (unsigned)(ai * 8 + m) * 16 * WP + bj * 256, 0, A_LS);
                }
    }
    WAIT_V(0);
    BAR;
    if (tid == 0) {
        if (mode == 1) st_sys(P.ws[qd] + OFF_FO + ((8 - qd) * 64 + (pm & 1) * 16 + pn) * 4, call);
        if (mode == 2) st_sys(myws + OFF_FO + (qd * 64 + 32 + (pm & 1) * 16 + pn) * 4, call);
        else if (__hip_atomic_fetch_add((unsigned*)(myws + OFF_GDC), 1u, __ATOMIC_RELAXED, __HIP_MEMORY_SCOPE_AGENT) == 256u * call - 1u) {
            for (int o = 1; o < 8; ++o) st_sys(P.ws[o] + OFF_GD + (8 - o) * 128, call);
        }
    }
    DBG_STAMP(mode == 2 ? 4 : 3);
#undef SA
#undef SB
#undef STAGE
#undef STAGEB
#undef LDA
#undef LDB
#undef TRD
#undef WAIT_B
#undef BF
#undef MMA
}

__global__ __launch_bounds__(512, 2) void fused_gemm_rs(const Params P) {
    extern __shared__ __attribute__((aligned(16))) unsigned char shm[];
    const int b = blockIdx.x, xcd = b & 7, slot = b >> 3;
    asm volatile("" :: "s"(P.ws[0]), "s"(P.rank), "s"(P.sched[10]), "s"(P.sched[26]), "s"(P.sched[42]), "s"(P.sched[58]));
    unsigned* cnt = (unsigned*)(P.ws[0] + OFF_CNT + (size_t)b * 128);
    const unsigned cntv = ld_agent(cnt);
    DBG_STAMP(0);
    unsigned pend = 0;
    int pw = -1;
    if (slot < CPX) {
        const unsigned call = __builtin_amdgcn_readfirstlane(cntv) + 1u;
        asm volatile("s_waitcnt vmcnt(0)" ::: "memory");
        __builtin_amdgcn_s_barrier();
        if (threadIdx.x == 448) __hip_atomic_store(cnt, call, __ATOMIC_RELAXED, __HIP_MEMORY_SCOPE_AGENT);
        pend = in_role(P, b, call);
        pw = NPW0 + b;
    } else {
        const int g = P.rank, s = slot - CPX;
        const int cl = s / 7, pcl = PORDER ? 2 : 1;
        const bool late = cl != 0 && cl != pcl;
        const int e = ((late && cl == 3 ? 7 : 0) + s % 7) * 8 + xcd;
        static_assert(NP0 <= 4, "pushed by the two early classes");
        { const int e2 = (late ? 112 : (cl == 0 ? 56 : 0)) + e; quick_role(P, e2 % 7 + 1, e2 / 7, cntv); }
        const unsigned call = __builtin_amdgcn_readfirstlane(cntv) + 1u;
        if (late) copy_role(P, e % 7 + 1, NCOMM / 4 + e / 7, call);
        for (int u = (late ? 0 : 2); u < (s < 4 ? 4 : 3); ++u) {
            int mode, pm, pn, qd;
            if (u < 2) { mode = 2; pm = 2 * g + u; pn = e / 7; qd = e % 7 + 1; }
            else if (u == 2) { mode = 1; qd = s % 7 + 1; pm = 2 * ((g + qd) & 7) + (cl >> 1); pn = xcd + 8 * (cl & 1); }
            else { mode = 0; qd = 0; pm = 2 * g + (s >> 1); pn = xcd + 8 * (s & 1); }
            gemm_tile(P, (LAS unsigned char*)shm, mode, pm, pn, qd, call);
        }
        if (threadIdx.x == 448) __hip_atomic_store(cnt, call, __ATOMIC_RELAXED, __HIP_MEMORY_SCOPE_AGENT);
        if (s >= 4 && s < 7) pw = (s - 4) * 8 + xcd;
        if (cl == pcl && s % 7 < XPW) pw = NPW0 + NCOMM + (s % 7) * 8 + xcd;
    }
    if (pw >= 0) pull_role(P, pw, __builtin_amdgcn_readfirstlane(cntv) + 1u, pend);
}

#define SCHED_V 7
void build_schedule(unsigned* sched) {
    unsigned char e[240]; int n = 0;
    auto W = [&](int j0, int j1, int c) { for (int j = j0; j < j1; ++j) e[n++] = (unsigned char)((j << 3) | c); };
    auto X = [&](int u0, int u1, int c) { for (int u = u0; u < u1; ++u) e[n++] = (unsigned char)(128 | (u << 3) | c); };
    for (int c = 0; c < 8; ++c) { X(2, 8, c); W(0, 8, c); if (c == 4) X(8, 14, 0); if (c == 6) X(8, 14, 1); }
    for (int c = 0; c < 6; ++c) { X(8, 14, c + 2); W(8, 16, c); if (c < 3) X(0, 2, c); if (c == 3) X(0, 2, 6); }
    W(8, 16, 6); W(8, 16, 7); X(0, 2, 3); X(0, 2, 4); X(0, 2, 5); X(0, 2, 7);
    for (int i = 0; i < 60; ++i) sched[i] = e[4 * i] | ((unsigned)e[4 * i + 1] << 8) | ((unsigned)e[4 * i + 2] << 16) | ((unsigned)e[4 * i + 3] << 24);
}
}

extern "C" void kernel_launch(int rank, int nranks, ncclComm_t comm, hipStream_t stream,
                              void* const* d_in, const int* in_sizes, int n_in,
                              void* d_out, int out_size, const int* in_dtypes, int out_dtype,
                              void* const* d_out_all, void* const* d_ws_all, size_t ws_bytes) {
    if (nranks != NR || (unsigned)rank >= (unsigned)NR || n_in < 2 || ws_bytes < WS_NEED ||
        in_sizes[0] != GM * KS || in_sizes[1] != KS * GN || out_size != MS * GN) {
        fprintf(stderr, "kernel_launch: unexpected problem shape\n");
        return;
    }
    HIP_CHECK(hipFuncSetAttribute((const void*)fused_gemm_rs, hipFuncAttributeMaxDynamicSharedMemorySize, LDS_BYTES));
    Params P;
    P.x = (const char*)d_in[0]; P.w = (const char*)d_in[1];
    for (int p = 0; p < NR; ++p) P.ws[p] = (char*)d_ws_all[(rank + p) & 7];
    P.out = (char*)d_out;
    P.rank = rank;
    build_schedule(P.sched);
    hipLaunchKernelGGL(fused_gemm_rs, dim3(GRID), dim3(512), LDS_BYTES, stream, P);
    HIP_CHECK(hipGetLastError());
}
```

```cpp
#include <hip/hip_runtime.h>
#include <rccl/rccl.h>
#include <cstdio>

#define HIP_CHECK(expr) do { hipError_t _e = (expr); if (_e != hipSuccess) { \
    fprintf(stderr, "HIP error %s at %s:%d\n", hipGetErrorString(_e), __FILE__, __LINE__); \
    return; } } while (0)

#define LAS __attribute__((address_space(3)))
typedef short s16x4 __attribute__((ext_vector_type(4)));
typedef short bf16x8 __attribute__((ext_vector_type(8)));
typedef float f32x4 __attribute__((ext_vector_type(4)));
typedef float f32x2 __attribute__((ext_vector_type(2)));
typedef __bf16 bf16x2 __attribute__((ext_vector_type(2)));
typedef unsigned u32x4 __attribute__((ext_vector_type(4)));

namespace {
constexpr int NR = 8;
constexpr int GM = 4096, GK = 4096, GN = 32768;
constexpr int KS = GK / NR;
constexpr int NB = GN / NR;
constexpr int MS = GM / NR;
constexpr unsigned XP = KS * 2;
constexpr unsigned WP = GN * 2;
constexpr int HTB = 128 * 64 * 2;
constexpr int KBP = 4096 + 16;
constexpr int HTBB = 4 * KBP;
constexpr int STAGE_BYTES = 4 * HTB + 4 * HTBB;
constexpr int LDS_BYTES = STAGE_BYTES + 64;
constexpr int GRID = 256;
constexpr int CPX = 4;
constexpr int NCOMM = 8 * CPX;
constexpr int NPW0 = 24;
constexpr int NPW = NPW0 + NCOMM + 8;
constexpr unsigned long long DEADLINE = 50000000ull;

constexpr size_t OFF_CNT = 0;
constexpr size_t OFF_TMO = 32768;
constexpr size_t OFF_GDC = OFF_TMO + 256;
constexpr size_t OFF_FX = 65536;
constexpr size_t OFF_FW = OFF_FX + 16384;
constexpr size_t OFF_AV = OFF_FW + 16384;
constexpr size_t OFF_FO = OFF_AV + 8192;
constexpr size_t OFF_Q = OFF_FO + 2048;
constexpr size_t OFF_GD = OFF_Q + 4096;
constexpr size_t OFF_GO = OFF_GD + 1024;
constexpr size_t OFF_AV2 = 524288;
constexpr size_t OFF_AV3 = OFF_AV2 + 262144;
constexpr size_t OFF_XS = 1u << 20;
constexpr size_t OFF_WS = OFF_XS + (size_t)NR * GM * XP;
constexpr size_t OFF_PS = OFF_WS + (size_t)KS * WP;
constexpr size_t PS_PAR = (size_t)256 * 131072;
constexpr size_t OFF_DS = OFF_PS + 2 * PS_PAR;
constexpr size_t OFF_OB = OFF_DS + (size_t)256 * 262144;
constexpr size_t WS_NEED = OFF_OB + (size_t)8 * 240 * 32768;
constexpr unsigned NP0 = 2;
constexpr unsigned NPL = 6;
constexpr unsigned NPOOL = 7 * 4 * NPL, POOL_STEP = NPOOL + 7 * NCOMM;
constexpr unsigned NBLK = 240 - NP0 - NPL;
constexpr unsigned NIN = 4 * NBLK;
constexpr unsigned NPULL = 1024;

struct Params {
    const char* x; const char* w;
    char* ws[NR];
    char* out;
    int rank;
    unsigned sched[60];
};

__device__ __forceinline__ int lds_byte(int r, int c) { const int st = (r >> 4) * 2 + (c >> 5), rr = r & 15, cc = c & 31, ob = rr * 64 + cc * 2; return st * 1024 + (ob ^ (((ob >> 9) & 1) << 5)); }
__device__ __forceinline__ void stage_rc(int b, int& R, int& C) { const int st = b / 1024, sb = b % 1024, swz = sb ^ (((sb >> 9) & 1) << 5); R = (st >> 1) * 16 + swz / 64; C = (st & 1) * 32 + (swz % 64) / 2; }
__device__ __forceinline__ __amdgpu_buffer_rsrc_t mk(const void* p, unsigned bytes) { return __builtin_amdgcn_make_buffer_rsrc((void*)p, 0, bytes, 0x00020000); }
__device__ __forceinline__ unsigned pk_bf16(float lo, float hi) { const bf16x2 r = __builtin_convertvector((f32x2){lo, hi}, bf16x2); return __builtin_bit_cast(unsigned, r); }
__device__ __forceinline__ float silu(float y) {
    const float c = __builtin_amdgcn_fmed3f(y, -60.0f, 60.0f);
    const float e = __builtin_amdgcn_exp2f(c * -1.44269504088896341f);
    return y * __builtin_amdgcn_rcpf(1.0f + e);
}
__device__ __forceinline__ unsigned ld_agent(const unsigned* p) { return __hip_atomic_load(p, __ATOMIC_RELAXED, __HIP_MEMORY_SCOPE_AGENT); }
__device__ __forceinline__ unsigned ld_sys(const void* p) { return __hip_atomic_load((const unsigned*)p, __ATOMIC_RELAXED, __HIP_MEMORY_SCOPE_SYSTEM); }
__device__ __forceinline__ void st_sys(void* p, unsigned v) { __hip_atomic_store((unsigned*)p, v, __ATOMIC_RELAXED, __HIP_MEMORY_SCOPE_SYSTEM); }
__device__ __forceinline__ bool give_up(unsigned* tmo, unsigned call, unsigned long long t0) {
    if (__builtin_amdgcn_readfirstlane(ld_agent(tmo)) != call && __builtin_amdgcn_s_memrealtime() - t0 <= DEADLINE) return false;
    __hip_atomic_store(tmo, call, __ATOMIC_RELAXED, __HIP_MEMORY_SCOPE_AGENT);
    return true;
}
__device__ __forceinline__ int panel_of(int u, int d, int q) {
    if (u < 2) return 2 * d + u;
    const int hi = u >= 8, j = u - (hi ? 7 : 1);
    return 2 * ((d + j + (j >= q)) & 7) + hi;
}

constexpr unsigned LEAD1 = 31;
constexpr unsigned AVR = 160;
constexpr int DS = 2;
constexpr int DP = 1, DD = 3;
constexpr int SAUX = 17;
constexpr int LAUX = 19;
constexpr int OAUX = 18;
constexpr unsigned QNAN2 = 0x7fc07fc0u;

struct Half { unsigned isx, src, dst, flag; };
__device__ __forceinline__ Half half_of(const Params& P, unsigned s, int d, int q) {
    const unsigned e = (P.sched[s >> 3] >> (((s >> 1) & 3) * 8)) & 0xffu;
    const unsigned c = e & 7, idx = (e >> 3) & 15, h = s & 1;
    Half r; r.isx = e >> 7;
    if (r.isx) {
        const unsigned pn = panel_of(idx, d, q);
        r.src = (pn * 256 + h * 128) * XP + c * 128; r.dst = (unsigned)OFF_XS + q * GM * XP + r.src;
        r.flag = (unsigned)OFF_FX + ((pn * 64 + c * 8 + q) * 2 + h) * 8;
    } else {
        const unsigned o = c * 64 * WP + idx * 512 + h * 256;
        r.src = o + d * NB * 2; r.dst = (unsigned)OFF_WS + q * NB * 2 + o;
        r.flag = (unsigned)OFF_FW + ((idx * 64 + c * 8 + q) * 2 + h) * 8;
    }
    return r;
}
constexpr unsigned XB = 127 * XP + 128, WB = 63 * WP + 256;
__device__ __forceinline__ bool wait_ge(const char* word, unsigned want, unsigned* tmo, unsigned call) {
    const unsigned long long t0 = __builtin_amdgcn_s_memrealtime();
    for (unsigned polls = 1; __builtin_amdgcn_readfirstlane(ld_sys(word)) < want; ++polls) {
        if ((polls & 15u) == 0u && give_up(tmo, call, t0)) return false;
        __builtin_amdgcn_s_sleep(1);
    }
    return true;
}

__device__ __forceinline__ unsigned in_role(const Params& P, int cw, unsigned call) {
    const int lane = threadIdx.x & 63, hw = __builtin_amdgcn_readfirstlane(threadIdx.x >> 6);
    if (hw >= 7) return 0;
    const int wid = (hw + cw) % 7;
    const int g = P.rank;
    char* myws = P.ws[0];
    const __amdgpu_buffer_rsrc_t qctr = mk(myws + OFF_Q + (wid + 1) * 256, 4);
    const unsigned vx = (lane >> 3) * XP + (lane & 7) * 16, vw = (lane >> 4) * WP + (lane & 15) * 16, vl = lane * 16;
    unsigned* tmo = (unsigned*)(myws + OFF_TMO);
    const unsigned qbase = (call - 1u) * NIN - 2 * NCOMM, pbase = (call - 1u) * POOL_STEP;
    const __amdgpu_buffer_rsrc_t pctr = mk(myws + OFF_Q, 4);
    const char* ews = P.ws[wid + 1];
    unsigned s = cw, pflag = 0, tk = qbase + NCOMM + cw;
    bool bad = false, pre = false;
    {
        const char* gd = myws + OFF_GD + ((lane & 7) ? lane & 7 : 1) * 128;
        if (!__all(ld_agent((const unsigned*)gd) >= call - 1u)) {
            const unsigned long long t0 = __builtin_amdgcn_s_memrealtime();
            while (!__all(ld_sys(gd) >= call - 1u)) { if (give_up(tmo, call, t0)) { bad = true; break; } __builtin_amdgcn_s_sleep(8); }
        }
    }
    __amdgpu_buffer_rsrc_t rs = mk(myws, 0);
    unsigned lv = vl, ls = 1024;
    u32x4 v[4];
    auto locate = [&](unsigned s, Half& hf, unsigned& sdat, unsigned& sflag) {
        const bool pool = s >= NIN;
        const unsigned t = pool && s != ~0u ? s - NIN : 0u, j = t / 7u, o = t - j * 7u + 1u, qe = (o - (wid + 1)) & 7u;
        const unsigned sq = pool ? NIN + j : s;
        hf = half_of(P, sq >> 1, g, pool ? o : wid + 1);
        const unsigned ss = hf.isx ? 8 * XP : 4 * WP;
        hf.dst += (sq & 1) * 8 * ss; hf.flag += (sq & 1) * 4;
        sdat = pool ? (unsigned)OFF_XS + qe * (GM * XP) + hf.src + (sq & 1) * 8 * ss : (unsigned)OFF_OB + ((7 - wid) * 960 + s) * 8192;
        sflag = pool ? hf.flag + qe * 16 - o * 16 : (s < NCOMM ? (unsigned)OFF_AV3 + ((7 - wid) * 32 + s) * 128 : (unsigned)OFF_AV2 + ((7 - wid) * 256 + (s >> 2)) * 128);
        asm volatile("" : "+s"(hf.dst), "+s"(hf.flag), "+s"(hf.isx), "+s"(sdat), "+s"(sflag));
    };
    Half hf; unsigned sdat, sflag;
    locate(s, hf, sdat, sflag);
    for (;;) {
        hf.dst = __builtin_amdgcn_readfirstlane(hf.dst); hf.flag = __builtin_amdgcn_readfirstlane(hf.flag); hf.isx = __builtin_amdgcn_readfirstlane(hf.isx);
        sdat = __builtin_amdgcn_readfirstlane(sdat); sflag = __builtin_amdgcn_readfirstlane(sflag);
        if (!pre) {
            if (s == ~0u) break;
            if (pflag) {
                asm volatile("s_waitcnt vmcnt(0)" ::: "memory");
                if (lane == 0) st_sys(myws + pflag, call);
                pflag = 0;
            }
            if (!wait_ge(ews + sflag, call, tmo, call)) bad = true;
            rs = mk(ews + sdat, bad ? 0 : 63 * XP + 128);
#pragma unroll
            for (int k = 0; k < DS; ++k) {
                v[k] = __builtin_amdgcn_raw_buffer_load_b128(rs, lv, k * ls, LAUX); __builtin_amdgcn_sched_barrier(0);
                __builtin_amdgcn_raw_buffer_store_b128((u32x4){0u, 0u, 0u, 0u}, mk(myws, 0), vl, 0, SAUX); __builtin_amdgcn_sched_barrier(0);
            }
        }
        const unsigned ss = hf.isx ? 8 * XP : 4 * WP;
        const __amdgpu_buffer_rsrc_t rd = mk(myws + hf.dst, hf.isx ? 63 * XP + 128 : 31 * WP + 256);
        const unsigned sv = hf.isx ? vx : vw, flag = hf.flag;
        const bool priv = s < NIN;
        if (priv) s = __builtin_amdgcn_readfirstlane(tk) - qbase;
        tk = (unsigned)__builtin_amdgcn_raw_ptr_buffer_atomic_add_i32(1, qctr, priv ? lane * 4 : 4, 0, 0);
        const unsigned tkp = (unsigned)__builtin_amdgcn_raw_ptr_buffer_atomic_add_i32(1, pctr, s >= NIN ? lane * 4 : 4, 0, 0);
        if (priv && s >= NIN && lane == 0)
            __hip_atomic_fetch_max((unsigned*)(myws + OFF_GO + (wid + 1) * 128), call * 1024u + (s - NIN), __ATOMIC_RELAXED, __HIP_MEMORY_SCOPE_AGENT);
        unsigned av = 0;
#define IN_ITEM(k) do { \
            if (k == 5) { \
                if (s >= NIN) { const unsigned t = __builtin_amdgcn_readfirstlane(tkp) - pbase; s = t < NPOOL ? NIN + t : ~0u; } \
                locate(s, hf, sdat, sflag); \
                const bool loc = s >= AVR && s < NIN;       \
                av = __builtin_amdgcn_raw_buffer_load_b32(mk(loc ? myws + OFF_AV + (wid + 1) * 1024 + (s >> 2) * 4 : ews + sflag, s != ~0u ? 4 : 0), 0, 0, SAUX); \
            } \
            if (k + DS == 8) {              \
                pre = s != ~0u && (__builtin_amdgcn_readfirstlane(av) >= call || bad); \
                rs = mk(ews + sdat, pre && !bad ? 63 * XP + 128 : 0); \
                lv = s >= NIN ? vx : vl; ls = s >= NIN ? 8 * XP : 1024; \
            } \
            asm volatile("" : "+v"(v[k & 3]));        \
            __builtin_amdgcn_sched_barrier(0); \
            if (k == DS && pflag) {         \
                asm volatile("s_waitcnt vmcnt(%0)" :: "i"(2 * DS) : "memory"); \
                if (lane == 0) st_sys(myws + pflag, call); \
                asm volatile("" ::: "memory"); \
            } \
            v[(k + DS) & 3] = __builtin_amdgcn_raw_buffer_load_b128(rs, lv, ((k + DS) & 7) * ls, LAUX); \
            __builtin_amdgcn_sched_barrier(0); \
            __builtin_amdgcn_raw_buffer_store_b128(v[k & 3], rd, sv, k * ss, SAUX); \
            __builtin_amdgcn_sched_barrier(0); \
            asm volatile("s_nop 1");        \
            __builtin_amdgcn_sched_barrier(0); } while (0)
        IN_ITEM(0); IN_ITEM(1); IN_ITEM(2); IN_ITEM(3); IN_ITEM(4); IN_ITEM(5); IN_ITEM(6); IN_ITEM(7);
#undef IN_ITEM
        pflag = flag;
    }
    return pflag;
}

__device__ __forceinline__ void copy_role(const Params& P, int dd, unsigned n0, unsigned call) {
    const int lane = threadIdx.x & 63, wv = __builtin_amdgcn_readfirstlane(threadIdx.x >> 6);
    const int g = P.rank, d = (g + dd) & 7;
    char* myws = P.ws[0]; char* dws = P.ws[dd];
    const unsigned vx = (lane >> 3) * XP + (lane & 7) * 16, vw = (lane >> 4) * WP + (lane & 15) * 16;
    const char* const px = P.x; const char* const pwm = P.w;
    constexpr unsigned n1 = NBLK + NPL;
    u32x4 a[4], b[4];
    Half ha, hb;
    auto fetch = [&](unsigned n, u32x4 (&r)[4], Half& hf) {
        hf = half_of(P, 2 * (n < n1 ? n : 0) + (wv >> 2), d, 8 - dd);
        const __amdgpu_buffer_rsrc_t rs = mk((hf.isx ? px : pwm) + hf.src, n < n1 ? (hf.isx ? XB : WB) : 0);
        const unsigned st = hf.isx ? 8 * XP : 4 * WP;
#pragma unroll
        for (int j = 0; j < 4; ++j) r[j] = __builtin_amdgcn_raw_buffer_load_b128(rs, hf.isx ? vx : vw, ((wv & 3) * 4 + j) * st, 0);
    };
    fetch(n0, a, ha);
    for (unsigned n = n0; n < n1; n += 16) {
        const bool pool = n >= NBLK;
        const unsigned st = pool ? 8 * XP : 1024u;
        const __amdgpu_buffer_rsrc_t rd = pool ? mk(myws + OFF_XS + ha.src, XB) : mk(myws + OFF_OB + (size_t)(dd * 240 + n) * 32768 + (wv >> 2) * 16384, 16384);
        const unsigned vo = (pool ? vx : lane * 16u) + (wv & 3) * 4 * st;
#pragma unroll
        for (int j = 0; j < 4; ++j) __builtin_amdgcn_raw_buffer_store_b128(a[j], rd, vo + j * st, 0, SAUX);
        __builtin_amdgcn_sched_barrier(0);
        fetch(n + 16, b, hb);
        if (n + 16 < n1) asm volatile("s_waitcnt vmcnt(4)" ::: "memory");
        else asm volatile("s_waitcnt vmcnt(0)" ::: "memory");
        __builtin_amdgcn_s_barrier();
        if (pool) { if ((threadIdx.x & 255) < 2) st_sys(myws + ha.flag - (8 - dd) * 16 + (threadIdx.x & 1) * 4, call); }
        else if (threadIdx.x < 2) st_sys(threadIdx.x ? dws + OFF_AV + ((8 - dd) * 256 + n) * 4 : myws + OFF_AV2 + (dd * 256 + n) * 128, call);
#pragma unroll
        for (int j = 0; j < 4; ++j) a[j] = b[j];
        ha = hb;
    }
}

__device__ __forceinline__ void quick_role(const Params& P, int dd, unsigned i, unsigned cntv) {
    const int lane = threadIdx.x & 63, wv = __builtin_amdgcn_readfirstlane(threadIdx.x >> 6);
    const int d = (P.rank + dd) & 7;
    char* myws = P.ws[0]; char* dws = P.ws[dd];
    const unsigned gdv = ld_agent((const unsigned*)(myws + OFF_GD + dd * 128));
    const unsigned vx = (lane >> 3) * XP + (lane & 7) * 16, vw = (lane >> 4) * WP + (lane & 15) * 16;
    const char* const px = P.x; const char* const pwm = P.w;
    const bool push = i < 4 * NP0;
    const Half h0 = half_of(P, i >> 1, d, 8 - dd);
    const unsigned o0 = (h0.isx ? vx : vw) + ((i & 1) * 8 + wv) * (h0.isx ? 8 * XP : 4 * WP);
    const u32x4 a = __builtin_amdgcn_raw_buffer_load_b128(mk((h0.isx ? px : pwm) + h0.src, h0.isx ? XB : WB), o0, 0, 0);
    asm volatile("" ::: "memory");
    const Half h1 = half_of(P, (push ? 4 * (240 - NP0) + i : 0) >> 1, d, 8 - dd);
    const unsigned o1 = (h1.isx ? vx : vw) + ((i & 1) * 8 + wv) * (h1.isx ? 8 * XP : 4 * WP);
    const u32x4 b = __builtin_amdgcn_raw_buffer_load_b128(mk((h1.isx ? px : pwm) + h1.src, push ? (h1.isx ? XB : WB) : 0), o1, 0, 0);
    __builtin_amdgcn_sched_barrier(0);
    const unsigned call = __builtin_amdgcn_readfirstlane(cntv) + 1u;
    if (__builtin_amdgcn_readfirstlane(gdv) < call - 1u)
        wait_ge(myws + OFF_GD + dd * 128, call - 1u, (unsigned*)(myws + OFF_TMO), call);
    __builtin_amdgcn_raw_buffer_store_b128(a, mk(myws + OFF_OB + (size_t)dd * 240 * 32768 + i * 8192, 8192), wv * 1024 + lane * 16, 0, SAUX);
    __builtin_amdgcn_sched_barrier(0);
    __builtin_amdgcn_raw_buffer_store_b128(b, mk(dws + h1.dst, i < 4 ? (h1.isx ? XB : WB) : 0), o1, 0, SAUX);
    if (i < 4) asm volatile("s_waitcnt vmcnt(1)" ::: "memory"); else asm volatile("s_waitcnt vmcnt(0)" ::: "memory");
    __builtin_amdgcn_s_barrier();
    if (threadIdx.x == 0) st_sys(myws + OFF_AV3 + (dd * 32 + i) * 128, call);
    if (push) {
        __builtin_amdgcn_raw_buffer_store_b128(b, mk(dws + h1.dst, i >= 4 ? (h1.isx ? XB : WB) : 0), o1, 0, SAUX);
        asm volatile("s_waitcnt vmcnt(0)" ::: "memory");
        __builtin_amdgcn_s_barrier();
        if (threadIdx.x == 0) st_sys(dws + h1.flag + (i & 1) * 4, call);
    }
}

__device__ __forceinline__ unsigned ptile(unsigned o) { return (o & 7) | ((o & 8) << 1) | ((o & 16) >> 1); }
__device__ __forceinline__ void pull_role(const Params& P, int pw, unsigned call, unsigned pend) {
    const int lane = threadIdx.x & 63, hw = __builtin_amdgcn_readfirstlane(threadIdx.x >> 6);
    if (hw >= 7) return;
    const int wid = (hw + pw) % 7;
    const int g = P.rank;
    char* myws = P.ws[0];
    const int d = (g + wid + 1) & 7;
    const __amdgpu_buffer_rsrc_t qctr = mk(myws + OFF_Q + (8 + wid + 1) * 256, 4);
    const unsigned qbase = (call - 1u) * NPULL - 2 * NPW;
    unsigned* tmo = (unsigned*)(myws + OFF_TMO);
    const char* fo = myws + OFF_FO + ((wid + 1) * 64 + lane) * 4;
    unsigned fv = ld_sys(fo), tk = qbase + NPW + pw, u = pw;
    asm volatile("s_waitcnt vmcnt(0)" ::: "memory");
    if (pend && lane == 0) st_sys(myws + pend, call);
    bool bad = false;
    if (pw < NPW0) {
        const unsigned want = call * 1024u + pw * LEAD1 / NPW0;
        const unsigned* go = (const unsigned*)(myws + OFF_GO + (wid + 1) * 128);
        const unsigned long long t0 = __builtin_amdgcn_s_memrealtime();
        for (unsigned polls = 1; __builtin_amdgcn_readfirstlane(ld_agent(go)) < want; ++polls) {
            if ((polls & 15u) == 0u && give_up(tmo, call, t0)) { bad = true; break; }
            __builtin_amdgcn_s_sleep(16);
        }
    }
    const char* dps = P.ws[wid + 1] + OFF_PS + (call & 1) * PS_PAR + (size_t)(2 * g) * 16 * 131072;
    const char* dds = myws + OFF_DS + (size_t)(wid + 1) * 32 * 262144;
    char* ob = P.out + (size_t)d * NB * 2;
    const unsigned vl = lane * 16, vs = (lane & 15) * WP + (lane >> 4) * 16;
    const __amdgpu_buffer_rsrc_t rfo = mk(myws + OFF_FO + (wid + 1) * 256, 256);
    const unsigned long long m0 = __ballot(fv >= call);
    unsigned rdy = bad ? ~0u : (unsigned)m0 & (unsigned)(m0 >> 32); bool pre = false;
    u32x4 vp[4]; f32x4 vd[4][2];
    __amdgpu_buffer_rsrc_t rs = mk(myws, 0), rsd = mk(myws, 0);
#define LOAD_P(item) vp[item] = __builtin_amdgcn_raw_buffer_load_b128(rs, vl + (item) * 1024, 0, LAUX)
#define LOAD_D(item) do { vd[item][0] = __builtin_bit_cast(f32x4, __builtin_amdgcn_raw_buffer_load_b128(rsd, vl + (item) * 2048, 0, LAUX)); \
        vd[item][1] = __builtin_bit_cast(f32x4, __builtin_amdgcn_raw_buffer_load_b128(rsd, vl + (item) * 2048 + 1024, 0, LAUX)); } while (0)
#define FINISH(k) do { const u32x4 p = vp[k]; const f32x4 d0 = vd[k][0], d1 = vd[k][1]; u32x4 o; \
            o.x = pk_bf16(silu(__uint_as_float(p.x << 16) + d0[0]), silu(__uint_as_float(p.x & 0xffff0000u) + d0[1])); \
            o.y = pk_bf16(silu(__uint_as_float(p.y << 16) + d0[2]), silu(__uint_as_float(p.y & 0xffff0000u) + d0[3])); \
            o.z = pk_bf16(silu(__uint_as_float(p.z << 16) + d1[0]), silu(__uint_as_float(p.z & 0xffff0000u) + d1[1])); \
            o.w = pk_bf16(silu(__uint_as_float(p.w << 16) + d1[2]), silu(__uint_as_float(p.w & 0xffff0000u) + d1[3])); \
            if (bad) o = (u32x4){QNAN2, QNAN2, QNAN2, QNAN2}; \
            __builtin_amdgcn_raw_buffer_store_b128(o, rd, vs + k * 64, 0, OAUX); } while (0)
    while (u < NPULL) {
        const unsigned tile = ptile(u >> 5), part = u & 31;
        if (!pre) {
            const unsigned long long t0 = __builtin_amdgcn_s_memrealtime();
            while (!((rdy >> tile) & 1u)) {
                const unsigned long long m = __ballot(ld_sys(fo) >= call);
                rdy = (unsigned)m & (unsigned)(m >> 32);
                if ((rdy >> tile) & 1u) break;
                if (give_up(tmo, call, t0)) { rdy = ~0u; bad = true; break; }
                __builtin_amdgcn_s_sleep(4);
            }
            rs = mk(dps + (size_t)(tile * 32 + part) * 4096, bad ? 0 : 4096); rsd = mk(dds + (size_t)(tile * 32 + part) * 8192, 8192);
#pragma unroll
            for (int k = 0; k < DD; ++k) {
                if (k >= DD - DP) LOAD_P(k - (DD - DP));
                LOAD_D(k);
                __builtin_amdgcn_sched_barrier(0);
                __builtin_amdgcn_raw_buffer_store_b128((u32x4){0u, 0u, 0u, 0u}, mk(myws, 0), vl, 0, OAUX);
                __builtin_amdgcn_sched_barrier(0);
            }
        }
        const unsigned un = __builtin_amdgcn_readfirstlane(tk) - qbase;
        {
            const unsigned long long m = __ballot(fv >= call);
            rdy |= (unsigned)m & (unsigned)(m >> 32);
        }
        tk = (unsigned)__builtin_amdgcn_raw_ptr_buffer_atomic_add_i32(1, qctr, lane * 4, 0, 0);
        {
            const unsigned oa = (un >> 5) + 3;
            fv = __builtin_amdgcn_raw_buffer_load_b32(rfo, ((rdy >> ptile(oa < 31 ? oa : 31)) & (rdy >> ptile(oa < 32 ? oa - 1 : 31)) & 1u) ? 256 : lane * 4, 0, SAUX);
        }
        const __amdgpu_buffer_rsrc_t rd = mk(ob + (size_t)((tile >> 4) * 256 + (part >> 1) * 16) * WP + (tile & 15) * 512 + (part & 1) * 256, 15 * WP + 256);
        const unsigned tn = ptile((un >> 5) & 31), on = tn * 32 + (un & 31);
        pre = un < NPULL && ((rdy >> tn) & 1u);
#pragma unroll
        for (int k = 0; k < 4; ++k) {
            if (k + DP == 4) rs = mk(dps + (size_t)on * 4096, pre && !bad ? 4096 : 0);
            if (k + DD == 4) rsd = mk(dds + (size_t)on * 8192, pre ? 8192 : 0);
            asm volatile("" : "+v"(vp[k]), "+v"(vd[k][0]), "+v"(vd[k][1]));
            __builtin_amdgcn_sched_barrier(0);
            LOAD_P((k + DP) & 3);
            LOAD_D((k + DD) & 3);
            __builtin_amdgcn_sched_barrier(0);
            FINISH(k);
            __builtin_amdgcn_sched_barrier(0);
        }
        u = un;
    }
#undef LOAD_P
#undef FINISH
#undef LOAD_D
}

__device__ __forceinline__ void gemm_tile(const Params& P, LAS unsigned char* lds, int mode, int pm, int pn, int qd, unsigned call) {
    const int tid = threadIdx.x, wid = __builtin_amdgcn_readfirstlane(tid >> 6), lane = tid & 63, wr = wid >> 2, wc = wid & 3, fr = lane & 15, fq = lane >> 4;
    const int g = P.rank;
    char* myws = P.ws[0];
    unsigned voffA[2], voffB[2];
#pragma unroll
    for (int i = 0; i < 2; ++i) {
        int R, C; stage_rc(tid * 16 + i * 8192, R, C); voffA[i] = (unsigned)R * XP + (unsigned)C * 2u;
        const int row = lane >> 3, ch = (lane & 7) ^ (((row >> 1) & 1) << 2);
        voffB[i] = (unsigned)(((wid >> 1) + 4 * i) * 8 + row) * WP + (unsigned)(wid & 1) * 128u + (unsigned)ch * 16u;
    }
    const unsigned ldsw = (unsigned)wid * 1024u, ldswb = (unsigned)((wid >> 2) * KBP + ((wid >> 1) & 1) * 2056 + (wid & 1) * 1024);
    const int aoff = lds_byte(wr * 64 + fr, fq * 8);
    const int boff = (fq >> 1) * KBP + (fq & 1) * 2056 + (wc >> 1) * 1024 + (fr >> 2) * 128 + ((4 * (wc & 1) + (fr & 3)) ^ (((fr >> 3) & 1) << 2)) * 16;
    constexpr size_t hA = (size_t)128 * XP, hB = 256;
    const int nt = mode == 0 ? 64 : (mode == 1 ? 56 : 8);
    const char* xl = P.x + (size_t)pm * 256 * XP;
    const char* wl = P.w + (size_t)(mode == 2 ? (g + qd) & 7 : g) * NB * 2 + pn * 512;
    const char* xs = myws + OFF_XS + (size_t)pm * 256 * XP;
    const char* wsb = myws + OFF_WS + pn * 512;
    auto ktile = [&](unsigned id, const char*& a, const char*& b) {
        const unsigned c = id >> 3, q = id & 7;
        a = (q ? xs : xl) + (q * (GM * XP) + c * 128); b = (q ? wsb : wl) + (c * (64 * WP) + q * (NB * 2));
    };
#define SA(b, h) (((b) * 2 + (h)) * HTB)
#define SB(b, h) (4 * HTB + ((b) * 2 + (h)) * HTBB)
#define STAGEB(bufoff, gbase) do { _Pragma("unroll") for (int _i = 0; _i < 2; ++_i) \
        __builtin_amdgcn_global_load_lds((const unsigned*)((const char*)(gbase) + voffB[_i]), (LAS unsigned*)(lds + (bufoff) + ldswb + _i * 2 * KBP), 16, 0, 0); } while (0)
#define STAGE(bufoff, gbase, voff) do { _Pragma("unroll") for (int _i = 0; _i < 2; ++_i) \
        __builtin_amdgcn_global_load_lds((const unsigned*)((const char*)(gbase) + (voff)[_i]), (LAS unsigned*)(lds + (bufoff) + ldsw + _i * 8192), 16, 0, 0); } while (0)
#define LDA(dst, b, h) do { _Pragma("unroll") for (int m = 0; m < 4; ++m) _Pragma("unroll") for (int k = 0; k < 2; ++k) dst[m][k] = *(const LAS bf16x8*)(lds + SA(b, h) + aoff + m * 2048 + k * 1024); } while (0)
#define TRD(dst, off) asm volatile("ds_read_b64_tr_b16 %0, %1 offset:%2" : "=v"(dst) : "v"(baddr), "i"(off) : "memory")
#define LDB(dst, b, h) do { _Pragma("unroll") for (int n = 0; n < 2; ++n) _Pragma("unroll") for (int k = 0; k < 2; ++k) { \
        TRD(dst[n][k][0], ((b) * 2 + (h)) * HTBB + k * 2 * KBP + n * 8); TRD(dst[n][k][1], ((b) * 2 + (h)) * HTBB + k * 2 * KBP + 512 + n * 8); } } while (0)
#define WAIT_B(Bt) asm volatile("s_waitcnt lgkmcnt(0)" : "+v"(Bt[0][0][0]), "+v"(Bt[0][0][1]), "+v"(Bt[0][1][0]), "+v"(Bt[0][1][1]), "+v"(Bt[1][0][0]), "+v"(Bt[1][0][1]), "+v"(Bt[1][1][0]), "+v"(Bt[1][1][1]) :: "memory")
#define BF(Bt, n, k) __builtin_shufflevector(Bt[n][k][0], Bt[n][k][1], 0, 1, 2, 3, 4, 5, 6, 7)
#define MMA(ai, bj, At, Bt) do { __builtin_amdgcn_s_setprio(1); _Pragma("unroll") for (int m = 0; m < 4; ++m) _Pragma("unroll") for (int n = 0; n < 2; ++n) _Pragma("unroll") for (int k = 0; k < 2; ++k) \
        acc[ai][bj][m][n] = __builtin_amdgcn_mfma_f32_16x16x32_bf16(BF(Bt, n, k), At[m][k], acc[ai][bj][m][n], 0, 0, 0); __builtin_amdgcn_s_setprio(0); } while (0)
#define WAIT_V(n) asm volatile("s_waitcnt vmcnt(" #n ")" ::: "memory")
#define WAIT_L(n) asm volatile("s_waitcnt lgkmcnt(" #n ")" ::: "memory")
#define BAR __builtin_amdgcn_s_barrier()
#define SCHED __builtin_amdgcn_sched_barrier(0)
    f32x4 acc[2][2][4][2];
#pragma unroll
    for (int a = 0; a < 2; ++a)
#pragma unroll
        for (int b = 0; b < 2; ++b)
#pragma unroll
            for (int m = 0; m < 4; ++m)
#pragma unroll
                for (int n = 0; n < 2; ++n) acc[a][b][m][n] = (f32x4){0.f, 0.f, 0.f, 0.f};
    bf16x8 At[4][2]; s16x4 B0[2][2][2], B1[2][2][2];
    const unsigned baddr = (unsigned)(size_t)lds + 4 * HTB + boff;
    constexpr unsigned long long LOCAL = 0x0101010101010101ull;
    unsigned long long rdy = LOCAL;
    unsigned long long todo = (mode == 0 ? ~0ull : (mode == 1 ? ~(LOCAL << qd) : LOCAL)) & ~0x101ull;
    LAS unsigned long long* rdy_lds = (LAS unsigned long long*)(lds + STAGE_BYTES);
    STAGEB(SB(0, 0), wl); STAGE(SA(0, 0), xl, voffA); STAGEB(SB(0, 1), wl + hB); STAGE(SA(0, 1), xl + hA, voffA);
    if (wr == 1) BAR;
    WAIT_V(4); BAR;
    STAGEB(SB(1, 0), wl + (size_t)64 * WP); STAGE(SA(1, 0), xl + 128, voffA); STAGEB(SB(1, 1), wl + (size_t)64 * WP + hB);
    WAIT_V(6); BAR;
    const char* a1 = xl + 128;
    for (int t = 0; t < nt; t += 2) {
        if (todo != 0ull && __builtin_popcountll(todo & rdy) < 2) {
            if (wid == 0) {
                const __amdgpu_buffer_rsrc_t rfx = mk(myws + OFF_FX + pm * 1024, 1024), rfw = mk(myws + OFF_FW + pn * 1024, 1024);
                const unsigned long long t0 = __builtin_amdgcn_s_memrealtime();
                unsigned long long mask = rdy;
                const unsigned long long pend = todo & ~rdy, pend2 = pend & (pend - 1ull);
                const int l1 = (__builtin_ctzll(pend2 ? pend2 : pend) | 7) + 8;
                unsigned vo = (pend >> lane) & 1ull ? lane * 16 : 4096;
                for (unsigned polls = 1;; ++polls) {
                    const u32x4 vx = __builtin_amdgcn_raw_buffer_load_b128(rfx, vo, 0, SAUX), vw = __builtin_amdgcn_raw_buffer_load_b128(rfw, vo, 0, SAUX);
                    asm volatile("" ::: "memory");
                    const bool ok = min(min(vx.x, vx.y), min(vx.z, vx.w)) >= call && min(min(vw.x, vw.y), min(vw.z, vw.w)) >= call;
                    mask |= __ballot(ok);
                    if (__builtin_popcountll(mask & todo) >= 2) break;
                    if ((polls & 7u) == 0u && give_up((unsigned*)(myws + OFF_TMO), call, t0)) { mask = ~0ull; break; }
                    if (lane > l1) vo = 4096;
                    __builtin_amdgcn_s_sleep(16);
                }
                __builtin_amdgcn_fence(__ATOMIC_ACQUIRE, "");
                if (lane == 0) *rdy_lds = mask;
                asm volatile("s_waitcnt vmcnt(0) lgkmcnt(0)" ::: "memory");
            }
            asm volatile("" ::: "memory"); BAR; asm volatile("" ::: "memory");
            const unsigned long long mv = *rdy_lds;
            rdy = ((unsigned long long)(unsigned)__builtin_amdgcn_readfirstlane((unsigned)(mv >> 32)) << 32) | (unsigned)__builtin_amdgcn_readfirstlane((unsigned)mv);
        }
        LDB(B0, 0, 0); SCHED; LDA(At, 0, 0); STAGE(SA(1, 1), a1 + hA, voffA); SCHED;
        unsigned long long avail = todo ? todo & rdy : 0x101ull;
        const unsigned id2 = __builtin_ctzll(avail); avail &= avail - 1ull;
        const unsigned id3 = __builtin_ctzll(avail);
        todo &= ~((1ull << id2) | (1ull << id3));
        const char *a2, *b2, *a3, *b3;
        ktile(id2, a2, b2); ktile(id3, a3, b3);
        WAIT_L(8); BAR; WAIT_B(B0); MMA(0, 0, At, B0); BAR; SCHED;
        LDB(B1, 0, 1); STAGEB(SB(0, 0), b2);
        BAR; WAIT_B(B1); MMA(0, 1, At, B1); BAR;
        LDA(At, 0, 1); STAGE(SA(0, 0), a2, voffA);
        BAR; WAIT_L(0); MMA(1, 0, At, B0); BAR; SCHED;
        STAGEB(SB(0, 1), b2 + hB);
        WAIT_V(6); BAR; MMA(1, 1, At, B1); BAR;
        LDB(B0, 1, 0); SCHED; LDA(At, 1, 0); STAGE(SA(0, 1), a2 + hA, voffA);
        WAIT_L(8); BAR; WAIT_B(B0); MMA(0, 0, At, B0); BAR; SCHED;
        LDB(B1, 1, 1); STAGEB(SB(1, 0), b3);
        BAR; WAIT_B(B1); MMA(0, 1, At, B1); BAR;
        LDA(At, 1, 1); STAGE(SA(1, 0), a3, voffA);
        BAR; WAIT_L(0); MMA(1, 0, At, B0); BAR; SCHED;
        STAGEB(SB(1, 1), b3 + hB);
        WAIT_V(6); BAR; MMA(1, 1, At, B1); BAR;
        a1 = a3;
    }
    if (wr == 0) BAR;
    const bool bad = __builtin_amdgcn_readfirstlane(ld_agent((const unsigned*)(myws + OFF_TMO))) == call;
    int tz = threadIdx.x; asm volatile("" : "+v"(tz));
    const int l2 = tz & 63;
    if (mode == 2) {
        const __amdgpu_buffer_rsrc_t ro = mk(myws + OFF_DS + (size_t)(qd * 32 + (pm & 1) * 16 + pn) * 262144, 262144);
        const unsigned vo = (unsigned)(wr * 4 * 16384 + wc * 2048 + l2 * 16);
#pragma unroll
        for (int ai = 0; ai < 2; ++ai)
#pragma unroll
            for (int m = 0; m < 4; ++m)
#pragma unroll
                for (int bj = 0; bj < 2; ++bj)
#pragma unroll
                    for (int n = 0; n < 2; ++n)
                        __builtin_amdgcn_raw_buffer_store_b128(__builtin_bit_cast(u32x4, acc[ai][bj][m][n]), ro, vo + (unsigned)((ai * 8 + m) * 16384 + bj * 8192 + n * 1024), 0, SAUX);
    } else if (mode == 1) {
        const __amdgpu_buffer_rsrc_t ro = mk(myws + OFF_PS + (call & 1) * PS_PAR + (size_t)(pm * 16 + pn) * 131072, 131072);
        const unsigned vo = (unsigned)(wr * 4 * 8192 + wc * 1024 + l2 * 16);
#pragma unroll
        for (int ai = 0; ai < 2; ++ai)
#pragma unroll
            for (int m = 0; m < 4; ++m)
#pragma unroll
                for (int bj = 0; bj < 2; ++bj) {
                    const f32x4 v0 = acc[ai][bj][m][0], v1 = acc[ai][bj][m][1];
                    u32x4 o = {pk_bf16(v0[0], v0[1]), pk_bf16(v0[2], v0[3]), pk_bf16(v1[0], v1[1]), pk_bf16(v1[2], v1[3])};
                    if (bad) o = (u32x4){QNAN2, QNAN2, QNAN2, QNAN2};
                    __builtin_amdgcn_raw_buffer_store_b128(o, ro, vo + (unsigned)((ai * 8 + m) * 8192 + bj * 4096), 0, SAUX);
                }
    } else {
        const __amdgpu_buffer_rsrc_t ro = mk(P.out + (size_t)(pm & 1) * 256 * WP + (size_t)g * NB * 2 + pn * 512, 255 * WP + 512);
        const unsigned vo = (unsigned)(wr * 64 + (l2 & 15)) * WP + (unsigned)(wc * 32 + 8 * (l2 >> 4)) * 2u;
#pragma unroll
        for (int ai = 0; ai < 2; ++ai)
#pragma unroll
            for (int m = 0; m < 4; ++m)
#pragma unroll
                for (int bj = 0; bj < 2; ++bj) {
                    const f32x4 v0 = acc[ai][bj][m][0], v1 = acc[ai][bj][m][1];
                    u32x4 o = {pk_bf16(silu(v0[0]), silu(v0[1])), pk_bf16(silu(v0[2]), silu(v0[3])), pk_bf16(silu(v1[0]), silu(v1[1])), pk_bf16(silu(v1[2]), silu(v1[3]))};
                    if (bad) o = (u32x4){QNAN2, QNAN2, QNAN2, QNAN2};
                    __builtin_amdgcn_raw_buffer_store_b128(o, ro, vo + (unsigned)(ai * 8 + m) * 16 * WP + bj * 256, 0, SAUX);
                }
    }
    WAIT_V(0);
    BAR;
    if (tid == 0) {
        if (mode == 1) st_sys(P.ws[qd] + OFF_FO + ((8 - qd) * 64 + (pm & 1) * 16 + pn) * 4, call);
        if (mode == 2) st_sys(myws + OFF_FO + (qd * 64 + 32 + (pm & 1) * 16 + pn) * 4, call);
        else if (__hip_atomic_fetch_add((unsigned*)(myws + OFF_GDC), 1u, __ATOMIC_RELAXED, __HIP_MEMORY_SCOPE_AGENT) == 256u * call - 1u) {
            for (int o = 1; o < 8; ++o) st_sys(P.ws[o] + OFF_GD + (8 - o) * 128, call);
        }
    }
#undef SA
#undef SB
#undef STAGE
#undef STAGEB
#undef LDA
#undef LDB
#undef TRD
#undef WAIT_B
#undef BF
#undef MMA
}

__global__ __launch_bounds__(512, 2) void fused_gemm_rs(const Params P) {
    extern __shared__ __attribute__((aligned(16))) unsigned char shm[];
    const int b = blockIdx.x, xcd = b & 7, slot = b >> 3;
    asm volatile("" :: "s"(P.ws[0]), "s"(P.rank), "s"(P.sched[10]), "s"(P.sched[26]), "s"(P.sched[42]), "s"(P.sched[58]));
    unsigned* cnt = (unsigned*)(P.ws[0] + OFF_CNT + (size_t)b * 128);
    const unsigned cntv = ld_agent(cnt);
    unsigned pend = 0;
    int pw = -1;
    if (slot < CPX) {
        const unsigned call = __builtin_amdgcn_readfirstlane(cntv) + 1u;
        asm volatile("s_waitcnt vmcnt(0)" ::: "memory");
        __builtin_amdgcn_s_barrier();
        if (threadIdx.x == 448) __hip_atomic_store(cnt, call, __ATOMIC_RELAXED, __HIP_MEMORY_SCOPE_AGENT);
        pend = in_role(P, b, call);
        pw = NPW0 + b;
    } else {
        const int g = P.rank, s = slot - CPX;
        const int cl = s / 7, pcl = 2;
        const bool late = cl != 0 && cl != pcl;
        const int e = ((late && cl == 3 ? 7 : 0) + s % 7) * 8 + xcd;
        static_assert(NP0 <= 4, "pushed by the two early classes");
        { const int e2 = (late ? 112 : (cl == 0 ? 56 : 0)) + e; quick_role(P, e2 % 7 + 1, e2 / 7, cntv); }
        const unsigned call = __builtin_amdgcn_readfirstlane(cntv) + 1u;
        if (late) copy_role(P, e % 7 + 1, NCOMM / 4 + e / 7, call);
        for (int u = (late ? 0 : 2); u < (s < 4 ? 4 : 3); ++u) {
            int mode, pm, pn, qd;
            if (u < 2) { mode = 2; pm = 2 * g + u; pn = e / 7; qd = e % 7 + 1; }
            else if (u == 2) { mode = 1; qd = s % 7 + 1; pm = 2 * ((g + qd) & 7) + (cl >> 1); pn = xcd + 8 * (cl & 1); }
            else { mode = 0; qd = 0; pm = 2 * g + (s >> 1); pn = xcd + 8 * (s & 1); }
            gemm_tile(P, (LAS unsigned char*)shm, mode, pm, pn, qd, call);
        }
        if (threadIdx.x == 448) __hip_atomic_store(cnt, call, __ATOMIC_RELAXED, __HIP_MEMORY_SCOPE_AGENT);
        if (s >= 4 && s < 7) pw = (s - 4) * 8 + xcd;
        if (s == 7 * pcl) pw = NPW0 + NCOMM + xcd;
    }
    if (pw >= 0) pull_role(P, pw, __builtin_amdgcn_readfirstlane(cntv) + 1u, pend);
}

void build_schedule(unsigned* sched) {
    unsigned char e[240]; int n = 0;
    auto W = [&](int j0, int j1, int c) { for (int j = j0; j < j1; ++j) e[n++] = (unsigned char)((j << 3) | c); };
    auto X = [&](int u0, int u1, int c) { for (int u = u0; u < u1; ++u) e[n++] = (unsigned char)(128 | (u << 3) | c); };
    for (int c = 0; c < 8; ++c) { X(2, 8, c); W(0, 8, c); if (c == 4) X(8, 14, 0); if (c == 6) X(8, 14, 1); }
    for (int c = 0; c < 6; ++c) { X(8, 14, c + 2); W(8, 16, c); if (c < 3) X(0, 2, c); if (c == 3) X(0, 2, 6); }
    W(8, 16, 6); W(8, 16, 7);
    X(0, 2, 3); X(0, 2, 4); X(0, 2, 5);
    X(0, 2, 7);
    for (int i = 0; i < 60; ++i) sched[i] = e[4 * i] | ((unsigned)e[4 * i + 1] << 8) | ((unsigned)e[4 * i + 2] << 16) | ((unsigned)e[4 * i + 3] << 24);
}
}

extern "C" void kernel_launch(int rank, int nranks, ncclComm_t comm, hipStream_t stream,
                              void* const* d_in, const int* in_sizes, int n_in,
                              void* d_out, int out_size, const int* in_dtypes, int out_dtype,
                              void* const* d_out_all, void* const* d_ws_all, size_t ws_bytes) {
    if (nranks != NR || (unsigned)rank >= (unsigned)NR || n_in < 2 || ws_bytes < WS_NEED ||
        in_sizes[0] != GM * KS || in_sizes[1] != KS * GN || out_size != MS * GN) {
        fprintf(stderr, "kernel_launch: unexpected problem shape\n");
        return;
    }
    HIP_CHECK(hipFuncSetAttribute((const void*)fused_gemm_rs, hipFuncAttributeMaxDynamicSharedMemorySize, LDS_BYTES));
    Params P;
    P.x = (const char*)d_in[0]; P.w = (const char*)d_in[1];
    for (int p = 0; p < NR; ++p) P.ws[p] = (char*)d_ws_all[(rank + p) & 7];
    P.out = (char*)d_out;
    P.rank = rank;
    build_schedule(P.sched);
    hipLaunchKernelGGL(fused_gemm_rs, dim3(GRID), dim3(512), LDS_BYTES, stream, P);
    HIP_CHECK(hipGetLastError());
}
```

```cpp
#include <hip/hip_runtime.h>
#include <rccl/rccl.h>
#include <cstdio>

#define HIP_CHECK(expr) do { hipError_t _e = (expr); if (_e != hipSuccess) { \
    fprintf(stderr, "HIP error %s at %s:%d\n", hipGetErrorString(_e), __FILE__, __LINE__); \
    return; } } while (0)

#define LAS __attribute__((address_space(3)))
typedef short s16x4 __attribute__((ext_vector_type(4)));
typedef short bf16x8 __attribute__((ext_vector_type(8)));
typedef float f32x4 __attribute__((ext_vector_type(4)));
typedef float f32x2 __attribute__((ext_vector_type(2)));
typedef __bf16 bf16x2 __attribute__((ext_vector_type(2)));
typedef unsigned u32x4 __attribute__((ext_vector_type(4)));

namespace {
constexpr int NR = 8;
constexpr int GM = 4096, GK = 4096, GN = 32768;
constexpr int KS = GK / NR;
constexpr int NB = GN / NR;
constexpr int MS = GM / NR;
constexpr unsigned XP = KS * 2;
constexpr unsigned WP = GN * 2;
constexpr int HTB = 128 * 64 * 2;
constexpr int KBP = 4096 + 16;
constexpr int HTBB = 4 * KBP;
constexpr int STAGE_BYTES = 4 * HTB + 4 * HTBB;
constexpr int LDS_BYTES = STAGE_BYTES + 64;
constexpr int GRID = 256;
constexpr int CPX = 4;
constexpr int NCOMM = 8 * CPX;
constexpr int NPW0 = 24;
constexpr int NPW = NPW0 + NCOMM + 8;
constexpr unsigned long long DEADLINE = 50000000ull;

constexpr size_t OFF_CNT = 0;
constexpr size_t OFF_TMO = 32768;
constexpr size_t OFF_GDC = OFF_TMO + 256;
constexpr size_t OFF_FX = 65536;
constexpr size_t OFF_FW = OFF_FX + 16384;
constexpr size_t OFF_AV = OFF_FW + 16384;
constexpr size_t OFF_FO = OFF_AV + 8192;
constexpr size_t OFF_Q = OFF_FO + 2048;
constexpr size_t OFF_GD = OFF_Q + 4096;
constexpr size_t OFF_GO = OFF_GD + 1024;
constexpr size_t OFF_AV2 = 524288;
constexpr size_t OFF_AV3 = OFF_AV2 + 262144;
constexpr size_t OFF_XS = 1u << 20;
constexpr size_t OFF_WS = OFF_XS + (size_t)NR * GM * XP;
constexpr size_t OFF_PS = OFF_WS + (size_t)KS * WP;
constexpr size_t PS_PAR = (size_t)256 * 131072;
constexpr size_t OFF_DS = OFF_PS + 2 * PS_PAR;
constexpr size_t OFF_OB = OFF_DS + (size_t)256 * 262144;
constexpr size_t WS_NEED = OFF_OB + (size_t)8 * 240 * 32768;
constexpr unsigned NP0 = 2;
constexpr unsigned NPL = 8;
constexpr unsigned NPOOL = 7 * 4 * NPL, POOL_STEP = NPOOL + 7 * NCOMM;
constexpr unsigned NBLK = 240 - NP0 - NPL;
constexpr unsigned NIN = 4 * NBLK;
constexpr unsigned NPULL = 1024;

struct Params {
    const char* x; const char* w;
    char* ws[NR];
    char* out;
    int rank;
    unsigned sched[60];
};

__device__ __forceinline__ int lds_byte(int r, int c) { const int st = (r >> 4) * 2 + (c >> 5), rr = r & 15, cc = c & 31, ob = rr * 64 + cc * 2; return st * 1024 + (ob ^ (((ob >> 9) & 1) << 5)); }
__device__ __forceinline__ void stage_rc(int b, int& R, int& C) { const int st = b / 1024, sb = b % 1024, swz = sb ^ (((sb >> 9) & 1) << 5); R = (st >> 1) * 16 + swz / 64; C = (st & 1) * 32 + (swz % 64) / 2; }
__device__ __forceinline__ __amdgpu_buffer_rsrc_t mk(const void* p, unsigned bytes) { return __builtin_amdgcn_make_buffer_rsrc((void*)p, 0, bytes, 0x00020000); }
__device__ __forceinline__ unsigned pk_bf16(float lo, float hi) { const bf16x2 r = __builtin_convertvector((f32x2){lo, hi}, bf16x2); return __builtin_bit_cast(unsigned, r); }
__device__ __forceinline__ float silu(float y) {
    const float c = __builtin_amdgcn_fmed3f(y, -60.0f, 60.0f);
    const float e = __builtin_amdgcn_exp2f(c * -1.44269504088896341f);
    return y * __builtin_amdgcn_rcpf(1.0f + e);
}
__device__ __forceinline__ unsigned ld_agent(const unsigned* p) { return __hip_atomic_load(p, __ATOMIC_RELAXED, __HIP_MEMORY_SCOPE_AGENT); }
__device__ __forceinline__ unsigned ld_sys(const void* p) { return __hip_atomic_load((const unsigned*)p, __ATOMIC_RELAXED, __HIP_MEMORY_SCOPE_SYSTEM); }
__device__ __forceinline__ void st_sys(void* p, unsigned v) { __hip_atomic_store((unsigned*)p, v, __ATOMIC_RELAXED, __HIP_MEMORY_SCOPE_SYSTEM); }
__device__ __forceinline__ bool give_up(unsigned* tmo, unsigned call, unsigned long long t0) {
    if (__builtin_amdgcn_readfirstlane(ld_agent(tmo)) != call && __builtin_amdgcn_s_memrealtime() - t0 <= DEADLINE) return false;
    __hip_atomic_store(tmo, call, __ATOMIC_RELAXED, __HIP_MEMORY_SCOPE_AGENT);
    return true;
}
__device__ __forceinline__ int panel_of(int u, int d, int q) {
    if (u < 2) return 2 * d + u;
    const int hi = u >= 8, j = u - (hi ? 7 : 1);
    return 2 * ((d + j + (j >= q)) & 7) + hi;
}

constexpr unsigned LEAD1 = 12;
constexpr unsigned AVR = 160;
constexpr int DS = 2;
constexpr int DP = 1, DD = 3;
constexpr int SAUX = 17;
constexpr int LAUX = 19;
constexpr int OAUX = 18;
constexpr unsigned QNAN2 = 0x7fc07fc0u;

struct Half { unsigned isx, src, dst, flag; };
__device__ __forceinline__ Half half_of(const Params& P, unsigned s, int d, int q) {
    const unsigned e = (P.sched[s >> 3] >> (((s >> 1) & 3) * 8)) & 0xffu;
    const unsigned c = e & 7, idx = (e >> 3) & 15, h = s & 1;
    Half r; r.isx = e >> 7;
    if (r.isx) {
        const unsigned pn = panel_of(idx, d, q);
        r.src = (pn * 256 + h * 128) * XP + c * 128; r.dst = (unsigned)OFF_XS + q * GM * XP + r.src;
        r.flag = (unsigned)OFF_FX + ((pn * 64 + c * 8 + q) * 2 + h) * 8;
    } else {
        const unsigned o = c * 64 * WP + idx * 512 + h * 256;
        r.src = o + d * NB * 2; r.dst = (unsigned)OFF_WS + q * NB * 2 + o;
        r.flag = (unsigned)OFF_FW + ((idx * 64 + c * 8 + q) * 2 + h) * 8;
    }
    return r;
}
constexpr unsigned XB = 127 * XP + 128, WB = 63 * WP + 256;
__device__ __forceinline__ bool wait_ge(const char* word, unsigned want, unsigned* tmo, unsigned call) {
    const unsigned long long t0 = __builtin_amdgcn_s_memrealtime();
    for (unsigned polls = 1; __builtin_amdgcn_readfirstlane(ld_sys(word)) < want; ++polls) {
        if ((polls & 15u) == 0u && give_up(tmo, call, t0)) return false;
        __builtin_amdgcn_s_sleep(1);
    }
    return true;
}

__device__ __forceinline__ unsigned in_role(const Params& P, int cw, unsigned call) {
    const int lane = threadIdx.x & 63, hw = __builtin_amdgcn_readfirstlane(threadIdx.x >> 6);
    if (hw >= 7) return 0;
    const int wid = (hw + cw) % 7;
    const int g = P.rank;
    char* myws = P.ws[0];
    const __amdgpu_buffer_rsrc_t qctr = mk(myws + OFF_Q + (wid + 1) * 256, 4);
    const unsigned vx = (lane >> 3) * XP + (lane & 7) * 16, vw = (lane >> 4) * WP + (lane & 15) * 16, vl = lane * 16;
    unsigned* tmo = (unsigned*)(myws + OFF_TMO);
    const unsigned qbase = (call - 1u) * NIN - 2 * NCOMM, pbase = (call - 1u) * POOL_STEP;
    const __amdgpu_buffer_rsrc_t pctr = mk(myws + OFF_Q, 4), goctr = mk(myws + OFF_GO + (wid + 1) * 256, 4);
    const char* ews = P.ws[wid + 1];
    unsigned s = cw, pflag = 0, tk = qbase + NCOMM + cw;
    bool bad = false, pre = false;
    {
        const char* gd = myws + OFF_GD + ((lane & 7) ? lane & 7 : 1) * 128;
        if (!__all(ld_agent((const unsigned*)gd) >= call - 1u)) {
            const unsigned long long t0 = __builtin_amdgcn_s_memrealtime();
            while (!__all(ld_sys(gd) >= call - 1u)) { if (give_up(tmo, call, t0)) { bad = true; break; } __builtin_amdgcn_s_sleep(8); }
        }
    }
    __amdgpu_buffer_rsrc_t rs = mk(myws, 0);
    unsigned lv = vl, ls = 1024;
    u32x4 v[4];
    auto locate = [&](unsigned s, Half& hf, unsigned& sdat, unsigned& sflag) {
        const bool pool = s >= NIN;
        const unsigned t = pool && s != ~0u ? s - NIN : 0u, j = t / 7u, o = t - j * 7u + 1u, qe = (o - (wid + 1)) & 7u;
        const unsigned sq = pool ? NIN + j : s;
        hf = half_of(P, sq >> 1, g, pool ? o : wid + 1);
        const unsigned ss = hf.isx ? 8 * XP : 4 * WP;
        hf.dst += (sq & 1) * 8 * ss; hf.flag += (sq & 1) * 4;
        sdat = pool ? (unsigned)OFF_XS + qe * (GM * XP) + hf.src + (sq & 1) * 8 * ss : (unsigned)OFF_OB + ((7 - wid) * 960 + s) * 8192;
        sflag = pool ? hf.flag + qe * 16 - o * 16 : (s < NCOMM ? (unsigned)OFF_AV3 + ((7 - wid) * 32 + s) * 128 : (unsigned)OFF_AV2 + ((7 - wid) * 256 + (s >> 2)) * 128);
    };
    Half hf; unsigned sdat, sflag;
    locate(s, hf, sdat, sflag);
    for (;;) {
        hf.dst = __builtin_amdgcn_readfirstlane(hf.dst); hf.flag = __builtin_amdgcn_readfirstlane(hf.flag); hf.isx = __builtin_amdgcn_readfirstlane(hf.isx);
        sdat = __builtin_amdgcn_readfirstlane(sdat); sflag = __builtin_amdgcn_readfirstlane(sflag);
        if (!pre) {
            if (s == ~0u) break;
            if (pflag) {
                asm volatile("s_waitcnt vmcnt(0)" ::: "memory");
                if (lane == 0) st_sys(myws + pflag, call);
                pflag = 0;
            }
            if (!wait_ge(ews + sflag, call, tmo, call)) bad = true;
            rs = mk(ews + sdat, bad ? 0 : 63 * XP + 128);
#pragma unroll
            for (int k = 0; k < DS; ++k) {
                v[k] = __builtin_amdgcn_raw_buffer_load_b128(rs, lv, k * ls, LAUX); __builtin_amdgcn_sched_barrier(0);
                __builtin_amdgcn_raw_buffer_store_b128((u32x4){0u, 0u, 0u, 0u}, mk(myws, 0), vl, 0, SAUX); __builtin_amdgcn_sched_barrier(0);
            }
        }
        const unsigned ss = hf.isx ? 8 * XP : 4 * WP;
        const __amdgpu_buffer_rsrc_t rd = mk(myws + hf.dst, hf.isx ? 63 * XP + 128 : 31 * WP + 256);
        const unsigned sv = hf.isx ? vx : vw, flag = hf.flag;
        const bool priv = s < NIN;
        if (priv) s = __builtin_amdgcn_readfirstlane(tk) - qbase;
        tk = (unsigned)__builtin_amdgcn_raw_ptr_buffer_atomic_add_i32(1, qctr, priv ? lane * 4 : 4, 0, 0);
        const unsigned tkp = (unsigned)__builtin_amdgcn_raw_ptr_buffer_atomic_add_i32(1, pctr, s >= NIN ? lane * 4 : 4, 0, 0);
        unsigned av = 0;
#define IN_ITEM(k) do { \
            if (k == 3 && s >= NIN) { \
                const unsigned t = __builtin_amdgcn_readfirstlane(tkp) - pbase; s = t < NPOOL ? NIN + t : ~0u; \
                __builtin_amdgcn_raw_ptr_buffer_atomic_add_i32(1, goctr, s == ~0u ? lane * 4 : 4, 0, 0);     \
            } \
            if (k == 5) { \
                locate(s, hf, sdat, sflag); \
                const bool loc = s >= AVR && s < NIN;       \
                av = __builtin_amdgcn_raw_buffer_load_b32(mk(loc ? myws + OFF_AV + (wid + 1) * 1024 + (s >> 2) * 4 : ews + sflag, s != ~0u ? 4 : 0), 0, 0, SAUX); \
            } \
            if (k + DS == 8) {              \
                pre = s != ~0u && (__builtin_amdgcn_readfirstlane(av) >= call || bad); \
                rs = mk(ews + sdat, pre && !bad ? 63 * XP + 128 : 0); \
                lv = s >= NIN ? vx : vl; ls = s >= NIN ? 8 * XP : 1024; \
            } \
            asm volatile("" : "+v"(v[k & 3]));        \
            __builtin_amdgcn_sched_barrier(0); \
            if (k == DS && pflag) {         \
                asm volatile("s_waitcnt vmcnt(%0)" :: "i"(2 * DS) : "memory"); \
                if (lane == 0) st_sys(myws + pflag, call); \
                asm volatile("" ::: "memory"); \
            } \
            v[(k + DS) & 3] = __builtin_amdgcn_raw_buffer_load_b128(rs, lv, ((k + DS) & 7) * ls, LAUX); \
            __builtin_amdgcn_sched_barrier(0); \
            __builtin_amdgcn_raw_buffer_store_b128(v[k & 3], rd, sv, k * ss, SAUX); \
            __builtin_amdgcn_sched_barrier(0); \
            asm volatile("s_nop 1");        \
            __builtin_amdgcn_sched_barrier(0); } while (0)
        IN_ITEM(0); IN_ITEM(1); IN_ITEM(2); IN_ITEM(3); IN_ITEM(4); IN_ITEM(5); IN_ITEM(6); IN_ITEM(7);
#undef IN_ITEM
        pflag = flag;
    }
    return pflag;
}

__device__ __forceinline__ void copy_role(const Params& P, int dd, unsigned n0, unsigned call) {
    const int lane = threadIdx.x & 63, wv = __builtin_amdgcn_readfirstlane(threadIdx.x >> 6);
    const int g = P.rank, d = (g + dd) & 7;
    char* myws = P.ws[0]; char* dws = P.ws[dd];
    const unsigned vx = (lane >> 3) * XP + (lane & 7) * 16, vw = (lane >> 4) * WP + (lane & 15) * 16;
    const char* const px = P.x; const char* const pwm = P.w;
    constexpr unsigned n1 = NBLK + NPL;
    u32x4 a[4], b[4];
    Half ha, hb;
    auto fetch = [&](unsigned n, u32x4 (&r)[4], Half& hf) {
        hf = half_of(P, 2 * (n < n1 ? n : 0) + (wv >> 2), d, 8 - dd);
        const __amdgpu_buffer_rsrc_t rs = mk((hf.isx ? px : pwm) + hf.src, n < n1 ? (hf.isx ? XB : WB) : 0);
        const unsigned st = hf.isx ? 8 * XP : 4 * WP;
#pragma unroll
        for (int j = 0; j < 4; ++j) r[j] = __builtin_amdgcn_raw_buffer_load_b128(rs, hf.isx ? vx : vw, ((wv & 3) * 4 + j) * st, 0);
    };
    fetch(n0, a, ha);
    for (unsigned n = n0; n < n1; n += 16) {
        const bool pool = n >= NBLK;
        const unsigned st = pool ? 8 * XP : 1024u;
        const __amdgpu_buffer_rsrc_t rd = pool ? mk(myws + OFF_XS + ha.src, XB) : mk(myws + OFF_OB + (size_t)(dd * 240 + n) * 32768 + (wv >> 2) * 16384, 16384);
        const unsigned vo = (pool ? vx : lane * 16u) + (wv & 3) * 4 * st;
#pragma unroll
        for (int j = 0; j < 4; ++j) __builtin_amdgcn_raw_buffer_store_b128(a[j], rd, vo + j * st, 0, SAUX);
        __builtin_amdgcn_sched_barrier(0);
        fetch(n + 16, b, hb);
        if (n + 16 < n1) asm volatile("s_waitcnt vmcnt(4)" ::: "memory");
        else asm volatile("s_waitcnt vmcnt(0)" ::: "memory");
        __builtin_amdgcn_s_barrier();
        if (pool) { if ((threadIdx.x & 255) < 2) st_sys(myws + ha.flag - (8 - dd) * 16 + (threadIdx.x & 1) * 4, call); }
        else if (threadIdx.x < 2) st_sys(threadIdx.x ? dws + OFF_AV + ((8 - dd) * 256 + n) * 4 : myws + OFF_AV2 + (dd * 256 + n) * 128, call);
#pragma unroll
        for (int j = 0; j < 4; ++j) a[j] = b[j];
        ha = hb;
    }
}

__device__ __forceinline__ void quick_role(const Params& P, int dd, unsigned i, unsigned cntv) {
    const int lane = threadIdx.x & 63, wv = __builtin_amdgcn_readfirstlane(threadIdx.x >> 6);
    const int d = (P.rank + dd) & 7;
    char* myws = P.ws[0]; char* dws = P.ws[dd];
    const unsigned gdv = ld_agent((const unsigned*)(myws + OFF_GD + dd * 128));
    const unsigned vx = (lane >> 3) * XP + (lane & 7) * 16, vw = (lane >> 4) * WP + (lane & 15) * 16;
    const char* const px = P.x; const char* const pwm = P.w;
    const bool push = i < 4 * NP0;
    const Half h0 = half_of(P, i >> 1, d, 8 - dd);
    const unsigned o0 = (h0.isx ? vx : vw) + ((i & 1) * 8 + wv) * (h0.isx ? 8 * XP : 4 * WP);
    const u32x4 a = __builtin_amdgcn_raw_buffer_load_b128(mk((h0.isx ? px : pwm) + h0.src, h0.isx ? XB : WB), o0, 0, 0);
    asm volatile("" ::: "memory");
    const Half h1 = half_of(P, (push ? 4 * (240 - NP0) + i : 0) >> 1, d, 8 - dd);
    const unsigned o1 = (h1.isx ? vx : vw) + ((i & 1) * 8 + wv) * (h1.isx ? 8 * XP : 4 * WP);
    const u32x4 b = __builtin_amdgcn_raw_buffer_load_b128(mk((h1.isx ? px : pwm) + h1.src, push ? (h1.isx ? XB : WB) : 0), o1, 0, 0);
    __builtin_amdgcn_sched_barrier(0);
    const unsigned call = __builtin_amdgcn_readfirstlane(cntv) + 1u;
    if (__builtin_amdgcn_readfirstlane(gdv) < call - 1u)
        wait_ge(myws + OFF_GD + dd * 128, call - 1u, (unsigned*)(myws + OFF_TMO), call);
    __builtin_amdgcn_raw_buffer_store_b128(a, mk(myws + OFF_OB + (size_t)dd * 240 * 32768 + i * 8192, 8192), wv * 1024 + lane * 16, 0, SAUX);
    __builtin_amdgcn_sched_barrier(0);
    __builtin_amdgcn_raw_buffer_store_b128(b, mk(dws + h1.dst, i < 4 ? (h1.isx ? XB : WB) : 0), o1, 0, SAUX);
    if (i < 4) asm volatile("s_waitcnt vmcnt(1)" ::: "memory"); else asm volatile("s_waitcnt vmcnt(0)" ::: "memory");
    __builtin_amdgcn_s_barrier();
    if (threadIdx.x == 0) st_sys(myws + OFF_AV3 + (dd * 32 + i) * 128, call);
    if (push) {
        __builtin_amdgcn_raw_buffer_store_b128(b, mk(dws + h1.dst, i >= 4 ? (h1.isx ? XB : WB) : 0), o1, 0, SAUX);
        asm volatile("s_waitcnt vmcnt(0)" ::: "memory");
        __builtin_amdgcn_s_barrier();
        if (threadIdx.x == 0) st_sys(dws + h1.flag + (i & 1) * 4, call);
    }
}

__device__ __forceinline__ unsigned ptile(unsigned o) { return (o & 7) | ((o & 8) << 1) | ((o & 16) >> 1); }
__device__ __forceinline__ void pull_role(const Params& P, int pw, unsigned call, unsigned pend) {
    const int lane = threadIdx.x & 63, hw = __builtin_amdgcn_readfirstlane(threadIdx.x >> 6);
    if (hw >= 7) return;
    const int wid = (hw + pw) % 7;
    const int g = P.rank;
    char* myws = P.ws[0];
    const int d = (g + wid + 1) & 7;
    const __amdgpu_buffer_rsrc_t qctr = mk(myws + OFF_Q + (8 + wid + 1) * 256, 4);
    const unsigned qbase = (call - 1u) * NPULL - 2 * NPW;
    unsigned* tmo = (unsigned*)(myws + OFF_TMO);
    const char* fo = myws + OFF_FO + ((wid + 1) * 64 + lane) * 4;
    unsigned fv = ld_sys(fo), tk = qbase + NPW + pw, u = pw;
    asm volatile("s_waitcnt vmcnt(0)" ::: "memory");
    if (pend && lane == 0) st_sys(myws + pend, call);
    bool bad = false;
    if (pw < NPW0) {
        const unsigned want = (call - 1u) * NCOMM + 1u + pw * LEAD1 / NPW0;
        const unsigned* go = (const unsigned*)(myws + OFF_GO + (wid + 1) * 256);
        const unsigned long long t0 = __builtin_amdgcn_s_memrealtime();
        for (unsigned polls = 1; __builtin_amdgcn_readfirstlane(ld_agent(go)) < want; ++polls) {
            if ((polls & 15u) == 0u && give_up(tmo, call, t0)) { bad = true; break; }
            __builtin_amdgcn_s_sleep(16);
        }
    }
    const char* dps = P.ws[wid + 1] + OFF_PS + (call & 1) * PS_PAR + (size_t)(2 * g) * 16 * 131072;
    const char* dds = myws + OFF_DS + (size_t)(wid + 1) * 32 * 262144;
    char* ob = P.out + (size_t)d * NB * 2;
    const unsigned vl = lane * 16, vs = (lane & 15) * WP + (lane >> 4) * 16;
    const __amdgpu_buffer_rsrc_t rfo = mk(myws + OFF_FO + (wid + 1) * 256, 256);
    const unsigned long long m0 = __ballot(fv >= call);
    unsigned rdy = bad ? ~0u : (unsigned)m0 & (unsigned)(m0 >> 32); bool pre = false;
    u32x4 vp[4]; f32x4 vd[4][2];
    __amdgpu_buffer_rsrc_t rs = mk(myws, 0), rsd = mk(myws, 0);
#define LOAD_P(item) vp[item] = __builtin_amdgcn_raw_buffer_load_b128(rs, vl + (item) * 1024, 0, LAUX)
#define LOAD_D(item) do { vd[item][0] = __builtin_bit_cast(f32x4, __builtin_amdgcn_raw_buffer_load_b128(rsd, vl + (item) * 2048, 0, LAUX)); \
        vd[item][1] = __builtin_bit_cast(f32x4, __builtin_amdgcn_raw_buffer_load_b128(rsd, vl + (item) * 2048 + 1024, 0, LAUX)); } while (0)
#define FINISH(k) do { const u32x4 p = vp[k]; const f32x4 d0 = vd[k][0], d1 = vd[k][1]; u32x4 o; \
            o.x = pk_bf16(silu(__uint_as_float(p.x << 16) + d0[0]), silu(__uint_as_float(p.x & 0xffff0000u) + d0[1])); \
            o.y = pk_bf16(silu(__uint_as_float(p.y << 16) + d0[2]), silu(__uint_as_float(p.y & 0xffff0000u) + d0[3])); \
            o.z = pk_bf16(silu(__uint_as_float(p.z << 16) + d1[0]), silu(__uint_as_float(p.z & 0xffff0000u) + d1[1])); \
            o.w = pk_bf16(silu(__uint_as_float(p.w << 16) + d1[2]), silu(__uint_as_float(p.w & 0xffff0000u) + d1[3])); \
            if (bad) o = (u32x4){QNAN2, QNAN2, QNAN2, QNAN2}; \
            __builtin_amdgcn_raw_buffer_store_b128(o, rd, vs + k * 64, 0, OAUX); } while (0)
    while (u < NPULL) {
        const unsigned tile = ptile(u >> 5), part = u & 31;
        if (!pre) {
            const unsigned long long t0 = __builtin_amdgcn_s_memrealtime();
            while (!((rdy >> tile) & 1u)) {
                const unsigned long long m = __ballot(ld_sys(fo) >= call);
                rdy = (unsigned)m & (unsigned)(m >> 32);
                if ((rdy >> tile) & 1u) break;
                if (give_up(tmo, call, t0)) { rdy = ~0u; bad = true; break; }
                __builtin_amdgcn_s_sleep(4);
            }
            rs = mk(dps + (size_t)(tile * 32 + part) * 4096, bad ? 0 : 4096); rsd = mk(dds + (size_t)(tile * 32 + part) * 8192, 8192);
#pragma unroll
            for (int k = 0; k < DD; ++k) {
                if (k >= DD - DP) LOAD_P(k - (DD - DP));
                LOAD_D(k);
                __builtin_amdgcn_sched_barrier(0);
                __builtin_amdgcn_raw_buffer_store_b128((u32x4){0u, 0u, 0u, 0u}, mk(myws, 0), vl, 0, OAUX);
                __builtin_amdgcn_sched_barrier(0);
            }
        }
        const unsigned un = __builtin_amdgcn_readfirstlane(tk) - qbase;
        {
            const unsigned long long m = __ballot(fv >= call);
            rdy |= (unsigned)m & (unsigned)(m >> 32);
        }
        tk = (unsigned)__builtin_amdgcn_raw_ptr_buffer_atomic_add_i32(1, qctr, lane * 4, 0, 0);
        {
            const unsigned oa = (un >> 5) + 3;
            fv = __builtin_amdgcn_raw_buffer_load_b32(rfo, ((rdy >> ptile(oa < 31 ? oa : 31)) & (rdy >> ptile(oa < 32 ? oa - 1 : 31)) & 1u) ? 256 : lane * 4, 0, SAUX);
        }
        const __amdgpu_buffer_rsrc_t rd = mk(ob + (size_t)((tile >> 4) * 256 + (part >> 1) * 16) * WP + (tile & 15) * 512 + (part & 1) * 256, 15 * WP + 256);
        const unsigned tn = ptile((un >> 5) & 31), on = tn * 32 + (un & 31);
        pre = un < NPULL && ((rdy >> tn) & 1u);
#pragma unroll
        for (int k = 0; k < 4; ++k) {
            if (k + DP == 4) rs = mk(dps + (size_t)on * 4096, pre && !bad ? 4096 : 0);
            if (k + DD == 4) rsd = mk(dds + (size_t)on * 8192, pre ? 8192 : 0);
            asm volatile("" : "+v"(vp[k]), "+v"(vd[k][0]), "+v"(vd[k][1]));
            __builtin_amdgcn_sched_barrier(0);
            LOAD_P((k + DP) & 3);
            LOAD_D((k + DD) & 3);
            __builtin_amdgcn_sched_barrier(0);
            FINISH(k);
            __builtin_amdgcn_sched_barrier(0);
        }
        u = un;
    }
#undef LOAD_P
#undef FINISH
#undef LOAD_D
}

__device__ __forceinline__ void gemm_tile(const Params& P, LAS unsigned char* lds, int mode, int pm, int pn, int qd, unsigned call) {
    const int tid = threadIdx.x, wid = __builtin_amdgcn_readfirstlane(tid >> 6), lane = tid & 63, wr = wid >> 2, wc = wid & 3, fr = lane & 15, fq = lane >> 4;
    const int g = P.rank;
    char* myws = P.ws[0];
    unsigned voffA[2], voffB[2];
#pragma unroll
    for (int i = 0; i < 2; ++i) {
        int R, C; stage_rc(tid * 16 + i * 8192, R, C); voffA[i] = (unsigned)R * XP + (unsigned)C * 2u;
        const int row = lane >> 3, ch = (lane & 7) ^ (((row >> 1) & 1) << 2);
        voffB[i] = (unsigned)(((wid >> 1) + 4 * i) * 8 + row) * WP + (unsigned)(wid & 1) * 128u + (unsigned)ch * 16u;
    }
    const unsigned ldsw = (unsigned)wid * 1024u, ldswb = (unsigned)((wid >> 2) * KBP + ((wid >> 1) & 1) * 2056 + (wid & 1) * 1024);
    const int aoff = lds_byte(wr * 64 + fr, fq * 8);
    const int boff = (fq >> 1) * KBP + (fq & 1) * 2056 + (wc >> 1) * 1024 + (fr >> 2) * 128 + ((4 * (wc & 1) + (fr & 3)) ^ (((fr >> 3) & 1) << 2)) * 16;
    constexpr size_t hA = (size_t)128 * XP, hB = 256;
    const int nt = mode == 0 ? 64 : (mode == 1 ? 56 : 8);
    const char* xl = P.x + (size_t)pm * 256 * XP;
    const char* wl = P.w + (size_t)(mode == 2 ? (g + qd) & 7 : g) * NB * 2 + pn * 512;
    const char* xs = myws + OFF_XS + (size_t)pm * 256 * XP;
    const char* wsb = myws + OFF_WS + pn * 512;
    auto ktile = [&](unsigned id, const char*& a, const char*& b) {
        const unsigned c = id >> 3, q = id & 7;
        a = (q ? xs : xl) + (q * (GM * XP) + c * 128); b = (q ? wsb : wl) + (c * (64 * WP) + q * (NB * 2));
    };
#define SA(b, h) (((b) * 2 + (h)) * HTB)
#define SB(b, h) (4 * HTB + ((b) * 2 + (h)) * HTBB)
#define STAGEB(bufoff, gbase) do { _Pragma("unroll") for (int _i = 0; _i < 2; ++_i) \
        __builtin_amdgcn_global_load_lds((const unsigned*)((const char*)(gbase) + voffB[_i]), (LAS unsigned*)(lds + (bufoff) + ldswb + _i * 2 * KBP), 16, 0, 0); } while (0)
#define STAGE(bufoff, gbase, voff) do { _Pragma("unroll") for (int _i = 0; _i < 2; ++_i) \
        __builtin_amdgcn_global_load_lds((const unsigned*)((const char*)(gbase) + (voff)[_i]), (LAS unsigned*)(lds + (bufoff) + ldsw + _i * 8192), 16, 0, 0); } while (0)
#define LDA(dst, b, h) do { _Pragma("unroll") for (int m = 0; m < 4; ++m) _Pragma("unroll") for (int k = 0; k < 2; ++k) dst[m][k] = *(const LAS bf16x8*)(lds + SA(b, h) + aoff + m * 2048 + k * 1024); } while (0)
#define TRD(dst, off) asm volatile("ds_read_b64_tr_b16 %0, %1 offset:%2" : "=v"(dst) : "v"(baddr), "i"(off) : "memory")
#define LDB(dst, b, h) do { _Pragma("unroll") for (int n = 0; n < 2; ++n) _Pragma("unroll") for (int k = 0; k < 2; ++k) { \
        TRD(dst[n][k][0], ((b) * 2 + (h)) * HTBB + k * 2 * KBP + n * 8); TRD(dst[n][k][1], ((b) * 2 + (h)) * HTBB + k * 2 * KBP + 512 + n * 8); } } while (0)
#define WAIT_B(Bt) asm volatile("s_waitcnt lgkmcnt(0)" : "+v"(Bt[0][0][0]), "+v"(Bt[0][0][1]), "+v"(Bt[0][1][0]), "+v"(Bt[0][1][1]), "+v"(Bt[1][0][0]), "+v"(Bt[1][0][1]), "+v"(Bt[1][1][0]), "+v"(Bt[1][1][1]) :: "memory")
#define BF(Bt, n, k) __builtin_shufflevector(Bt[n][k][0], Bt[n][k][1], 0, 1, 2, 3, 4, 5, 6, 7)
#define MMA(ai, bj, At, Bt) do { __builtin_amdgcn_s_setprio(1); _Pragma("unroll") for (int m = 0; m < 4; ++m) _Pragma("unroll") for (int n = 0; n < 2; ++n) _Pragma("unroll") for (int k = 0; k < 2; ++k) \
        acc[ai][bj][m][n] = __builtin_amdgcn_mfma_f32_16x16x32_bf16(BF(Bt, n, k), At[m][k], acc[ai][bj][m][n], 0, 0, 0); __builtin_amdgcn_s_setprio(0); } while (0)
#define WAIT_V(n) asm volatile("s_waitcnt vmcnt(" #n ")" ::: "memory")
#define WAIT_L(n) asm volatile("s_waitcnt lgkmcnt(" #n ")" ::: "memory")
#define BAR __builtin_amdgcn_s_barrier()
#define SCHED __builtin_amdgcn_sched_barrier(0)
    f32x4 acc[2][2][4][2];
#pragma unroll
    for (int a = 0; a < 2; ++a)
#pragma unroll
        for (int b = 0; b < 2; ++b)
#pragma unroll
            for (int m = 0; m < 4; ++m)
#pragma unroll
                for (int n = 0; n < 2; ++n) acc[a][b][m][n] = (f32x4){0.f, 0.f, 0.f, 0.f};
    bf16x8 At[4][2]; s16x4 B0[2][2][2], B1[2][2][2];
    const unsigned baddr = (unsigned)(size_t)lds + 4 * HTB + boff;
    constexpr unsigned long long LOCAL = 0x0101010101010101ull;
    unsigned long long rdy = LOCAL;
    unsigned long long todo = (mode == 0 ? ~0ull : (mode == 1 ? ~(LOCAL << qd) : LOCAL)) & ~0x101ull;
    LAS unsigned long long* rdy_lds = (LAS unsigned long long*)(lds + STAGE_BYTES);
    STAGEB(SB(0, 0), wl); STAGE(SA(0, 0), xl, voffA); STAGEB(SB(0, 1), wl + hB); STAGE(SA(0, 1), xl + hA, voffA);
    if (wr == 1) BAR;
    WAIT_V(4); BAR;
    STAGEB(SB(1, 0), wl + (size_t)64 * WP); STAGE(SA(1, 0), xl + 128, voffA); STAGEB(SB(1, 1), wl + (size_t)64 * WP + hB);
    WAIT_V(6); BAR;
    const char* a1 = xl + 128;
    for (int t = 0; t < nt; t += 2) {
        if (todo != 0ull && __builtin_popcountll(todo & rdy) < 2) {
            if (wid == 0) {
                const __amdgpu_buffer_rsrc_t rfx = mk(myws + OFF_FX + pm * 1024, 1024), rfw = mk(myws + OFF_FW + pn * 1024, 1024);
                const unsigned long long t0 = __builtin_amdgcn_s_memrealtime();
                unsigned long long mask = rdy;
                const unsigned long long pend = todo & ~rdy, pend2 = pend & (pend - 1ull);
                const int l1 = (__builtin_ctzll(pend2 ? pend2 : pend) | 7) + 8;
                unsigned vo = (pend >> lane) & 1ull ? lane * 16 : 4096;
                for (unsigned polls = 1;; ++polls) {
                    const u32x4 vx = __builtin_amdgcn_raw_buffer_load_b128(rfx, vo, 0, SAUX), vw = __builtin_amdgcn_raw_buffer_load_b128(rfw, vo, 0, SAUX);
                    asm volatile("" ::: "memory");
                    const bool ok = min(min(vx.x, vx.y), min(vx.z, vx.w)) >= call && min(min(vw.x, vw.y), min(vw.z, vw.w)) >= call;
                    mask |= __ballot(ok);
                    if (__builtin_popcountll(mask & todo) >= 2) break;
                    if ((polls & 7u) == 0u && give_up((unsigned*)(myws + OFF_TMO), call, t0)) { mask = ~0ull; break; }
                    if (lane > l1) vo = 4096;
                    __builtin_amdgcn_s_sleep(16);
                }
                __builtin_amdgcn_fence(__ATOMIC_ACQUIRE, "");
                if (lane == 0) *rdy_lds = mask;
                asm volatile("s_waitcnt vmcnt(0) lgkmcnt(0)" ::: "memory");
            }
            asm volatile("" ::: "memory"); BAR; asm volatile("" ::: "memory");
            const unsigned long long mv = *rdy_lds;
            rdy = ((unsigned long long)(unsigned)__builtin_amdgcn_readfirstlane((unsigned)(mv >> 32)) << 32) | (unsigned)__builtin_amdgcn_readfirstlane((unsigned)mv);
        }
        LDB(B0, 0, 0); SCHED; LDA(At, 0, 0); STAGE(SA(1, 1), a1 + hA, voffA); SCHED;
        unsigned long long avail = todo ? todo & rdy : 0x101ull;
        const unsigned id2 = __builtin_ctzll(avail); avail &= avail - 1ull;
        const unsigned id3 = __builtin_ctzll(avail);
        todo &= ~((1ull << id2) | (1ull << id3));
        const char *a2, *b2, *a3, *b3;
        ktile(id2, a2, b2); ktile(id3, a3, b3);
        WAIT_L(8); BAR; WAIT_B(B0); MMA(0, 0, At, B0); BAR; SCHED;
        LDB(B1, 0, 1); STAGEB(SB(0, 0), b2);
        BAR; WAIT_B(B1); MMA(0, 1, At, B1); BAR;
        LDA(At, 0, 1); STAGE(SA(0, 0), a2, voffA);
        BAR; WAIT_L(0); MMA(1, 0, At, B0); BAR; SCHED;
        STAGEB(SB(0, 1), b2 + hB);
        WAIT_V(6); BAR; MMA(1, 1, At, B1); BAR;
        LDB(B0, 1, 0); SCHED; LDA(At, 1, 0); STAGE(SA(0, 1), a2 + hA, voffA);
        WAIT_L(8); BAR; WAIT_B(B0); MMA(0, 0, At, B0); BAR; SCHED;
        LDB(B1, 1, 1); STAGEB(SB(1, 0), b3);
        BAR; WAIT_B(B1); MMA(0, 1, At, B1); BAR;
        LDA(At, 1, 1); STAGE(SA(1, 0), a3, voffA);
        BAR; WAIT_L(0); MMA(1, 0, At, B0); BAR; SCHED;
        STAGEB(SB(1, 1), b3 + hB);
        WAIT_V(6); BAR; MMA(1, 1, At, B1); BAR;
        a1 = a3;
    }
    if (wr == 0) BAR;
    const bool bad = __builtin_amdgcn_readfirstlane(ld_agent((const unsigned*)(myws + OFF_TMO))) == call;
    int tz = threadIdx.x; asm volatile("" : "+v"(tz));
    const int l2 = tz & 63;
    if (mode == 2) {
        const __amdgpu_buffer_rsrc_t ro = mk(myws + OFF_DS + (size_t)(qd * 32 + (pm & 1) * 16 + pn) * 262144, 262144);
        const unsigned vo = (unsigned)(wr * 4 * 16384 + wc * 2048 + l2 * 16);
#pragma unroll
        for (int ai = 0; ai < 2; ++ai)
#pragma unroll
            for (int m = 0; m < 4; ++m)
#pragma unroll
                for (int bj = 0; bj < 2; ++bj)
#pragma unroll
                    for (int n = 0; n < 2; ++n)
                        __builtin_amdgcn_raw_buffer_store_b128(__builtin_bit_cast(u32x4, acc[ai][bj][m][n]), ro, vo + (unsigned)((ai * 8 + m) * 16384 + bj * 8192 + n * 1024), 0, SAUX);
    } else if (mode == 1) {
        const __amdgpu_buffer_rsrc_t ro = mk(myws + OFF_PS + (call & 1) * PS_PAR + (size_t)(pm * 16 + pn) * 131072, 131072);
        const unsigned vo = (unsigned)(wr * 4 * 8192 + wc * 1024 + l2 * 16);
#pragma unroll
        for (int ai = 0; ai < 2; ++ai)
#pragma unroll
            for (int m = 0; m < 4; ++m)
#pragma unroll
                for (int bj = 0; bj < 2; ++bj) {
                    const f32x4 v0 = acc[ai][bj][m][0], v1 = acc[ai][bj][m][1];
                    u32x4 o = {pk_bf16(v0[0], v0[1]), pk_bf16(v0[2], v0[3]), pk_bf16(v1[0], v1[1]), pk_bf16(v1[2], v1[3])};
                    if (bad) o = (u32x4){QNAN2, QNAN2, QNAN2, QNAN2};
                    __builtin_amdgcn_raw_buffer_store_b128(o, ro, vo + (unsigned)((ai * 8 + m) * 8192 + bj * 4096), 0, SAUX);
                }
    } else {
        const __amdgpu_buffer_rsrc_t ro = mk(P.out + (size_t)(pm & 1) * 256 * WP + (size_t)g * NB * 2 + pn * 512, 255 * WP + 512);
        const unsigned vo = (unsigned)(wr * 64 + (l2 & 15)) * WP + (unsigned)(wc * 32 + 8 * (l2 >> 4)) * 2u;
#pragma unroll
        for (int ai = 0; ai < 2; ++ai)
#pragma unroll
            for (int m = 0; m < 4; ++m)
#pragma unroll
                for (int bj = 0; bj < 2; ++bj) {
                    const f32x4 v0 = acc[ai][bj][m][0], v1 = acc[ai][bj][m][1];
                    u32x4 o = {pk_bf16(silu(v0[0]), silu(v0[1])), pk_bf16(silu(v0[2]), silu(v0[3])), pk_bf16(silu(v1[0]), silu(v1[1])), pk_bf16(silu(v1[2]), silu(v1[3]))};
                    if (bad) o = (u32x4){QNAN2, QNAN2, QNAN2, QNAN2};
                    __builtin_amdgcn_raw_buffer_store_b128(o, ro, vo + (unsigned)(ai * 8 + m) * 16 * WP + bj * 256, 0, SAUX);
                }
    }
    WAIT_V(0);
    BAR;
    if (tid == 0) {
        if (mode == 1) st_sys(P.ws[qd] + OFF_FO + ((8 - qd) * 64 + (pm & 1) * 16 + pn) * 4, call);
        if (mode == 2) st_sys(myws + OFF_FO + (qd * 64 + 32 + (pm & 1) * 16 + pn) * 4, call);
        else if (__hip_atomic_fetch_add((unsigned*)(myws + OFF_GDC), 1u, __ATOMIC_RELAXED, __HIP_MEMORY_SCOPE_AGENT) == 256u * call - 1u) {
            for (int o = 1; o < 8; ++o) st_sys(P.ws[o] + OFF_GD + (8 - o) * 128, call);
        }
    }
#undef SA
#undef SB
#undef STAGE
#undef STAGEB
#undef LDA
#undef LDB
#undef TRD
#undef WAIT_B
#undef BF
#undef MMA
}

__global__ __launch_bounds__(512, 2) void fused_gemm_rs(const Params P) {
    extern __shared__ __attribute__((aligned(16))) unsigned char shm[];
    const int b = blockIdx.x, xcd = b & 7, slot = b >> 3;
    asm volatile("" :: "s"(P.ws[0]), "s"(P.rank), "s"(P.sched[10]), "s"(P.sched[26]), "s"(P.sched[42]), "s"(P.sched[58]));
    unsigned* cnt = (unsigned*)(P.ws[0] + OFF_CNT + (size_t)b * 128);
    const unsigned cntv = ld_agent(cnt);
    unsigned pend = 0;
    int pw = -1;
    if (slot < CPX) {
        const unsigned call = __builtin_amdgcn_readfirstlane(cntv) + 1u;
        asm volatile("s_waitcnt vmcnt(0)" ::: "memory");
        __builtin_amdgcn_s_barrier();
        if (threadIdx.x == 448) __hip_atomic_store(cnt, call, __ATOMIC_RELAXED, __HIP_MEMORY_SCOPE_AGENT);
        pend = in_role(P, b, call);
        pw = NPW0 + b;
    } else {
        const int g = P.rank, s = slot - CPX;
        const int cl = s / 7, pcl = 2;
        const bool late = cl != 0 && cl != pcl;
        const int e = ((late && cl == 3 ? 7 : 0) + s % 7) * 8 + xcd;
        static_assert(NP0 <= 4, "pushed by the two early classes");
        { const int e2 = (late ? 112 : (cl == 0 ? 56 : 0)) + e; quick_role(P, e2 % 7 + 1, e2 / 7, cntv); }
        const unsigned call = __builtin_amdgcn_readfirstlane(cntv) + 1u;
        if (late) copy_role(P, e % 7 + 1, NCOMM / 4 + e / 7, call);
        for (int u = (late ? 0 : 2); u < (s < 4 ? 4 : 3); ++u) {
            int mode, pm, pn, qd;
            if (u < 2) { mode = 2; pm = 2 * g + u; pn = e / 7; qd = e % 7 + 1; }
            else if (u == 2) { mode = 1; qd = s % 7 + 1; pm = 2 * ((g + qd) & 7) + (cl >> 1); pn = xcd + 8 * (cl & 1); }
            else { mode = 0; qd = 0; pm = 2 * g + (s >> 1); pn = xcd + 8 * (s & 1); }
            gemm_tile(P, (LAS unsigned char*)shm, mode, pm, pn, qd, call);
        }
        if (threadIdx.x == 448) __hip_atomic_store(cnt, call, __ATOMIC_RELAXED, __HIP_MEMORY_SCOPE_AGENT);
        if (s >= 4 && s < 7) pw = (s - 4) * 8 + xcd;
        if (s == 7 * pcl) pw = NPW0 + NCOMM + xcd;
    }
    if (pw >= 0) pull_role(P, pw, __builtin_amdgcn_readfirstlane(cntv) + 1u, pend);
}

void build_schedule(unsigned* sched) {
    unsigned char e[240]; int n = 0;
    auto W = [&](int j0, int j1, int c) { for (int j = j0; j < j1; ++j) e[n++] = (unsigned char)((j << 3) | c); };
    auto X = [&](int u0, int u1, int c) { for (int u = u0; u < u1; ++u) e[n++] = (unsigned char)(128 | (u << 3) | c); };
    for (int c = 0; c < 8; ++c) { X(2, 8, c); W(0, 8, c); if (c == 4) X(8, 14, 0); if (c == 6) X(8, 14, 1); }
    for (int c = 0; c < 6; ++c) { X(8, 14, c + 2); W(8, 16, c); if (c < 6 - (int)NPL / 2) X(0, 2, c); if (c == 3) X(0, 2, 6); }
    W(8, 16, 6); W(8, 16, 7);
    for (int c = 6 - (int)NPL / 2; c < 6; ++c) X(0, 2, c);
    X(0, 2, 7);
    for (int i = 0; i < 60; ++i) sched[i] = e[4 * i] | ((unsigned)e[4 * i + 1] << 8) | ((unsigned)e[4 * i + 2] << 16) | ((unsigned)e[4 * i + 3] << 24);
}
}

extern "C" void kernel_launch(int rank, int nranks, ncclComm_t comm, hipStream_t stream,
                              void* const* d_in, const int* in_sizes, int n_in,
                              void* d_out, int out_size, const int* in_dtypes, int out_dtype,
                              void* const* d_out_all, void* const* d_ws_all, size_t ws_bytes) {
    if (nranks != NR || (unsigned)rank >= (unsigned)NR || n_in < 2 || ws_bytes < WS_NEED ||
        in_sizes[0] != GM * KS || in_sizes[1] != KS * GN || out_size != MS * GN) {
        fprintf(stderr, "kernel_launch: unexpected problem shape\n");
        return;
    }
    HIP_CHECK(hipFuncSetAttribute((const void*)fused_gemm_rs, hipFuncAttributeMaxDynamicSharedMemorySize, LDS_BYTES));
    Params P;
    P.x = (const char*)d_in[0]; P.w = (const char*)d_in[1];
    for (int p = 0; p < NR; ++p) P.ws[p] = (char*)d_ws_all[(rank + p) & 7];
    P.out = (char*)d_out;
    P.rank = rank;
    build_schedule(P.sched);
    hipLaunchKernelGGL(fused_gemm_rs, dim3(GRID), dim3(512), LDS_BYTES, stream, P);
    HIP_CHECK(hipGetLastError());
}
```
